```python
import numpy as np

kernel_name = "gpu_ag_gemm_m4096_k4096_n4096_e4m3e4m3_dqsilu"


ABSMAX_RTOL = 0.03

COMPUTE_DTYPE = "float32"
EPILOGUE = "dequant_silu"
IN_QUANT = None
OUT_QUANT = None
IN_FP8 = "e4m3e4m3"
IN_DTYPES = {"x": "e4m3", "w_mat": "e4m3",
             "scale_x": "float32", "scale_w": "float32"}
WORLD_SIZE = 8

def _snap_fp8(x, mant_bits, max_abs, min_normal):
    a = np.abs(x.astype(np.float64))
    _m, _e = np.frexp(a)
    q = np.where(a >= min_normal, np.ldexp(1.0, _e - 1 - mant_bits),
                 min_normal * 2.0 ** -mant_bits)
    snapped = np.minimum(np.round(a / q) * q, max_abs)
    return (np.sign(x) * snapped).astype(np.float32)

def setup_inputs():

    x = _snap_fp8(np.random.randn(4096, 4096).astype(np.float32), 3, 448.0, 2.0 ** -6)
    w_mat = _snap_fp8(np.random.randn(4096, 4096).astype(np.float32), 3, 448.0, 2.0 ** -6)
    scale_x = (0.02 * (1.0 + np.random.rand(1))).astype(np.float32)
    scale_w = (0.02 * (1.0 + np.random.rand(1))).astype(np.float32)
    return {"x": x, "w_mat": w_mat, "scale_x": scale_x, "scale_w": scale_w}

def reference(x, w_mat, scale_x, scale_w):
    acc = (x @ w_mat).astype(np.float32)
    y = (acc * (scale_x * scale_w)).astype(np.float32)
    return (y / (1.0 + np.exp(-np.clip(y, -60.0, 60.0)))).astype(np.float32)
```

```cpp
#include <hip/hip_runtime.h>
#include <rccl/rccl.h>
#include <cstdint>
#include <cstdio>

#define HIP_CHECK(expr) do { hipError_t _e = (expr); if (_e != hipSuccess) { \
    fprintf(stderr, "HIP error %s at %s:%d\n", hipGetErrorString(_e), __FILE__, __LINE__); \
    return; } } while (0)

typedef unsigned u32x4 __attribute__((ext_vector_type(4)));
typedef int i32x2 __attribute__((ext_vector_type(2)));
typedef int i32x8 __attribute__((ext_vector_type(8)));
typedef float f32x4 __attribute__((ext_vector_type(4)));
#define LAS __attribute__((address_space(3)))

#define NPROD 64
#define PACE 14
#define NB 8
#define PUSH_AUX 17
#define POLL_SLEEP 4

constexpr int R = 8;
constexpr int MS = 512, KD = 4096, NL = 512;
constexpr unsigned SHARD = MS * KD;
constexpr size_t SLOT = (size_t)R * SHARD;
constexpr int NSLOT = 3;
constexpr size_t CTRL = NSLOT * SLOT;
constexpr size_t CNT_OFF = CTRL;
constexpr size_t TMO_OFF = CTRL + 32768;
constexpr size_t DBG_OFF = CTRL + 65536;
constexpr size_t WS_NEED = CTRL + 65536 + 256 * 5 * 128;
constexpr int GRID = 256, BLOCK = 320;
constexpr int LDS_B = KD * 32;
constexpr int LDS_BYTES = LDS_B + 64;
constexpr int NFRAG = SHARD / 2048;
constexpr unsigned long long DEADLINE = 150000000ull;
constexpr int AUX_SYS = 17;

struct Params {
  const uint8_t* x;
  const uint8_t* w;
  const float* sx;
  const float* sw;
  float* out;
  uint8_t* myws;
  uint8_t* peer[R - 1];
  int rank;
  int pad;
};

__device__ __forceinline__ __amdgpu_buffer_rsrc_t mk_rsrc(const void* p, unsigned bytes) {
  return __builtin_amdgcn_make_buffer_rsrc((void*)p, (short)0, (int)bytes, 0x00020000);
}
__device__ __forceinline__ unsigned long long now() { return __builtin_amdgcn_s_memrealtime(); }
__device__ __forceinline__ unsigned nz(unsigned v) { return v ? v : 0x80808080u; }
__device__ __forceinline__ unsigned umin3(unsigned a, unsigned b, unsigned c) { return min(min(a, b), c); }

#define STAMP(i) do { } while (0)

__global__ void __launch_bounds__(BLOCK, 2) ag_gemm_kernel(Params P) {
  extern __shared__ __attribute__((aligned(16))) uint8_t lds[];
  const int lane = threadIdx.x & 63;
  const int wave = __builtin_amdgcn_readfirstlane(threadIdx.x >> 6);
  const int b = blockIdx.x;
  const int rank = P.rank;
  uint8_t* const myws = P.myws;
  volatile LAS unsigned* sync = (volatile LAS unsigned*)(lds + LDS_B);
  const unsigned long long t0 = now();

  if (wave == 4) {
    unsigned* cntp = (unsigned*)(myws + CNT_OFF + b * 64);
    const unsigned prev = __hip_atomic_load(cntp, __ATOMIC_RELAXED, __HIP_MEMORY_SCOPE_AGENT);
    if (b >= NPROD) {
      __builtin_amdgcn_s_barrier();
      __builtin_amdgcn_sched_barrier(0);
      const unsigned call = __builtin_amdgcn_readfirstlane(prev) + 1;
      if (lane == 0) {
        sync[1] = call;
        __hip_atomic_store(cntp, call, __ATOMIC_RELAXED, __HIP_MEMORY_SCOPE_AGENT);
      }
      return;
    }
    constexpr int FPW = NFRAG / NPROD;
    u32x4 d[FPW][2];
    const __amdgpu_buffer_rsrc_t xr = mk_rsrc(P.x, SHARD);
    const unsigned vo = (lane & 15) * KD + (lane >> 4) * 16;
#pragma unroll
    for (int i = 0; i < FPW; ++i) {
      const unsigned f = i * NPROD + b;
      const unsigned so = ((f >> 8) * 128 + (f & 7) * 16) * KD + ((f >> 3) & 31) * 128;
      d[i][0] = __builtin_amdgcn_raw_buffer_load_b128(xr, vo, so, 0);
      d[i][1] = __builtin_amdgcn_raw_buffer_load_b128(xr, vo + 64, so, 0);
    }
    __amdgpu_buffer_rsrc_t pr[R - 1];
#pragma unroll
    for (int j = 0; j < R - 1; ++j) pr[j] = mk_rsrc(P.peer[j], (unsigned)((NSLOT - 1) * SLOT + SHARD));
    __builtin_amdgcn_s_barrier();
    __builtin_amdgcn_sched_barrier(0);
    const unsigned call = __builtin_amdgcn_readfirstlane(prev) + 1;
    if (lane == 0) sync[1] = call;
    const unsigned dst = (call % NSLOT) * (unsigned)SLOT + b * 2048;
    STAMP(1);
#pragma unroll
    for (int i = 0; i < FPW; ++i) {
      const unsigned so = dst + i * NPROD * 2048;
#pragma unroll
      for (int h = 0; h < 2; ++h) {
        u32x4 v = d[i][h];
        v.x = nz(v.x); v.y = nz(v.y); v.z = nz(v.z); v.w = nz(v.w);
        if (i * 2 + h > 0) asm volatile("s_waitcnt vmcnt(%0)" :: "n"(PACE) : "memory");
#pragma unroll
        for (int j = 0; j < R - 1; ++j)
          __builtin_amdgcn_raw_buffer_store_b128(v, pr[j], lane * 16 + h * 1024, so, PUSH_AUX);
        __builtin_amdgcn_sched_barrier(0);
      }
    }
    if (lane == 0) __hip_atomic_store(cntp, call, __ATOMIC_RELAXED, __HIP_MEMORY_SCOPE_AGENT);
    STAMP(2);
    asm volatile("s_waitcnt vmcnt(0)" ::: "memory");
    STAMP(3);
    return;
  }

  const int s = b & 7;
  const int hf = (b >> 7) & 1;
  const int nj = (b >> 3) & 15;
  if (wave == 0) {
    if (lane == 0) { sync[0] = 0; sync[1] = 0; }
    asm volatile("s_waitcnt lgkmcnt(0)" ::: "memory");
  }
  __builtin_amdgcn_s_barrier();
  {
    const uint8_t* wsrc = P.w + (size_t)(lane >> 1) * NL + nj * 32 + (((lane & 1) ^ ((lane >> 5) & 1)) << 4);
#pragma unroll
    for (int i = 0; i < 32; ++i) {
      const int kblk = i * 4 + wave;
      __builtin_amdgcn_global_load_lds((const __attribute__((address_space(1))) void*)(wsrc + (size_t)kblk * 32 * NL),
                                       (LAS void*)(lds + kblk * 1024), 16, 0, 0);
    }
  }
  const float scale = P.sx[0] * P.sw[0];
  asm volatile("s_waitcnt vmcnt(0)" ::: "memory");
  if (lane == 0) __hip_atomic_fetch_add((LAS unsigned*)sync, 1u, __ATOMIC_RELAXED, __HIP_MEMORY_SCOPE_WORKGROUP);
  unsigned call;
  for (;;) {
    const unsigned c0 = sync[0];
    call = sync[1];
    if (__builtin_amdgcn_readfirstlane(c0) == 4 && __builtin_amdgcn_readfirstlane(call) != 0) break;
    __builtin_amdgcn_s_sleep(1);
  }
  call = __builtin_amdgcn_readfirstlane(call);
  asm volatile("" ::: "memory");
  STAMP(1);

  const bool local = (s == rank);
  const uint8_t* abase = local ? P.x : myws + (size_t)(call % NSLOT) * SLOT + (size_t)s * SHARD;
  const __amdgpu_buffer_rsrc_t ar = mk_rsrc(abase, SHARD);
  const unsigned rb = hf * 4 + wave;
  const unsigned vo0 = local ? (rb * 16 + (lane & 15)) * KD + (lane >> 4) * 16 : rb * 2048 + lane * 16;
  const unsigned vo1 = vo0 + (local ? 64 : 1024);
  const unsigned kstr = local ? 128 : 16384;
  const unsigned pollo = rb * 2048 + (lane & 31) * 16384 + 2044;
  unsigned* tmo = (unsigned*)(myws + TMO_OFF);
  const int q = lane >> 4, li = lane & 15;
  const unsigned lb0 = (q * 16 + (li >> 1)) * 32 + ((q & 1) << 4) + (li & 1) * 8;
  const unsigned lb1 = lb0 ^ 16;
  bool dead = false;

  for (int p = 0; p < 4; ++p) {
    const unsigned po = p * (SHARD / 4);
    f32x4 acc0 = {0.f, 0.f, 0.f, 0.f}, acc1 = {0.f, 0.f, 0.f, 0.f};
    unsigned pending = 0xffffffffu;
    while (pending && !dead) {
      unsigned ready = pending;
      asm volatile("" ::: "memory");
      if (!local && __builtin_popcount(pending) > NB) {
        const unsigned v = __builtin_amdgcn_raw_buffer_load_b32(ar, pollo, po, AUX_SYS);
        ready &= (unsigned)__builtin_amdgcn_ballot_w64(v != 0);
      }
      const unsigned before = pending;
      if (ready) {
        u32x4 a[NB][2];
        unsigned kb[NB];
#pragma unroll
        for (int i = 0; i < NB; ++i) {
          kb[i] = ready ? __builtin_ctz(ready) : 32u;
          ready &= ready - 1;
          if (kb[i] < 32) {
            a[i][0] = __builtin_amdgcn_raw_buffer_load_b128(ar, vo0, po + kb[i] * kstr, AUX_SYS);
            a[i][1] = __builtin_amdgcn_raw_buffer_load_b128(ar, vo1, po + kb[i] * kstr, AUX_SYS);
          }
        }
#pragma unroll
        for (int i = 0; i < NB; ++i) {
          if (kb[i] < 32) {
            const unsigned m = min(umin3(a[i][0].x, a[i][0].y, a[i][0].z),
                                   umin3(umin3(a[i][0].w, a[i][1].x, a[i][1].y), a[i][1].z, a[i][1].w));
            if (local || __builtin_amdgcn_ballot_w64(m == 0) == 0) {
              const LAS uint8_t* l0 = (const LAS uint8_t*)lds + kb[i] * 4096 + lb0;
              const LAS uint8_t* l1 = (const LAS uint8_t*)lds + kb[i] * 4096 + lb1;
              i32x8 w0, w1, xa;
#pragma unroll
              for (int j = 0; j < 4; ++j) {
                const int off = (j >> 1) * 2048 + (j & 1) * 256;
                const i32x2 r0 = __builtin_amdgcn_ds_read_tr8_b64_v2i32((LAS i32x2*)(l0 + off));
                const i32x2 r1 = __builtin_amdgcn_ds_read_tr8_b64_v2i32((LAS i32x2*)(l1 + off));
                w0[2 * j] = r0[0]; w0[2 * j + 1] = r0[1];
                w1[2 * j] = r1[0]; w1[2 * j + 1] = r1[1];
              }
              xa[0] = a[i][0].x; xa[1] = a[i][0].y; xa[2] = a[i][0].z; xa[3] = a[i][0].w;
              xa[4] = a[i][1].x; xa[5] = a[i][1].y; xa[6] = a[i][1].z; xa[7] = a[i][1].w;
              acc0 = __builtin_amdgcn_mfma_scale_f32_16x16x128_f8f6f4(w0, xa, acc0, 0, 0, 0, 0, 0, 0);
              acc1 = __builtin_amdgcn_mfma_scale_f32_16x16x128_f8f6f4(w1, xa, acc1, 0, 0, 0, 0, 0, 0);
              pending &= ~(1u << kb[i]);
            }
          }
        }
      }
      if (pending == before) {
        if (__hip_atomic_load(tmo, __ATOMIC_RELAXED, __HIP_MEMORY_SCOPE_AGENT) == call || now() - t0 > DEADLINE) {
          __hip_atomic_store(tmo, call, __ATOMIC_RELAXED, __HIP_MEMORY_SCOPE_AGENT);
          dead = true;
        }
        __builtin_amdgcn_s_sleep(POLL_SLEEP);
      }
    }
    f32x4 o0, o1;
#pragma unroll
    for (int r = 0; r < 4; ++r) {
      const float y0 = acc0[r] * scale, y1 = acc1[r] * scale;
      const float c0 = fminf(fmaxf(y0, -60.f), 60.f), c1 = fminf(fmaxf(y1, -60.f), 60.f);
      o0[r] = y0 * __builtin_amdgcn_rcpf(1.f + __builtin_amdgcn_exp2f(c0 * -1.44269504088896341f));
      o1[r] = y1 * __builtin_amdgcn_rcpf(1.f + __builtin_amdgcn_exp2f(c1 * -1.44269504088896341f));
      if (dead) { o0[r] = __builtin_nanf(""); o1[r] = o0[r]; }
    }
    const __amdgpu_buffer_rsrc_t orr = mk_rsrc(P.out, (unsigned)(R * MS * NL * 4));
    const unsigned oo = ((rb * 16 + li) * NL + q * 4) * 4;
    const unsigned os = ((s * MS + p * 128) * NL + nj * 32) * 4;
    __builtin_amdgcn_raw_buffer_store_b128(__builtin_bit_cast(u32x4, o0), orr, oo, os, AUX_SYS);
    __builtin_amdgcn_raw_buffer_store_b128(__builtin_bit_cast(u32x4, o1), orr, oo + 64, os, AUX_SYS);
    STAMP(2 + p);
  }

  if (local) {
    const __amdgpu_buffer_rsrc_t zr = mk_rsrc(myws + (size_t)((call + 2) % NSLOT) * SLOT, (unsigned)SLOT);
    const unsigned z = ((b >> 3) * 4 + wave) * 16384 + lane * 16;
    const u32x4 zero = {0u, 0u, 0u, 0u};
    for (int j = 1; j < R; ++j) {
      const unsigned so = ((rank + j) & (R - 1)) * SHARD;
#pragma unroll
      for (int i = 0; i < 16; ++i) __builtin_amdgcn_raw_buffer_store_b128(zero, zr, z + i * 1024, so, AUX_SYS);
    }
  }
  asm volatile("s_waitcnt vmcnt(0)" ::: "memory");
  STAMP(6);
}

extern "C" void kernel_launch(int rank, int nranks, ncclComm_t comm, hipStream_t stream,
                              void* const* d_in, const int* in_sizes, int n_in,
                              void* d_out, int out_size, const int* in_dtypes, int out_dtype,
                              void* const* d_out_all, void* const* d_ws_all, size_t ws_bytes) {
  (void)comm; (void)in_dtypes; (void)out_dtype; (void)d_out_all;
  if (nranks != R || (unsigned)rank >= (unsigned)R || n_in != 4 || ws_bytes < WS_NEED ||
      in_sizes[0] != (int)SHARD || in_sizes[1] != KD * NL || out_size != R * MS * NL) {
    fprintf(stderr, "unexpected problem shape\n");
    return;
  }
  static bool attr_set = false;
  if (!attr_set) {
    HIP_CHECK(hipFuncSetAttribute((const void*)ag_gemm_kernel, hipFuncAttributeMaxDynamicSharedMemorySize, LDS_BYTES));
    attr_set = true;
  }
  Params P;
  P.x = (const uint8_t*)d_in[0];
  P.w = (const uint8_t*)d_in[1];
  P.sx = (const float*)d_in[2];
  P.sw = (const float*)d_in[3];
  P.out = (float*)d_out;
  P.myws = (uint8_t*)d_ws_all[rank];
  for (int j = 0; j < R - 1; ++j) P.peer[j] = (uint8_t*)d_ws_all[(rank + 1 + j) & (R - 1)] + (size_t)rank * SHARD;
  P.rank = rank;
  P.pad = 0;
  hipLaunchKernelGGL(ag_gemm_kernel, dim3(GRID), dim3(BLOCK), LDS_BYTES, stream, P);
  HIP_CHECK(hipGetLastError());
}
```

```cpp
#include <hip/hip_runtime.h>
#include <rccl/rccl.h>
#include <cstdint>
#include <cstdio>

#define HIP_CHECK(expr) do { hipError_t _e = (expr); if (_e != hipSuccess) { \
    fprintf(stderr, "HIP error %s at %s:%d\n", hipGetErrorString(_e), __FILE__, __LINE__); \
    return; } } while (0)

typedef unsigned u32x4 __attribute__((ext_vector_type(4)));
typedef int i32x2 __attribute__((ext_vector_type(2)));
typedef int i32x8 __attribute__((ext_vector_type(8)));
typedef float f32x4 __attribute__((ext_vector_type(4)));
#define LAS __attribute__((address_space(3)))

#define PW 1
#define PACE 16
#define NB 8
#define PUSH_AUX 17
#define POLL_SLEEP 2
#define NPROD 32
#define PSTRIDE 1

constexpr int R = 8;
constexpr int MS = 512, KD = 4096, NL = 512;
constexpr unsigned SHARD = MS * KD;
constexpr size_t SLOT = (size_t)R * SHARD;
constexpr int NSLOT = 3;
constexpr size_t CTRL = NSLOT * SLOT;
constexpr size_t CNT_OFF = CTRL;
constexpr size_t TMO_OFF = CTRL + 32768;
constexpr size_t DBG_OFF = CTRL + 65536;
constexpr int GRID = 256, BLOCK = 512;
constexpr size_t WS_NEED = CTRL + 65536 + GRID * (BLOCK / 64) * 128;
constexpr int LDS_B = KD * 32;
constexpr int LDS_X = LDS_B;
constexpr int LDS_F = LDS_X + 16384;
constexpr int LDS_BYTES = LDS_F + 64;
constexpr int NFRAG = SHARD / 2048;
constexpr unsigned long long DEADLINE = 150000000ull;
constexpr int AUX_SYS = 17;

struct Params {
  const uint8_t* x;
  const uint8_t* w;
  const float* sx;
  const float* sw;
  float* out;
  uint8_t* myws;
  uint8_t* peer[R - 1];
  int rank;
  int pad;
};

__device__ __forceinline__ __amdgpu_buffer_rsrc_t mk_rsrc(const void* p, unsigned bytes) {
  return __builtin_amdgcn_make_buffer_rsrc((void*)p, (short)0, (int)bytes, 0x00020000);
}
__device__ __forceinline__ unsigned long long now() { return __builtin_amdgcn_s_memrealtime(); }
__device__ __forceinline__ unsigned nz(unsigned v) { return v ? v : 0x80808080u; }
__device__ __forceinline__ unsigned umin3(unsigned a, unsigned b, unsigned c) { return min(min(a, b), c); }
__device__ __forceinline__ float silu(float acc, float scale) {
  const float y = acc * scale;
  const float c = fminf(fmaxf(y, -60.f), 60.f);
  return y * __builtin_amdgcn_rcpf(1.f + __builtin_amdgcn_exp2f(c * -1.44269504088896341f));
}
template <int N> __device__ __forceinline__ void wait_vm(u32x4& d) {
  asm volatile("s_waitcnt vmcnt(%1)" : "+v"(d) : "n"(N) : "memory");
}
__device__ __forceinline__ void ld_x(u32x4& d, unsigned off, const uint8_t* base) {
  asm volatile("global_load_dwordx4 %0, %1, %2" : "=v"(d) : "v"(off), "s"(base) : "memory");
}

#define STAMP(i) do { } while (0)

__global__ void __launch_bounds__(BLOCK, 2) ag_gemm_kernel(Params P) {
  extern __shared__ __attribute__((aligned(16))) uint8_t lds[];
  const int lane = threadIdx.x & 63;
  const int wave = __builtin_amdgcn_readfirstlane(threadIdx.x >> 6);
  const int b = blockIdx.x;
  uint8_t* const myws = P.myws;

  if (b < NPROD) {
    if (wave >= PW) return;
    unsigned* cntp = (unsigned*)(myws + CNT_OFF + b * 64 + wave * 4);
    constexpr int NPW = NPROD * PW;
    constexpr int HPW = 2 * NFRAG / NPW;
    constexpr int PF = 8;
    static_assert(HPW % PF == 0 && HPW >= 2 * PF && PACE <= 8 * PF - 8, "");
    const unsigned pid = b * PW + wave;
    const unsigned vo = (lane & 15) * KD + (lane >> 4) * 16;
    auto src = [&](unsigned i) {
      const unsigned f = (i >> 1) * NPW + pid;
      return vo + ((f >> 8) * 128 + (f & 7) * 16) * KD + ((f >> 3) & 31) * 128 + (i & 1) * 64;
    };
    u32x4 c, d[PF];
    asm volatile("global_load_dword %0, %1, %2 sc1" : "=v"(c.x) : "v"(0u), "s"(cntp) : "memory");
#pragma unroll
    for (int u = 0; u < PF; ++u) ld_x(d[u], src(u), P.x);
    __amdgpu_buffer_rsrc_t pr[R - 1];
#pragma unroll
    for (int j = 0; j < R - 1; ++j) pr[j] = mk_rsrc(P.peer[j], (unsigned)((NSLOT - 1) * SLOT + SHARD));
    asm volatile("s_waitcnt vmcnt(%1)" : "+v"(c.x) : "n"(PF) : "memory");
    const unsigned call = __builtin_amdgcn_readfirstlane(c.x) + 1;
    const unsigned dst = (call % NSLOT) * (unsigned)SLOT + pid * 2048;
    STAMP(1);
    auto push = [&](u32x4 v, unsigned i) {
      v.x = nz(v.x); v.y = nz(v.y); v.z = nz(v.z); v.w = nz(v.w);
      const unsigned so = dst + (i >> 1) * (NPW * 2048) + (i & 1) * 1024;
#pragma unroll
      for (int j = 0; j < R - 1; ++j) __builtin_amdgcn_raw_buffer_store_b128(v, pr[j], lane * 16, so, PUSH_AUX);
    };
#define STEP(N, u, i, more) do { wait_vm<(N)>(d[u]); push(d[u], (i)); \
      if (more) ld_x(d[u], src((i) + PF), P.x); __builtin_amdgcn_sched_barrier(0); } while (0)
#define W0(u) ((PF - 1 - (u) + 8 * (u)) < PACE ? (PF - 1 - (u) + 8 * (u)) : PACE)
    STEP(W0(0), 0, 0, true); STEP(W0(1), 1, 1, true); STEP(W0(2), 2, 2, true); STEP(W0(3), 3, 3, true);
    STEP(W0(4), 4, 4, true); STEP(W0(5), 5, 5, true); STEP(W0(6), 6, 6, true); STEP(W0(7), 7, 7, true);
#pragma unroll 1
    for (unsigned i0 = PF; i0 < HPW; i0 += PF) {
      const bool more = i0 + PF < HPW;
      STEP(PACE, 0, i0 + 0, more); STEP(PACE, 1, i0 + 1, more); STEP(PACE, 2, i0 + 2, more); STEP(PACE, 3, i0 + 3, more);
      STEP(PACE, 4, i0 + 4, more); STEP(PACE, 5, i0 + 5, more); STEP(PACE, 6, i0 + 6, more); STEP(PACE, 7, i0 + 7, more);
    }
    if (lane == 0) __hip_atomic_store(cntp, call, __ATOMIC_RELAXED, __HIP_MEMORY_SCOPE_AGENT);
    STAMP(2);
    asm volatile("s_waitcnt vmcnt(0)" ::: "memory");
    STAMP(3);
    return;
  }

  const unsigned long long t0 = now();
  const int rank = P.rank;
  const int idx = b >> 3, lo = (b & 7) * 12 + idx - 20;
  const int set = idx < 20 ? (b & 7) : 8 + (lo >> 4);
  const int nj = idx < 20 ? idx - 4 : (lo & 15);
  const int s = (rank + 1 + (set >> 1)) & (R - 1);
  const int pair = wave >> 1, kpar = wave & 1;
  unsigned* cntp = (unsigned*)(myws + CNT_OFF + b * 64);
  const unsigned prev = __hip_atomic_load(cntp, __ATOMIC_RELAXED, __HIP_MEMORY_SCOPE_AGENT);
  const float scale = P.sx[0] * P.sw[0];
  volatile LAS unsigned* flags = (volatile LAS unsigned*)(lds + LDS_F);
  if (lane == 0) flags[wave] = 0;
  {
    const uint8_t* wsrc = P.w + (size_t)(lane >> 1) * NL + nj * 32 + (((lane & 1) ^ ((lane >> 5) & 1)) << 4);
#pragma unroll
    for (int i = 0; i < 16; ++i) {
      const int kblk = i * 8 + wave;
      __builtin_amdgcn_global_load_lds((const __attribute__((address_space(1))) void*)(wsrc + (size_t)kblk * 32 * NL),
                                       (LAS void*)(lds + kblk * 1024), 16, 0, 0);
    }
  }
  asm volatile("s_waitcnt vmcnt(0) lgkmcnt(0)" ::: "memory");
  __builtin_amdgcn_s_barrier();
  asm volatile("" ::: "memory");
  const unsigned call = __builtin_amdgcn_readfirstlane(prev) + 1;
  if (threadIdx.x == 0) __hip_atomic_store(cntp, call, __ATOMIC_RELAXED, __HIP_MEMORY_SCOPE_AGENT);
  STAMP(1);

  unsigned* tmo = (unsigned*)(myws + TMO_OFF);
  const int q = lane >> 4, li = lane & 15;
  const unsigned lb0 = (q * 16 + (li >> 1)) * 32 + ((q & 1) << 4) + (li & 1) * 8 + kpar * 4096;
  const unsigned lb1 = lb0 ^ 16;
  const __amdgpu_buffer_rsrc_t orr = mk_rsrc(P.out, (unsigned)(R * MS * NL * 4));
  const unsigned rb = (set & 1) * 4 + pair;
  const int lblk = pair < 2 ? set * 2 + pair : (pair == 2 && set < 4 ? 28 + set : -1);
  bool dead = false;
  unsigned seq = 0, spins = 0;

  for (int p = lblk >= 0 ? -1 : 0; p < 4; ++p) {
    const bool local = p < 0;
    const uint8_t* abase = local ? P.x : myws + (size_t)(call % NSLOT) * SLOT + (size_t)s * SHARD;
    const __amdgpu_buffer_rsrc_t ar = mk_rsrc(abase, SHARD);
    const unsigned vo0 = local ? li * KD + q * 16 : lane * 16;
    const unsigned vo1 = vo0 + (local ? 64 : 1024);
    const unsigned kstr = local ? 256 : 32768;
    const unsigned po = local ? lblk * 16 * KD + kpar * 128 : p * (SHARD / 4) + kpar * 16384 + rb * 2048;
    const unsigned orow = local ? rank * MS + lblk * 16 : s * MS + p * 128 + rb * 16;
    f32x4 acc0 = {0.f, 0.f, 0.f, 0.f}, acc1 = {0.f, 0.f, 0.f, 0.f};
    unsigned pending = 0xffffu;
    while (pending && !dead) {
      unsigned ready = pending;
      asm volatile("" ::: "memory");
      if (!local && __builtin_popcount(pending) > NB) {
        const unsigned v = __builtin_amdgcn_raw_buffer_load_b32(ar, li * 32768 + 2044, po, AUX_SYS);
        ready &= (unsigned)__builtin_amdgcn_ballot_w64(v != 0);
      }
      const unsigned before = pending;
      if (ready) {
        u32x4 a[NB][2];
        unsigned kb[NB];
#pragma unroll
        for (int i = 0; i < NB; ++i) {
          kb[i] = ready ? __builtin_ctz(ready) : 32u;
          ready &= ready - 1;
          if (kb[i] < 32) {
            a[i][0] = __builtin_amdgcn_raw_buffer_load_b128(ar, vo0, po + kb[i] * kstr, AUX_SYS);
            a[i][1] = __builtin_amdgcn_raw_buffer_load_b128(ar, vo1, po + kb[i] * kstr, AUX_SYS);
          }
        }
#pragma unroll
        for (int i = 0; i < NB; ++i) {
          if (kb[i] < 32) {
            const LAS uint8_t* l0 = (const LAS uint8_t*)lds + kb[i] * 8192 + lb0;
            const LAS uint8_t* l1 = (const LAS uint8_t*)lds + kb[i] * 8192 + lb1;
            i32x8 w0, w1, xa;
#pragma unroll
            for (int j = 0; j < 4; ++j) {
              const int off = (j >> 1) * 2048 + (j & 1) * 256;
              const i32x2 r0 = __builtin_amdgcn_ds_read_tr8_b64_v2i32((LAS i32x2*)(l0 + off));
              const i32x2 r1 = __builtin_amdgcn_ds_read_tr8_b64_v2i32((LAS i32x2*)(l1 + off));
              w0[2 * j] = r0[0]; w0[2 * j + 1] = r0[1];
              w1[2 * j] = r1[0]; w1[2 * j + 1] = r1[1];
            }
            const unsigned m = min(umin3(a[i][0].x, a[i][0].y, a[i][0].z),
                                   umin3(umin3(a[i][0].w, a[i][1].x, a[i][1].y), a[i][1].z, a[i][1].w));
            if (local || __builtin_amdgcn_ballot_w64(m == 0) == 0) {
              xa[0] = a[i][0].x; xa[1] = a[i][0].y; xa[2] = a[i][0].z; xa[3] = a[i][0].w;
              xa[4] = a[i][1].x; xa[5] = a[i][1].y; xa[6] = a[i][1].z; xa[7] = a[i][1].w;
              acc0 = __builtin_amdgcn_mfma_scale_f32_16x16x128_f8f6f4(w0, xa, acc0, 0, 0, 0, 0, 0, 0);
              acc1 = __builtin_amdgcn_mfma_scale_f32_16x16x128_f8f6f4(w1, xa, acc1, 0, 0, 0, 0, 0, 0);
              pending &= ~(1u << kb[i]);
            }
          }
        }
      }
      if (pending == before) {
        if ((++spins & 63) == 0 &&
            (__hip_atomic_load(tmo, __ATOMIC_RELAXED, __HIP_MEMORY_SCOPE_AGENT) == call || now() - t0 > DEADLINE)) {
          __hip_atomic_store(tmo, call, __ATOMIC_RELAXED, __HIP_MEMORY_SCOPE_AGENT);
          dead = true;
        }
        __builtin_amdgcn_s_sleep(POLL_SLEEP);
      }
    }
    ++seq;
    f32x4 give = kpar ? acc0 : acc1, keep = kpar ? acc1 : acc0;
    if (dead) { give[0] = give[1] = give[2] = give[3] = __builtin_nanf(""); keep = give; }
    *(LAS f32x4*)((LAS uint8_t*)lds + LDS_X + (wave * 2 + (seq & 1)) * 1024 + lane * 16) = give;
    asm volatile("s_waitcnt lgkmcnt(0)" ::: "memory");
    if (lane == 0) flags[wave] = seq;
    while (__builtin_amdgcn_readfirstlane(flags[wave ^ 1]) < seq) { }
    asm volatile("" ::: "memory");
    const f32x4 got = *(const LAS f32x4*)((LAS uint8_t*)lds + LDS_X + ((wave ^ 1) * 2 + (seq & 1)) * 1024 + lane * 16);
    f32x4 o;
#pragma unroll
    for (int r = 0; r < 4; ++r) o[r] = silu(keep[r] + got[r], scale);
    __builtin_amdgcn_raw_buffer_store_b128(__builtin_bit_cast(u32x4, o), orr, (li * NL + q * 4) * 4,
                                           (orow * NL + nj * 32 + kpar * 16) * 4, AUX_SYS);
    if (p == 1) {
      const unsigned g = (set * 16 + nj) * 8 + wave;
      const __amdgpu_buffer_rsrc_t zr = mk_rsrc(myws + (size_t)((call + 2) % NSLOT) * SLOT, (unsigned)SLOT);
      const unsigned so = ((rank + 1 + (g >> 8)) & (R - 1)) * SHARD + (g & 255) * 8192;
      const u32x4 zero = {0u, 0u, 0u, 0u};
#pragma unroll
      for (int i = 0; i < 8; ++i) __builtin_amdgcn_raw_buffer_store_b128(zero, zr, lane * 16 + i * 1024, so, AUX_SYS);
    }
    STAMP(p < 0 ? 7 : 2 + p);
  }
  asm volatile("s_waitcnt vmcnt(0)" ::: "memory");
  STAMP(6);
}

extern "C" void kernel_launch(int rank, int nranks, ncclComm_t comm, hipStream_t stream,
                              void* const* d_in, const int* in_sizes, int n_in,
                              void* d_out, int out_size, const int* in_dtypes, int out_dtype,
                              void* const* d_out_all, void* const* d_ws_all, size_t ws_bytes) {
  (void)comm; (void)in_dtypes; (void)out_dtype; (void)d_out_all;
  if (nranks != R || (unsigned)rank >= (unsigned)R || n_in != 4 || ws_bytes < WS_NEED ||
      in_sizes[0] != (int)SHARD || in_sizes[1] != KD * NL || out_size != R * MS * NL) {
    fprintf(stderr, "unexpected problem shape\n");
    return;
  }
  static bool attr_set = false;
  if (!attr_set) {
    HIP_CHECK(hipFuncSetAttribute((const void*)ag_gemm_kernel, hipFuncAttributeMaxDynamicSharedMemorySize, LDS_BYTES));
    attr_set = true;
  }
  Params P;
  P.x = (const uint8_t*)d_in[0];
  P.w = (const uint8_t*)d_in[1];
  P.sx = (const float*)d_in[2];
  P.sw = (const float*)d_in[3];
  P.out = (float*)d_out;
  P.myws = (uint8_t*)d_ws_all[rank];
  for (int j = 0; j < R - 1; ++j) P.peer[j] = (uint8_t*)d_ws_all[(rank + 1 + j) & (R - 1)] + (size_t)rank * SHARD;
  P.rank = rank;
  P.pad = 0;
  hipLaunchKernelGGL(ag_gemm_kernel, dim3(GRID), dim3(BLOCK), LDS_BYTES, stream, P);
  HIP_CHECK(hipGetLastError());
}
```

```cpp
#include <hip/hip_runtime.h>
#include <rccl/rccl.h>
#include <cstdint>
#include <cstdio>

#define HIP_CHECK(expr) do { hipError_t _e = (expr); if (_e != hipSuccess) { \
    fprintf(stderr, "HIP error %s at %s:%d\n", hipGetErrorString(_e), __FILE__, __LINE__); \
    return; } } while (0)

typedef unsigned u32x4 __attribute__((ext_vector_type(4)));
typedef unsigned u32x2 __attribute__((ext_vector_type(2)));
typedef int i32x2 __attribute__((ext_vector_type(2)));
typedef int i32x8 __attribute__((ext_vector_type(8)));
typedef float f32x4 __attribute__((ext_vector_type(4)));
#define LAS __attribute__((address_space(3)))

#define PW 1
#define PACE 16
#define LASTPACE 49
#define DIRECT_FROM 7
#define POLL_SLEEP 2
#define RND 4
#define NPROD 32

constexpr int R = 8;
constexpr int MS = 512, KD = 4096, NL = 512;
constexpr unsigned SHARD = MS * KD;
constexpr unsigned NU = 512;
constexpr unsigned U_SIDE = 3584;
constexpr unsigned U_PLANE = U_SIDE + 256;
constexpr unsigned USTR = U_PLANE + 512;
constexpr unsigned CAP = 124;
constexpr unsigned SH2 = NU * USTR;
constexpr unsigned SLOT = R * SH2;
constexpr int NSLOT = 3;
constexpr size_t CTRL = (size_t)NSLOT * SLOT;
constexpr size_t CNT_OFF = CTRL;
constexpr size_t TMO_OFF = CTRL + 32768;
constexpr size_t TICK_OFF = CTRL + 36864;
constexpr size_t DBG_OFF = CTRL + 65536;
constexpr int GRID = 256, BLOCK = 512;
constexpr size_t WS_NEED = CTRL + 65536 + GRID * (BLOCK / 64) * 128;
constexpr int LDS_P = 0;
constexpr int LDS_H = 131072;
constexpr int LDS_F = LDS_H + 4096;
constexpr int LDS_BYTES = LDS_F + 64;
constexpr unsigned long long DEADLINE = 150000000ull;
constexpr int AUX_SYS = 17;
constexpr unsigned M40 = 0x40404040u, M01 = 0x01010101u;

struct Params {
  const uint8_t* x;
  const uint8_t* w;
  const float* sx;
  const float* sw;
  float* out;
  uint8_t* myws;
  uint8_t* peer[R - 1];
  int rank;
  int pad;
};

__device__ __forceinline__ __amdgpu_buffer_rsrc_t mk_rsrc(const void* p, unsigned bytes) {
  return __builtin_amdgcn_make_buffer_rsrc((void*)p, (short)0, (int)bytes, 0x00020000);
}
__device__ __forceinline__ unsigned long long now() { return __builtin_amdgcn_s_memrealtime(); }
__device__ __forceinline__ unsigned umin3(unsigned a, unsigned b, unsigned c) { return min(min(a, b), c); }
__device__ __forceinline__ unsigned bfi(unsigned m, unsigned a, unsigned b) {
  return __builtin_amdgcn_bitop3_b32(m, a, b, 0xCA);
}
__device__ __forceinline__ unsigned xad(unsigned a, unsigned k, unsigned c) {
  unsigned r;
  asm("v_xad_u32 %0, %1, %2, %3" : "=v"(r) : "v"(a), "s"(k), "v"(c));
  return r;
}
__device__ __forceinline__ unsigned shl(unsigned v, int s) { return s >= 0 ? v << s : v >> -s; }
__device__ __forceinline__ float silu(float acc, float scale) {
  const float y = acc * scale;
  const float c = fminf(fmaxf(y, -60.f), 60.f);
  return y * __builtin_amdgcn_rcpf(1.f + __builtin_amdgcn_exp2f(c * -1.44269504088896341f));
}
__device__ __forceinline__ void ld_x(u32x4& d, unsigned off, const uint8_t* base) {
  asm volatile("global_load_dwordx4 %0, %1, %2" : "=v"(d) : "v"(off), "s"(base) : "memory");
}
template <int CTRL_, int RM> __device__ __forceinline__ unsigned dpp0(unsigned v) {
  return __builtin_amdgcn_update_dpp(0u, v, CTRL_, RM, 0xf, false);
}
__device__ __forceinline__ unsigned scan_add(unsigned v) {
  v += dpp0<0x111, 0xf>(v); v += dpp0<0x112, 0xf>(v); v += dpp0<0x114, 0xf>(v); v += dpp0<0x118, 0xf>(v);
  v += dpp0<0x142, 0xa>(v); v += dpp0<0x143, 0xc>(v);
  return v;
}
__device__ __forceinline__ unsigned wave_max(unsigned v) {
  v = max(v, dpp0<0x111, 0xf>(v)); v = max(v, dpp0<0x112, 0xf>(v)); v = max(v, dpp0<0x114, 0xf>(v));
  v = max(v, dpp0<0x118, 0xf>(v)); v = max(v, dpp0<0x142, 0xa>(v)); v = max(v, dpp0<0x143, 0xc>(v));
  return __builtin_amdgcn_readlane(v, 63);
}
__device__ __forceinline__ constexpr int pos7(int j) { return j < 6 ? j : 7; }

#define STAMP(i) do { } while (0)

__global__ void __launch_bounds__(BLOCK, 2) ag_gemm_kernel(Params P) {
  extern __shared__ __attribute__((aligned(16))) uint8_t lds[];
  const unsigned lane = threadIdx.x & 63;
  const int wave = __builtin_amdgcn_readfirstlane(threadIdx.x >> 6);
  const int b = blockIdx.x;
  uint8_t* const myws = P.myws;

  if (b < NPROD) {
    if (wave >= PW) return;
    unsigned* cntp = (unsigned*)(myws + CNT_OFF + b * 64 + wave * 4);
    unsigned* tick = (unsigned*)(myws + TICK_OFF);
    constexpr unsigned NPW = NPROD * PW;
    static_assert(PACE <= 21, "");
    const unsigned pid = b * PW + wave;
    const unsigned vo = (lane & 15) * KD + (lane >> 4) * 16;
    u32x4 d[4];
    auto fetch = [&](unsigned u) {
      const unsigned o = vo + ((u >> 6) * 64 + ((u >> 3) & 3) * 16) * KD + ((u & 7) * 4 + ((u >> 5) & 1) * 2) * 128;
      ld_x(d[0], o, P.x); ld_x(d[1], o + 64, P.x); ld_x(d[2], o + 128, P.x); ld_x(d[3], o + 192, P.x);
    };
    unsigned c;
    asm volatile("global_load_dword %0, %1, %2 sc1" : "=v"(c) : "v"(0u), "s"(cntp) : "memory");
    fetch(pid);
    __amdgpu_buffer_rsrc_t pr[R - 1];
#pragma unroll
    for (int j = 0; j < R - 1; ++j) pr[j] = mk_rsrc(P.peer[j], (NSLOT - 1) * SLOT + SH2);
    asm volatile("s_waitcnt vmcnt(0)" : "+v"(c), "+v"(d[0]), "+v"(d[1]), "+v"(d[2]), "+v"(d[3]) :: "memory");
    const unsigned call = __builtin_amdgcn_readfirstlane(c) + 1;
    const unsigned dst = (call % NSLOT) * SLOT;
    const unsigned tbase = (call - 1) * NU - 2 * NPW;
    LAS uint8_t* stg = (LAS uint8_t*)lds + wave * 2048;
    unsigned cur = pid, nxt = pid + NPW;
    STAMP(1);
    for (;;) {
      unsigned t[16];
#pragma unroll
      for (int j = 0; j < 16; ++j) t[j] = d[j >> 2][j & 3];
      unsigned mx = 0;
#pragma unroll
      for (int j = 0; j < 16; j += 2) mx = max(mx, t[j] & 0x7f000000u);
      const unsigned top = wave_max(mx) >> 24;
      const unsigned base = top > 63 ? top - 63 : 0;
      unsigned h[2] = {0u, 0u}, e[14];
#pragma unroll
      for (int j = 0; j < 16; ++j) {
        t[j] -= base * M01;
        h[j >> 3] = bfi(M01 << (j & 7), shl(t[j], (j & 7) - 6), h[j >> 3]);
      }
#pragma unroll
      for (int j = 0; j < 14; ++j) e[j] = bfi(M40, shl(t[(j / 7) * 8 + 7], 6 - pos7(j % 7)), t[(j / 7) * 8 + j % 7]);
      const bool more = nxt < NU;
      if (more) fetch(nxt);
      unsigned tk;
      asm volatile("s_mov_b64 exec, 1\n\tglobal_atomic_add %0, %1, %2, %3 sc0\n\ts_mov_b64 exec, -1"
                   : "=v"(tk) : "v"(0u), "v"(1u), "s"(tick) : "memory");
      const unsigned n = __builtin_popcount(h[0]) + __builtin_popcount(h[1]);
      const unsigned incl = scan_add(n);
      const unsigned cnt = __builtin_amdgcn_readlane(incl, 63);
      const bool dense = cnt > CAP;
      unsigned key = 0;
      for (;;) {
        key += 0x9E3779B1u;
        unsigned m = 0xffffffffu;
#pragma unroll
        for (int j = 0; j < 14; j += 2) m = umin3(m, e[j] ^ key, e[j + 1] ^ key);
        if (dense) m = umin3(m, h[0] ^ key, h[1] ^ key);
        if (__builtin_amdgcn_ballot_w64(m == 0) == 0) break;
      }
#pragma unroll
      for (int j = 0; j < 14; ++j) e[j] ^= key;
      const unsigned meta = 0x80000000u | (dense ? 0x40000000u : cnt) | (base << 8);
      *(LAS u32x2*)(stg + lane * 8) = u32x2{e[12], e[13]};
      *(LAS unsigned*)(stg + 512 + lane * 4) = lane == 0 ? key : (lane == 1 ? meta : 0x40004000u);
      unsigned n3;
      if (!dense) {
        unsigned pos = 4 + incl - n, h0 = h[0], h1 = h[1];
        while (h0 | h1) {
          const bool lo = h0 != 0;
          const unsigned bit = __builtin_ctz(lo ? h0 : h1);
          *(LAS unsigned short*)(stg + 512 + pos * 2) = (unsigned short)(0x8000u | ((lane + (lo ? 0 : 64)) << 5) | bit);
          ++pos;
          if (lo) h0 &= h0 - 1; else h1 &= h1 - 1;
        }
        n3 = 32 + (((cnt + 11) >> 3) + RND - 1) / RND * RND;
      } else {
        *(LAS unsigned*)(stg + 768 + lane * 4) = h[0] ^ key;
        *(LAS unsigned*)(stg + 1024 + lane * 4) = h[1] ^ key;
        n3 = 64;
      }
      const u32x4 s3 = *(const LAS u32x4*)(stg + lane * 16);
      const unsigned so = dst + cur * USTR;
#define PIECE(v, off, N) do { asm volatile("s_waitcnt vmcnt(%0)" :: "n"(N) : "memory"); \
      _Pragma("unroll") for (int j = 0; j < R - 1; ++j) \
        __builtin_amdgcn_raw_buffer_store_b128((v), pr[j], lane * 16 + (off), so, AUX_SYS); \
      __builtin_amdgcn_sched_barrier(0); } while (0)
      const u32x4 s0 = {e[0], e[1], e[2], e[3]}, s1 = {e[4], e[5], e[6], e[7]}, s2 = {e[8], e[9], e[10], e[11]};
      if (more) {
        PIECE(s0, 0, PACE + 5); PIECE(s1, 1024, PACE + 5); PIECE(s2, 2048, PACE + 5);
        if (lane < n3) PIECE(s3, 3072, PACE);
      } else {
        PIECE(s0, 0, LASTPACE); PIECE(s1, 1024, LASTPACE); PIECE(s2, 2048, LASTPACE);
        if (lane < n3) PIECE(s3, 3072, LASTPACE);
      }
      if (dense) {
        const u32x4 s4 = *(const LAS u32x4*)(stg + 1024 + (lane & 15) * 16);
        if (lane < 16) PIECE(s4, 4096, 21);
      }
      if (!more) break;
      asm volatile("s_waitcnt vmcnt(%5)" : "+v"(tk), "+v"(d[0]), "+v"(d[1]), "+v"(d[2]), "+v"(d[3]) : "n"(PACE + 7) : "memory");
      cur = nxt;
      nxt = __builtin_amdgcn_readfirstlane(tk) - tbase;
    }
    if (lane == 0) __hip_atomic_store(cntp, call, __ATOMIC_RELAXED, __HIP_MEMORY_SCOPE_AGENT);
    STAMP(2);
    asm volatile("s_waitcnt vmcnt(0)" ::: "memory");
    STAMP(3);
    return;
  }

  const unsigned long long t0 = now();
  const int rank = P.rank;
  const int ci = (b >> 3) - 4, lo = (b & 7) * 4 + ci - 24;
  const int set = ci < 24 ? (ci >> 3) * 8 + (b & 7) : 24 + (lo >> 3);
  const int nj = ci < 24 ? (ci & 7) : (lo & 7);
  const int s = (rank + 1 + (set >> 2)) & (R - 1);
  const unsigned rb = set & 3;
  unsigned* cntp = (unsigned*)(myws + CNT_OFF + b * 64);
  const unsigned prev = __hip_atomic_load(cntp, __ATOMIC_RELAXED, __HIP_MEMORY_SCOPE_AGENT);
  const float scale = P.sx[0] * P.sw[0];
  LAS unsigned* arrived = (LAS unsigned*)((LAS uint8_t*)lds + LDS_F);
  if (threadIdx.x < 2) arrived[threadIdx.x] = 0;
  const unsigned q = lane >> 4, li = lane & 15;

  i32x8 wr[4][4];
  {
    const unsigned row = lane >> 2;
    const uint8_t* wsrc = P.w + (size_t)(wave * 512) * NL + nj * 64;
    const unsigned c0 = row * NL + (((lane & 3) ^ ((row >> 2) & 1)) << 4);
    LAS uint8_t* stg = (LAS uint8_t*)lds + LDS_P + wave * 16384;
    auto stage = [&](int kb) {
#pragma unroll
      for (int i = 0; i < 8; ++i) {
        unsigned keep;
        asm volatile("s_mov_b32 %0, m0\n\ts_mov_b32 m0, %3\n\ts_nop 0\n\tglobal_load_lds_dwordx4 %1, %2\n\ts_mov_b32 m0, %0"
                     : "=&s"(keep) : "v"(c0 ^ ((i & 1) << 5)), "s"(wsrc + (kb * 128 + i * 16) * NL),
                       "s"((unsigned)(uintptr_t)(stg + (kb & 1) * 8192 + i * 1024)) : "memory");
      }
    };
    const unsigned f = ((li >> 3) & 1) | ((q & 1) << 1);
    const unsigned lb = (q * 16 + (li >> 1)) * 64 + (li & 1) * 8;
    auto unstage = [&](int kb) {
#pragma unroll
      for (int t = 0; t < 4; ++t)
#pragma unroll
        for (int j = 0; j < 4; ++j) {
          const i32x2 r = __builtin_amdgcn_ds_read_tr8_b64_v2i32(
              (LAS i32x2*)(stg + (kb & 1) * 8192 + lb + ((t ^ f) << 4) + (j >> 1) * 4096 + (j & 1) * 512));
          wr[kb][t][2 * j] = r[0]; wr[kb][t][2 * j + 1] = r[1];
        }
      asm volatile("s_waitcnt lgkmcnt(0)" : "+v"(wr[kb][0]), "+v"(wr[kb][1]), "+v"(wr[kb][2]), "+v"(wr[kb][3]) :: "memory");
    };
    stage(0); stage(1);
    asm volatile("s_waitcnt vmcnt(8)" ::: "memory");
    unstage(0); stage(2);
    asm volatile("s_waitcnt vmcnt(8)" ::: "memory");
    unstage(1); stage(3);
    asm volatile("s_waitcnt vmcnt(8)" ::: "memory");
    unstage(2);
    asm volatile("s_waitcnt vmcnt(0)" ::: "memory");
    unstage(3);
  }
  __builtin_amdgcn_s_barrier();
  asm volatile("" ::: "memory");
  const unsigned call = __builtin_amdgcn_readfirstlane(prev) + 1;
  if (threadIdx.x == 0) __hip_atomic_store(cntp, call, __ATOMIC_RELAXED, __HIP_MEMORY_SCOPE_AGENT);
  STAMP(1);

  unsigned* tmo = (unsigned*)(myws + TMO_OFF);
  const __amdgpu_buffer_rsrc_t orr = mk_rsrc(P.out, (unsigned)(R * MS * NL * 4));
  LAS unsigned* hb = (LAS unsigned*)((LAS uint8_t*)lds + LDS_H + wave * 512);
  bool dead = false;
  unsigned seq = 0, spins = 0;
  f32x4 acc[4];
#define MMA(xa, kb) do { _Pragma("unroll") for (int t_ = 0; t_ < 4; ++t_) \
    acc[t_] = __builtin_amdgcn_mfma_scale_f32_16x16x128_f8f6f4(wr[kb][t_], (xa), acc[t_], 0, 0, 0, 0, 0, 0); } while (0)
  auto wait_arrived = [&](unsigned n) {
    while (__builtin_amdgcn_readfirstlane(*(volatile LAS unsigned*)(arrived + (n & 1))) < 8 * ((n >> 1) + 1)) { }
    asm volatile("" ::: "memory");
  };
  auto finish = [&](unsigned orow) {
    if (seq >= 2) wait_arrived(seq - 1);
    LAS uint8_t* part = (LAS uint8_t*)lds + LDS_P + (seq & 1) * 32768;
#pragma unroll
    for (int t = 0; t < 4; ++t) {
      if (dead) acc[t][0] = acc[t][1] = acc[t][2] = acc[t][3] = __builtin_nanf("");
      *(LAS f32x4*)(part + wave * 4096 + t * 1024 + lane * 16) = acc[t];
    }
    asm volatile("s_waitcnt lgkmcnt(0)" ::: "memory");
    if (lane == 0) __hip_atomic_fetch_add(arrived + (seq & 1), 1u, __ATOMIC_RELAXED, __HIP_MEMORY_SCOPE_WORKGROUP);
    if ((unsigned)(wave >> 2) == (seq & 1)) {
      wait_arrived(seq);
      const unsigned t = wave & 3;
      f32x4 v[8];
#pragma unroll
      for (int w = 0; w < 8; ++w) v[w] = *(const LAS f32x4*)(part + w * 4096 + t * 1024 + lane * 16);
      const f32x4 sum = ((v[0] + v[1]) + (v[2] + v[3])) + ((v[4] + v[5]) + (v[6] + v[7]));
      f32x4 o;
#pragma unroll
      for (int r = 0; r < 4; ++r) o[r] = silu(sum[r], scale);
      __builtin_amdgcn_raw_buffer_store_b128(__builtin_bit_cast(u32x4, o), orr, (li * NL + q * 4) * 4,
                                             (orow * NL + nj * 64 + t * 16) * 4, AUX_SYS);
    }
    ++seq;
  };

  {
    const __amdgpu_buffer_rsrc_t xr = mk_rsrc(P.x, SHARD);
#pragma unroll 1
    for (int lblk = set; lblk < 32; lblk += 28) {
      u32x4 a[8];
#pragma unroll
      for (int i = 0; i < 8; ++i)
        a[i] = __builtin_amdgcn_raw_buffer_load_b128(xr, li * KD + q * 16 + i * 64, lblk * 16 * KD + wave * 512, 0);
#pragma unroll
      for (int t = 0; t < 4; ++t) acc[t] = f32x4{0.f, 0.f, 0.f, 0.f};
#pragma unroll
      for (int i = 0; i < 4; ++i) {
        const i32x8 xa = {(int)a[2 * i].x, (int)a[2 * i].y, (int)a[2 * i].z, (int)a[2 * i].w,
                          (int)a[2 * i + 1].x, (int)a[2 * i + 1].y, (int)a[2 * i + 1].z, (int)a[2 * i + 1].w};
        MMA(xa, i);
      }
      finish(rank * MS + lblk * 16);
    }
    STAMP(7);
  }

  const __amdgpu_buffer_rsrc_t ar = mk_rsrc(myws + (size_t)(call % NSLOT) * SLOT + (size_t)s * SH2, SH2);
#pragma unroll 1
  for (int p = 0; p < 8; ++p) {
    const unsigned po = (p * 64 + rb * 8 + wave) * USTR;
#pragma unroll
    for (int t = 0; t < 4; ++t) acc[t] = f32x4{0.f, 0.f, 0.f, 0.f};
    unsigned pending = 3u;
    while (pending && !dead) {
      unsigned ready = pending;
      asm volatile("" ::: "memory");
      if (p < DIRECT_FROM) {
        const unsigned v = __builtin_amdgcn_raw_buffer_load_b32(ar, (lane & 1) * (32 * USTR) + U_SIDE + 4, po, AUX_SYS);
        ready &= (unsigned)__builtin_amdgcn_ballot_w64(v != 0);
      }
      const unsigned before = pending;
      u32x4 a[2][3];
      u32x2 a3[2];
      unsigned sd[2];
#pragma unroll
      for (int i = 0; i < 2; ++i) {
        if (ready & (1u << i)) {
          const unsigned uo = po + i * (32 * USTR);
          sd[i] = __builtin_amdgcn_raw_buffer_load_b32(ar, lane * 4 + U_SIDE, uo, AUX_SYS);
          a[i][0] = __builtin_amdgcn_raw_buffer_load_b128(ar, lane * 16, uo, AUX_SYS);
          a[i][1] = __builtin_amdgcn_raw_buffer_load_b128(ar, lane * 16 + 1024, uo, AUX_SYS);
          a[i][2] = __builtin_amdgcn_raw_buffer_load_b128(ar, lane * 16 + 2048, uo, AUX_SYS);
          a3[i] = __builtin_amdgcn_raw_buffer_load_b64(ar, lane * 8 + 3072, uo, AUX_SYS);
        }
      }
#pragma unroll
      for (int i = 0; i < 2; ++i) {
        if (ready & (1u << i)) {
          const unsigned x[14] = {a[i][0].x, a[i][0].y, a[i][0].z, a[i][0].w, a[i][1].x, a[i][1].y, a[i][1].z,
                                  a[i][1].w, a[i][2].x, a[i][2].y, a[i][2].z, a[i][2].w, a3[i].x, a3[i].y};
          const unsigned key = __builtin_amdgcn_readlane(sd[i], 0), meta = __builtin_amdgcn_readlane(sd[i], 1);
          const unsigned cnt = meta & 0xffu;
          unsigned m = lane < 2 + ((cnt + 1) >> 1) ? sd[i] : 1u;
#pragma unroll
          for (int j = 0; j < 14; j += 2) m = umin3(m, x[j], x[j + 1]);
          unsigned h[2];
          if (meta & 0x40000000u) {
            const unsigned uo = po + i * (32 * USTR);
            h[0] = __builtin_amdgcn_raw_buffer_load_b32(ar, lane * 4 + U_PLANE, uo, AUX_SYS);
            h[1] = __builtin_amdgcn_raw_buffer_load_b32(ar, lane * 4 + U_PLANE + 256, uo, AUX_SYS);
            m = umin3(m, h[0], h[1]);
            h[0] ^= key; h[1] ^= key;
          } else {
            hb[lane] = 0; hb[lane + 64] = 0;
            const unsigned e0 = sd[i] & 0xffffu, e1 = sd[i] >> 16;
            if (lane >= 2 && (e0 & 0x8000u))
              __hip_atomic_fetch_or(hb + ((e0 >> 5) & 127), 1u << (e0 & 31), __ATOMIC_RELAXED, __HIP_MEMORY_SCOPE_WAVEFRONT);
            if (lane >= 2 && (e1 & 0x8000u))
              __hip_atomic_fetch_or(hb + ((e1 >> 5) & 127), 1u << (e1 & 31), __ATOMIC_RELAXED, __HIP_MEMORY_SCOPE_WAVEFRONT);
            asm volatile("" ::: "memory");
            h[0] = hb[lane]; h[1] = hb[lane + 64];
          }
          if (__builtin_amdgcn_ballot_w64(m == 0) == 0) {
            unsigned base4 = ((meta >> 8) & 0x7fu) * M01;
            const unsigned keyA = key & ~M40, keyB = ((key >> 6) & M01) * 0xbfu;
            asm volatile("" : "+v"(base4));
#pragma unroll
            for (int f = 0; f < 2; ++f) {
              i32x8 xa;
              unsigned t7 = shl(h[f], -1);
#pragma unroll
              for (int j = 0; j < 7; ++j) {
                xa[j] = (int)xad(bfi(M40, shl(h[f], 6 - j), x[7 * f + j]), keyA, base4);
                t7 = bfi(M01 << pos7(j), shl(x[7 * f + j], pos7(j) - 6), t7);
              }
              xa[7] = (int)xad(t7, keyB, base4);
              MMA(xa, 2 * i + f);
            }
            pending &= ~(1u << i);
          }
        }
      }
      if (pending == before) {
        if ((++spins & 63) == 0 &&
            (__hip_atomic_load(tmo, __ATOMIC_RELAXED, __HIP_MEMORY_SCOPE_AGENT) == call || now() - t0 > DEADLINE)) {
          __hip_atomic_store(tmo, call, __ATOMIC_RELAXED, __HIP_MEMORY_SCOPE_AGENT);
          dead = true;
        }
        __builtin_amdgcn_s_sleep(POLL_SLEEP);
      }
    }
    finish(s * MS + p * 64 + rb * 16);
    if (p == 3) {
      const unsigned z = (set * 8 + nj) * 8 + wave;
      const __amdgpu_buffer_rsrc_t zr = mk_rsrc(myws + (size_t)((call + 2) % NSLOT) * SLOT, SLOT);
      const unsigned so = ((rank + 1 + (z >> 8)) & (R - 1)) * SH2 + (z & 255) * (2 * USTR);
      const u32x4 zero = {0u, 0u, 0u, 0u};
#pragma unroll
      for (int i = 0; i < 8; ++i) __builtin_amdgcn_raw_buffer_store_b128(zero, zr, lane * 16 + i * 1024, so, AUX_SYS);
      if (lane < 32) __builtin_amdgcn_raw_buffer_store_b128(zero, zr, lane * 16 + 8192, so, AUX_SYS);
    }
    if (p & 1) STAMP(2 + (p >> 1));
  }
  asm volatile("s_waitcnt vmcnt(0)" ::: "memory");
  STAMP(6);
}

extern "C" void kernel_launch(int rank, int nranks, ncclComm_t comm, hipStream_t stream,
                              void* const* d_in, const int* in_sizes, int n_in,
                              void* d_out, int out_size, const int* in_dtypes, int out_dtype,
                              void* const* d_out_all, void* const* d_ws_all, size_t ws_bytes) {
  (void)comm; (void)in_dtypes; (void)out_dtype; (void)d_out_all;
  if (nranks != R || (unsigned)rank >= (unsigned)R || n_in != 4 || ws_bytes < WS_NEED ||
      in_sizes[0] != (int)SHARD || in_sizes[1] != KD * NL || out_size != R * MS * NL) {
    fprintf(stderr, "unexpected problem shape\n");
    return;
  }
  static bool attr_set = false;
  if (!attr_set) {
    HIP_CHECK(hipFuncSetAttribute((const void*)ag_gemm_kernel, hipFuncAttributeMaxDynamicSharedMemorySize, LDS_BYTES));
    attr_set = true;
  }
  Params P;
  P.x = (const uint8_t*)d_in[0];
  P.w = (const uint8_t*)d_in[1];
  P.sx = (const float*)d_in[2];
  P.sw = (const float*)d_in[3];
  P.out = (float*)d_out;
  P.myws = (uint8_t*)d_ws_all[rank];
  for (int j = 0; j < R - 1; ++j) P.peer[j] = (uint8_t*)d_ws_all[(rank + 1 + j) & (R - 1)] + (size_t)rank * SH2;
  P.rank = rank;
  P.pad = 0;
  hipLaunchKernelGGL(ag_gemm_kernel, dim3(GRID), dim3(BLOCK), LDS_BYTES, stream, P);
  HIP_CHECK(hipGetLastError());
}
```

```cpp
#include <hip/hip_runtime.h>
#include <rccl/rccl.h>
#include <cstdint>
#include <cstdio>

#define HIP_CHECK(expr) do { hipError_t _e = (expr); if (_e != hipSuccess) { \
    fprintf(stderr, "HIP error %s at %s:%d\n", hipGetErrorString(_e), __FILE__, __LINE__); \
    return; } } while (0)

typedef unsigned u32x4 __attribute__((ext_vector_type(4)));
typedef unsigned u32x2 __attribute__((ext_vector_type(2)));
typedef int i32x2 __attribute__((ext_vector_type(2)));
typedef int i32x8 __attribute__((ext_vector_type(8)));
typedef float f32x4 __attribute__((ext_vector_type(4)));
#define LAS __attribute__((address_space(3)))

#define HAND 10
#define TOPADD 224
#define DLIMIT 4
#define TEARLY 1
#define POLL_SLEEP 4
#define NPROD 32
#define PW 5
#define PACE 10

constexpr int R = 8;
constexpr int MS = 512, KD = 4096, NL = 512;
constexpr unsigned SHARD = MS * KD;
constexpr unsigned NU = 512;
constexpr unsigned U_T = 3072;
constexpr unsigned U_PLANE = U_T + 512;
constexpr unsigned USTR = U_PLANE + 1024;
constexpr unsigned SH2 = NU * USTR;
constexpr unsigned SLOT = R * SH2;
constexpr int NSLOT = 3;
constexpr size_t CTRL = (size_t)NSLOT * SLOT;
constexpr size_t CNT_OFF = CTRL;
constexpr size_t TMO_OFF = CTRL + 32768;
constexpr size_t TICK_OFF = CTRL + 36864;
constexpr size_t DBG_OFF = CTRL + 65536;
constexpr int GRID = 256, BLOCK = 512;
constexpr size_t WS_NEED = CTRL + 65536 + GRID * (BLOCK / 64) * 128;
constexpr int LDS_P = 8192;
constexpr int LDS_H = 0;
constexpr int LDS_F = LDS_P + 131072;
constexpr int LDS_S = LDS_F + 64;
constexpr int LDS_BYTES = LDS_S + 2048;
constexpr unsigned DEADLINE = 150000000u;
constexpr int AUX_SYS = 17;
constexpr unsigned M01 = 0x01010101u, M55 = 0x55555555u;

struct Params {
  const uint8_t* x;
  const uint8_t* w;
  const float* sx;
  const float* sw;
  float* out;
  uint8_t* myws;
  uint8_t* peer[R - 1];
  int rank;
  int pad;
};

__device__ __forceinline__ __amdgpu_buffer_rsrc_t mk_rsrc(const void* p, unsigned bytes) {
  return __builtin_amdgcn_make_buffer_rsrc((void*)p, (short)0, (int)bytes, 0x00020000);
}
__device__ __forceinline__ void stored(const u32x4& v) { asm volatile("s_nop 1" :: "v"(v) : "memory"); }
__device__ __forceinline__ unsigned long long now() { return __builtin_amdgcn_s_memrealtime(); }
__device__ __forceinline__ unsigned umin3(unsigned a, unsigned b, unsigned c) { return min(min(a, b), c); }
__device__ __forceinline__ unsigned bfi(unsigned m, unsigned a, unsigned b) {
  return __builtin_amdgcn_bitop3_b32(m, a, b, 0xCA);
}
__device__ __forceinline__ unsigned shl(unsigned v, int s) { return s >= 0 ? v << s : v >> -s; }
__device__ __forceinline__ float silu(float acc, float scale) {
  const float y = acc * scale;
  const float c = fminf(fmaxf(y, -60.f), 60.f);
  return y * __builtin_amdgcn_rcpf(1.f + __builtin_amdgcn_exp2f(c * -1.44269504088896341f));
}
__device__ __forceinline__ void ld_x(u32x4& d, unsigned off, const uint8_t* base) {
  asm volatile("global_load_dwordx4 %0, %1, %2" : "=v"(d) : "v"(off), "s"(base) : "memory");
}
template <int CTRL_, int RM> __device__ __forceinline__ unsigned dpp0(unsigned v) {
  return __builtin_amdgcn_update_dpp(0u, v, CTRL_, RM, 0xf, false);
}
__device__ __forceinline__ unsigned scan_add(unsigned v) {
  v += dpp0<0x111, 0xf>(v); v += dpp0<0x112, 0xf>(v); v += dpp0<0x114, 0xf>(v); v += dpp0<0x118, 0xf>(v);
  v += dpp0<0x142, 0xa>(v); v += dpp0<0x143, 0xc>(v);
  return v;
}
__device__ __forceinline__ unsigned scan_max_excl(unsigned v) {
  v = max(v, dpp0<0x111, 0xf>(v)); v = max(v, dpp0<0x112, 0xf>(v)); v = max(v, dpp0<0x114, 0xf>(v));
  v = max(v, dpp0<0x118, 0xf>(v)); v = max(v, dpp0<0x142, 0xa>(v)); v = max(v, dpp0<0x143, 0xc>(v));
  return dpp0<0x138, 0xf>(v);
}
__device__ __forceinline__ unsigned ffbl(unsigned v) { return (unsigned)__builtin_ctzg(v, -1); }
__device__ __forceinline__ unsigned bits(unsigned v) { return 32 - (unsigned)__builtin_clzg(v, 32); }
__device__ __forceinline__ bool none(bool c) { return __builtin_amdgcn_ballot_w64(c) == 0; }
__device__ __forceinline__ void lds_or(LAS uint8_t* a, unsigned v) {
  __hip_atomic_fetch_or((LAS unsigned*)a, v, __ATOMIC_RELAXED, __HIP_MEMORY_SCOPE_WAVEFRONT);
}

template <int SB> struct GapList {
  static constexpr unsigned E = (1u << SB) - 1, PER = 32 / SB;
  unsigned lo[2], hi[2], skip[2], len[2], off, total;
  bool fail;
  __device__ __forceinline__ void build(unsigned m0, unsigned m1, unsigned lane) {
    unsigned m[2] = {m0, m1}, thr[2];
    const unsigned e0 = bits(m0), e1 = bits(m1);
    const unsigned prev = scan_max_excl(m1 ? lane * 64 + 32 + e1 : (m0 ? lane * 64 + e0 : 0u));
    const unsigned pv[2] = {prev, m0 ? lane * 64 + e0 : prev};
#pragma unroll
    for (int h = 0; h < 2; ++h) {
      const unsigned gap = lane * 64 + 32 * h + ffbl(m[h]) + 1 - pv[h];
      skip[h] = m[h] ? (SB == 4 ? (gap * 4370u) >> 16 : (gap * 4113u) >> 20) : 0u;
      thr[h] = pv[h] + E * skip[h] + E - 1 - (lane * 64 + 32 * h);
      lo[h] = hi[h] = 0;
    }
#pragma unroll
    for (unsigned g = 0; g < 2 * PER; g += 4) {
      if (g > 0 && none((m[0] | m[1]) != 0)) break;
#pragma unroll
      for (unsigned i = g; i < g + 4; ++i)
#pragma unroll
        for (int h = 0; h < 2; ++h) {
          const unsigned bb = ffbl(m[h]);
          const unsigned u = __builtin_elementwise_sub_sat(thr[h], bb);
          if (i < PER) lo[h] |= u << (SB * i); else hi[h] |= u << (SB * (i - PER));
          thr[h] = min(bb, thr[h]) + E;
          m[h] &= m[h] - (u != 0 ? 1u : 0u);
        }
    }
    fail = !none((m[0] | m[1]) != 0);
#pragma unroll
    for (int h = 0; h < 2; ++h)
      len[h] = skip[h] + (hi[h] ? PER + (bits(hi[h]) + SB - 1) / SB : (bits(lo[h]) + SB - 1) / SB);
    const unsigned incl = scan_add(len[0] + len[1]);
    off = incl - len[0] - len[1];
    total = __builtin_amdgcn_readlane(incl, 63);
  }
  __device__ __forceinline__ void place(LAS uint8_t* img) {
#pragma unroll
    for (int h = 0; h < 2; ++h) {
      const unsigned o = (off + (h ? len[0] : 0u) + skip[h]) * SB, sh = o & 31;
      const unsigned long long v = (((unsigned long long)hi[h] << 32) | lo[h]) << sh;
      LAS uint8_t* a = img + ((o >> 5) << 2);
      lds_or(a, (unsigned)v); lds_or(a + 4, (unsigned)(v >> 32)); lds_or(a + 8, (hi[h] >> 1) >> (31 - sh));
    }
  }
};

#define STAMP(i) do { } while (0)

__global__ void __launch_bounds__(BLOCK, 2) ag_gemm_kernel(Params P) {
  extern __shared__ __attribute__((aligned(16))) uint8_t lds[];
  const unsigned lane = threadIdx.x & 63;
  const int wave = __builtin_amdgcn_readfirstlane(threadIdx.x >> 6);
  const int b = blockIdx.x;
  uint8_t* const myws = P.myws;
  const uint8_t* const xp = P.x;
  asm volatile("" :: "s"(myws), "s"(xp));

  if (__builtin_expect(b < NPROD, 1)) {
    if (wave >= PW) return;
    unsigned* cntp = (unsigned*)(myws + CNT_OFF + b * 64 + wave * 4);
    unsigned* tick = (unsigned*)(myws + TICK_OFF);
    constexpr unsigned NPW = NPROD * PW;
    const unsigned vo = (lane & 15) * KD + (lane >> 4) * 16;
    u32x4 d[4];
    auto fetch = [&](unsigned u) {
      const unsigned o = vo + ((u >> 6) * 64 + ((u >> 3) & 3) * 16) * KD + ((u & 7) * 4 + ((u >> 5) & 1) * 2) * 128;
      ld_x(d[0], o, xp); ld_x(d[1], o + 64, xp); ld_x(d[2], o + 128, xp); ld_x(d[3], o + 192, xp);
    };
    unsigned c, cur = b + NPROD * wave, call, turn = wave;
    fetch(cur);
    asm volatile("global_load_dword %0, %1, %2 sc1" : "=v"(c) : "v"(0u), "s"(cntp) : "memory");
    __amdgpu_buffer_rsrc_t pr[R - 1];
#pragma unroll
    for (int j = 0; j < R - 1; ++j) pr[j] = mk_rsrc(P.peer[j], (NSLOT - 1) * SLOT + SH2);
    LAS uint8_t* stg = (LAS uint8_t*)lds + wave * 1024;
    volatile LAS unsigned* token = (volatile LAS unsigned*)((LAS uint8_t*)lds + 8192);
    auto pass = [&](unsigned v) {
      if (lane == 0) __hip_atomic_fetch_max((LAS unsigned*)token, v, __ATOMIC_RELAXED, __HIP_MEMORY_SCOPE_WORKGROUP);
    };
    volatile LAS unsigned* gone = token + 1;
    if (threadIdx.x < 8) token[threadIdx.x] = 0;
    asm volatile("s_waitcnt lgkmcnt(0)" ::: "memory");
    __builtin_amdgcn_s_barrier();
    bool first = true;
    for (;;) {
      asm volatile("s_waitcnt vmcnt(0)" : "+v"(c), "+v"(d[0]), "+v"(d[1]), "+v"(d[2]), "+v"(d[3]) :: "memory");
      call = __builtin_amdgcn_readfirstlane(c) + 1;
      const unsigned so = (call % NSLOT) * SLOT + cur * USTR;
      const bool early = first && wave == 0;
      unsigned t[16], e[12];
#pragma unroll
      for (int j = 0; j < 16; ++j) t[j] = d[j >> 2][j & 3];
      unsigned mx = 0;
#pragma unroll
      for (int j = 0; j < 16; ++j) mx = max(mx, t[j] & 0x7f000000u);
      const unsigned top = (__builtin_amdgcn_readlane(scan_add(mx >> 24), 63) + TOPADD) >> 6;
      const unsigned base = top > 127 ? 96 : (top > 31 ? top - 31 : 0);
#pragma unroll
      for (int j = 0; j < 16; ++j) t[j] -= base * M01;
#pragma unroll
      for (int g = 0; g < 4; ++g) {
        const unsigned* tt = t + (g >> 1) * 8 + (g & 1) * 3;
        const unsigned z = t[(g >> 1) * 8 + 6 + (g & 1)];
        e[3 * g] = bfi(M01 * 0x60, z << 5, tt[0]);
        e[3 * g + 1] = bfi(M01 * 0x60, z << 3, tt[1]);
        e[3 * g + 2] = bfi(M01 * 0x20, z << 1, bfi(M01 * 0x40, z >> 1, tt[2]));
      }
      unsigned key = 0;
      for (;;) {
        key += 0x9E3779B1u;
        unsigned m = 0xffffffffu;
#pragma unroll
        for (int j = 0; j < 12; j += 2) m = umin3(m, e[j] ^ key, e[j + 1] ^ key);
        if (none(m == 0)) break;
      }
      const u32x4 pA = {e[0] ^ key, e[1] ^ key, e[2] ^ key, e[3] ^ key};
      const u32x4 pB = {e[4] ^ key, e[5] ^ key, e[6] ^ key, e[7] ^ key};
      const u32x4 pC = {e[8] ^ key, e[9] ^ key, e[10] ^ key, e[11] ^ key};
#define PIECE(v, off) do { _Pragma("unroll") for (int j = 0; j < R - 1; ++j) \
        __builtin_amdgcn_raw_buffer_store_b128((v), pr[j], lane * 16 + (off), so, AUX_SYS); \
      stored(v); \
      __builtin_amdgcn_sched_barrier(0); } while (0)
      if (early) {
        PIECE(pA, 0);
        PIECE(pB, 1024); PIECE(pC, 2048);
        pass(1);
      }
      unsigned X[2], Y[2], w[4] = {0u, 0u, 0u, 0u};
#pragma unroll
      for (int j = 0; j < 16; ++j) w[j >> 2] = bfi((M01 * 3) << (2 * (j & 3)), shl(t[j], 2 * (j & 3) - 5), w[j >> 2]);
#pragma unroll
      for (int f = 0; f < 2; ++f) {
        X[f] = bfi(M55, w[2 * f], w[2 * f + 1] << 1);
        Y[f] = bfi(M55, w[2 * f] >> 1, w[2 * f + 1]);
      }
      *(LAS u32x4*)(stg + lane * 16) = u32x4{0u, 0u, 0u, 0u};
      GapList<4> G;
      GapList<8> H;
      G.build(X[0], X[1], lane);
      H.build(X[0] ^ Y[0], X[1] ^ Y[1], lane);
      unsigned l0 = 1 + ((H.total + 7) >> 3);
      unsigned nl = l0 + ((G.total + 15) >> 4);
      const bool dense = G.fail || H.fail || nl > 64;
      unsigned k2 = 0;
      if (!dense) {
        H.place(stg + 8);
        G.place(stg + l0 * 8);
      } else {
        nl = 2; l0 = 1;
        do k2 += 0x9E3779B1u; while (!none(w[0] == k2 || w[1] == k2 || w[2] == k2 || w[3] == k2));
      }
      asm volatile("" ::: "memory");
      u32x4 pT = ~*(const LAS u32x4*)(stg + lane * 16);
      if (lane == 0) {
        pT.x = key;
        pT.y = 0x80000000u | (dense ? 0x40000000u : 0u) | (base << 16) | (l0 << 8) | nl;
        if (dense) pT.z = k2;
      }
      const unsigned nT = ((nl + 7) >> 3) * 4;
      if (TEARLY && lane < nT) PIECE(pT, U_T);
      if (!early) {
        for (;;) {
          unsigned need = turn;
#pragma unroll
          for (unsigned k = 1; k < PW; ++k)
            if (need == turn - (k - 1) && __builtin_amdgcn_readfirstlane(gone[(wave + PW - k) % PW])) --need;
          if (__builtin_amdgcn_readfirstlane(*token) >= need) break;
          __builtin_amdgcn_s_sleep(1);
        }
      }
      if (!TEARLY && lane < nT) PIECE(pT, U_T);
      if (dense) {
        const u32x4 pD = {w[0] ^ k2, w[1] ^ k2, w[2] ^ k2, w[3] ^ k2};
        PIECE(pD, U_PLANE);
      }
      if (!early) {
        PIECE(pA, 0);
        asm volatile("s_waitcnt vmcnt(%0)" :: "n"(PACE) : "memory");
        PIECE(pB, 1024);
        asm volatile("s_waitcnt vmcnt(%0)" :: "n"(PACE) : "memory");
        PIECE(pC, 2048);
        asm volatile("s_waitcnt vmcnt(%0)" :: "n"(HAND) : "memory");
        pass(turn + 1);
      }
      unsigned tk;
      asm volatile("s_mov_b64 exec, 1\n\tglobal_atomic_add %0, %1, %2, %3 sc0\n\ts_mov_b64 exec, -1\n\ts_waitcnt vmcnt(0)"
                   : "=&v"(tk) : "v"(0u), "v"(1u), "s"(tick) : "memory");
      cur = __builtin_amdgcn_readfirstlane(tk) - ((call - 1) * NU - NPW);
      const bool more = cur < NU;
      if (more) fetch(cur);
      else if (lane == 0) gone[wave] = 1;
      if (!more) break;
      turn += PW;
      first = false;
    }
    if (lane == 0) __hip_atomic_store(cntp, call, __ATOMIC_RELAXED, __HIP_MEMORY_SCOPE_AGENT);
    STAMP(2);
    asm volatile("s_waitcnt vmcnt(0)" ::: "memory");
    STAMP(3);
    return;
  }

  const unsigned t0 = (unsigned)now();
  const int rank = P.rank;
  const int ci = (b >> 3) - 4, lo = (b & 7) * 4 + ci - 24;
  const int set = ci < 24 ? (ci >> 3) * 8 + (b & 7) : 24 + (lo >> 3);
  const int nj = ci < 24 ? (ci & 7) : (lo & 7);
  const int s = (rank + 1 + (set >> 2)) & (R - 1);
  const unsigned rb = set & 3;
  unsigned* cntp = (unsigned*)(myws + CNT_OFF + b * 64);
  const unsigned prev = __hip_atomic_load(cntp, __ATOMIC_RELAXED, __HIP_MEMORY_SCOPE_AGENT);
  const float scale = P.sx[0] * P.sw[0];
  LAS unsigned* arrived = (LAS unsigned*)((LAS uint8_t*)lds + LDS_F);
  if (threadIdx.x < 4) arrived[threadIdx.x] = 0;
  const unsigned q = lane >> 4, li = lane & 15;

  i32x8 wr[4][4];
  {
    const unsigned row = lane >> 2;
    const uint8_t* wsrc = P.w + (size_t)(wave * 512) * NL + nj * 64;
    const unsigned c0 = row * NL + (((lane & 3) ^ ((row >> 2) & 1)) << 4);
    LAS uint8_t* stg = (LAS uint8_t*)lds + LDS_P + wave * 16384;
    auto stage = [&](int kb) {
#pragma unroll
      for (int i = 0; i < 8; ++i) {
        unsigned keep;
        asm volatile("s_mov_b32 %0, m0\n\ts_mov_b32 m0, %3\n\ts_nop 0\n\tglobal_load_lds_dwordx4 %1, %2\n\ts_mov_b32 m0, %0"
                     : "=&s"(keep) : "v"(c0 ^ ((i & 1) << 5)), "s"(wsrc + (kb * 128 + i * 16) * NL),
                       "s"((unsigned)(uintptr_t)(stg + (kb & 1) * 8192 + i * 1024)) : "memory");
      }
    };
    const unsigned f = ((li >> 3) & 1) | ((q & 1) << 1);
    const unsigned lb = (q * 16 + (li >> 1)) * 64 + (li & 1) * 8;
    auto unstage = [&](int kb) {
#pragma unroll
      for (int t = 0; t < 4; ++t)
#pragma unroll
        for (int j = 0; j < 4; ++j) {
          const i32x2 r = __builtin_amdgcn_ds_read_tr8_b64_v2i32(
              (LAS i32x2*)(stg + (kb & 1) * 8192 + lb + ((t ^ f) << 4) + (j >> 1) * 4096 + (j & 1) * 512));
          wr[kb][t][2 * j] = r[0]; wr[kb][t][2 * j + 1] = r[1];
        }
      asm volatile("s_waitcnt lgkmcnt(0)" : "+v"(wr[kb][0]), "+v"(wr[kb][1]), "+v"(wr[kb][2]), "+v"(wr[kb][3]) :: "memory");
    };
    stage(0); stage(1);
    asm volatile("s_waitcnt vmcnt(8)" ::: "memory");
    unstage(0); stage(2);
    asm volatile("s_waitcnt vmcnt(8)" ::: "memory");
    unstage(1); stage(3);
    asm volatile("s_waitcnt vmcnt(8)" ::: "memory");
    unstage(2);
    asm volatile("s_waitcnt vmcnt(0)" ::: "memory");
    unstage(3);
  }
  __builtin_amdgcn_s_barrier();
  asm volatile("" ::: "memory");
  const unsigned call = __builtin_amdgcn_readfirstlane(prev) + 1;
  if (threadIdx.x == 0) __hip_atomic_store(cntp, call, __ATOMIC_RELAXED, __HIP_MEMORY_SCOPE_AGENT);
  STAMP(1);

  unsigned* tmo = (unsigned*)(myws + TMO_OFF);
  const __amdgpu_buffer_rsrc_t orr = mk_rsrc(P.out, (unsigned)(R * MS * NL * 4));
  LAS uint8_t* hb = (LAS uint8_t*)lds + LDS_H + wave * 1024;
  bool dead = false;
  unsigned seq = 0, spins = 0;
  f32x4 acc[4];
#define MMA(xa, kb) do { _Pragma("unroll") for (int t_ = 0; t_ < 4; ++t_) \
    acc[t_] = __builtin_amdgcn_mfma_scale_f32_16x16x128_f8f6f4(wr[kb][t_], (xa), acc[t_], 0, 0, 0, 0, 0, 0); } while (0)
  auto lds_word = [&](unsigned i) { return __builtin_amdgcn_readfirstlane(*(volatile LAS unsigned*)(arrived + i)); };
  unsigned todo = ~0u, todo_row = 0;
  auto finalize = [&](bool block) {
    if (todo == ~0u) return;
    while (lds_word(todo & 1) < 8 * ((todo >> 1) + 1))
      if (!block) return;
    asm volatile("" ::: "memory");
    __builtin_amdgcn_s_setprio(2);
    const LAS uint8_t* part = (const LAS uint8_t*)lds + LDS_P + (todo & 1) * 32768 + (wave & 3) * 1024 + lane * 16;
    f32x4 v[5];
#pragma unroll
    for (int w = 0; w < 5; ++w) v[w] = *(const LAS f32x4*)(part + w * 4096);
    f32x4 sum = (v[0] + v[1]) + (v[2] + v[3]);
    asm volatile("" : "+v"(sum));
#pragma unroll
    for (int w = 0; w < 3; ++w) v[w] = *(const LAS f32x4*)(part + (5 + w) * 4096);
    sum += (v[4] + v[0]) + (v[1] + v[2]);
    f32x4 o;
#pragma unroll
    for (int r = 0; r < 4; ++r) o[r] = silu(sum[r], scale);
    __builtin_amdgcn_raw_buffer_store_b128(__builtin_bit_cast(u32x4, o), orr, (li * NL + q * 4) * 4,
                                           (todo_row * NL + nj * 64 + (wave & 3) * 16) * 4, AUX_SYS);
    stored(__builtin_bit_cast(u32x4, o));
    if (lane == 0) __hip_atomic_fetch_add(arrived + 2 + (todo & 1), 1u, __ATOMIC_RELAXED, __HIP_MEMORY_SCOPE_WORKGROUP);
    todo = ~0u;
    __builtin_amdgcn_s_setprio(0);
  };
  bool clear = true;
  auto check_clear = [&]() { if (!clear) clear = lds_word(2 + (seq & 1)) >= 4 * (seq >> 1); };
  auto deposit = [&](unsigned orow) {
    if (!clear) {
      if (todo == seq - 2) finalize(true);
      do check_clear(); while (!clear);
    }
    asm volatile("" ::: "memory");
    if (dead) {
#pragma unroll
      for (int t = 0; t < 4; ++t) acc[t][0] = acc[t][1] = acc[t][2] = acc[t][3] = __builtin_nanf("");
    }
    LAS uint8_t* part = (LAS uint8_t*)lds + LDS_P + (seq & 1) * 32768 + wave * 4096 + lane * 16;
#pragma unroll
    for (int t = 0; t < 4; ++t) *(LAS f32x4*)(part + t * 1024) = acc[t];
    asm volatile("" ::: "memory");
    if (lane == 0) __hip_atomic_fetch_add(arrived + (seq & 1), 1u, __ATOMIC_RELAXED, __HIP_MEMORY_SCOPE_WORKGROUP);
    if ((unsigned)(wave >> 2) == (seq & 1)) { todo = seq; todo_row = orow; }
    ++seq;
    clear = seq < 2;
  };

  const __amdgpu_buffer_rsrc_t ar = mk_rsrc(myws + (size_t)(call % NSLOT) * SLOT + (size_t)s * SH2, SH2);
  const uint64_t abase = (uint64_t)(myws + (size_t)(call % NSLOT) * SLOT + (size_t)s * SH2);
  const u32x4 asrd = {(unsigned)abase, (unsigned)(abase >> 32) & 0xffffu, SH2, 0x00020000u};
  const unsigned l16 = lane * 16, l8 = lane * 8;
  const unsigned po0 = (rb * 8 + wave) * USTR;
  unsigned look = (nj - wave) & 7;
  u32x4 pa0, pb0, pc0, pa1, pb1, pc1, w0, w1;
  u32x2 pt0, pt1;
#define REQUEST(c, u, on, pt, pa, pb, pc, w) \
    asm volatile("s_cmp_eq_u32 %9, 0\n\ts_cbranch_scc1 1f\n\t" \
                 "buffer_load_dwordx2 %0, %6, %7, %8 offen offset:3072 sc0 sc1\n\t" \
                 "buffer_load_dwordx4 %4, %5, %7, %8 offen offset:3584 sc0 sc1\n\t" \
                 "buffer_load_dwordx4 %1, %5, %7, %8 offen sc0 sc1\n\t" \
                 "buffer_load_dwordx4 %2, %5, %7, %8 offen offset:1024 sc0 sc1\n\t" \
                 "buffer_load_dwordx4 %3, %5, %7, %8 offen offset:2048 sc0 sc1\n1:" \
                 : c(pt), c(pa), c(pb), c(pc), c(w) \
                 : "v"(l16), "v"(l8), "s"(asrd), "s"(po0 + (u) * (32 * USTR)), "s"(on) : "memory", "scc")
#define LANDED(n, pt, pa, pb, pc, w) \
    asm volatile("s_waitcnt vmcnt(" #n ")" : "+v"(pt), "+v"(pa), "+v"(pb), "+v"(pc), "+v"(w) :: "memory")
#define UNPACK(fr, wa, wb, keyA, keyB, base, x0, x1, x2, x3, x4, x5) { \
      const unsigned base4 = base; \
      i32x8 xa; \
      xa[0] = (int)((bfi(M01 * 0x60, (wa) << 5, x0) ^ keyA) + base4); \
      xa[1] = (int)((bfi(M01 * 0x60, (wa) << 3, x1) ^ keyA) + base4); \
      xa[2] = (int)((bfi(M01 * 0x60, (wa) << 1, x2) ^ keyA) + base4); \
      xa[3] = (int)((bfi(M01 * 0x60, (wa) >> 1, x3) ^ keyA) + base4); \
      xa[4] = (int)((bfi(M01 * 0x60, (wb) << 5, x4) ^ keyA) + base4); \
      xa[5] = (int)((bfi(M01 * 0x60, (wb) << 3, x5) ^ keyA) + base4); \
      xa[6] = (int)((bfi(M01 * 3, (x0) >> 5, bfi(M01 * 0xc, (x1) >> 3, bfi(M01 * 0x10, (x2) >> 1, \
                     bfi(M01 * 0x80, (x2) << 1, (wb) << 1)))) ^ keyB) + base4); \
      xa[7] = (int)((bfi(M01 * 3, (x3) >> 5, bfi(M01 * 0xc, (x4) >> 3, bfi(M01 * 0x10, (x5) >> 1, \
                     bfi(M01 * 0x80, (x5) << 1, (wb) >> 1)))) ^ keyB) + base4); \
      MMA(xa, fr); }
#define FULL(v) none(min(min((v).x, (v).y), min((v).z, (v).w)) == 0)
  const unsigned sink = (unsigned)(uintptr_t)((LAS uint8_t*)lds + LDS_S) + threadIdx.x * 4;
  constexpr unsigned KD_LIST = 0xaaaaaaaau;
  auto lists = [&](const u32x2& pt, u32x4& w) {
    const unsigned meta = __builtin_amdgcn_readlane(pt.y, 0);
    const unsigned nl = meta & 0x7fu, l0 = (meta >> 8) & 0x7fu;
    if ((meta >> 30) != 2 || !none(lane < nl && min(pt.x, pt.y) == 0)) return false;
    *(LAS u32x4*)(hb + l16) = u32x4{0u, 0u, 0u, 0u};
    const bool on = lane - 1 < nl - 1, isb = lane < l0;
    const unsigned lo = on ? pt.x : ~0u, hi = on ? pt.y : ~0u;
    const unsigned F = M01 * 15;
    const unsigned tn = __builtin_amdgcn_sad_u8((lo & F) + (hi & F) + ((lo >> 4) & F) + ((hi >> 4) & F), 0u, 0u);
    const unsigned tb = __builtin_amdgcn_sad_u8(lo, 0u, __builtin_amdgcn_sad_u8(hi, 0u, 0u));
    const unsigned tot = lane == 0 ? 0u : (isb ? tb : tn);
    const unsigned incl = scan_add(tot);
    unsigned cur = (incl - tot - 1 + (isb ? 4096u : 0u - __builtin_amdgcn_readlane(incl, l0 - 1))) * 8193u +
                   ((unsigned)(uintptr_t)hb << 16);
    const unsigned sw = isb ? 8u : 4u, esc = isb ? 255u : 15u;
    const unsigned src[4] = {lo, isb ? hi : lo >> 16, isb ? ~0u : hi, isb ? ~0u : hi >> 16};
#pragma unroll
    for (int i = 0; i < 16; ++i) {
      const unsigned sy = __builtin_amdgcn_ubfe(src[i >> 2], sw * (i & 3), sw);
      cur += __umul24(sy, 8193u);
      __hip_atomic_fetch_xor((LAS unsigned*)(uintptr_t)(sy != esc ? (cur >> 16) & 0xfffcu : sink), 1u << (cur & 31),
                             __ATOMIC_RELAXED, __HIP_MEMORY_SCOPE_WAVEFRONT);
    }
    asm volatile("" ::: "memory");
    const u32x2 X = *(const LAS u32x2*)(hb + l8);
    const u32x2 Y = X ^ *(const LAS u32x2*)(hb + 512 + l8);
    w = u32x4{bfi(M55, X.x, Y.x << 1), bfi(M55, X.x >> 1, Y.x), bfi(M55, X.y, Y.y << 1), bfi(M55, X.y >> 1, Y.y)};
    return true;
  };
  auto leave = [&](const u32x4& w, unsigned uo) {
    const u32x4 v = w ^ KD_LIST;
    __builtin_amdgcn_raw_buffer_store_b128(v, ar, l16 + U_PLANE, uo, AUX_SYS);
    stored(v);
  };
  auto look_load = [&]() { return __builtin_amdgcn_raw_buffer_load_b64(ar, l8 + U_T, po0 + look * (32 * USTR), AUX_SYS); };
  auto look_decode = [&](const u32x2& pt) {
    u32x4 w;
    __builtin_amdgcn_s_setprio(3);
    const bool ok = lists(pt, w);
    __builtin_amdgcn_s_setprio(0);
    if (ok) { leave(w, po0 + look * (32 * USTR)); look += 8; }
    else if (__builtin_amdgcn_readlane(pt.y, 0) >> 30 == 3) look += 8;
  };
  if (look < 4) look_decode(look_load());
  {
    const __amdgpu_buffer_rsrc_t xr = mk_rsrc(xp, SHARD);
#pragma unroll 1
    for (int lblk = set; lblk < 32; lblk += 28) {
      u32x4 a[8];
#pragma unroll
      for (int i = 0; i < 8; ++i)
        a[i] = __builtin_amdgcn_raw_buffer_load_b128(xr, li * KD + q * 16 + i * 64, lblk * 16 * KD + wave * 512, 0);
      __builtin_amdgcn_sched_barrier(0);
#pragma unroll
      for (int t = 0; t < 4; ++t) acc[t] = f32x4{0.f, 0.f, 0.f, 0.f};
#pragma unroll
      for (int i = 0; i < 4; ++i) {
        const i32x8 xa = {(int)a[2 * i].x, (int)a[2 * i].y, (int)a[2 * i].z, (int)a[2 * i].w,
                          (int)a[2 * i + 1].x, (int)a[2 * i + 1].y, (int)a[2 * i + 1].z, (int)a[2 * i + 1].w};
        MMA(xa, i);
      }
      deposit(rank * MS + lblk * 16);
    }
    STAMP(7);
  }

#pragma unroll
  for (int t = 0; t < 4; ++t) acc[t] = f32x4{0.f, 0.f, 0.f, 0.f};
#define DBG_FIN() do { } while (0)
  const unsigned pconst = lane == 0 ? 3068 : (lane == 1 ? U_PLANE + 1020 : (lane == 2 ? U_T + 4 : SH2));
  auto probe = [&](unsigned u) {
    return __builtin_amdgcn_raw_buffer_load_b32(ar, pconst + (lane == 2 ? look * (32 * USTR) : u * (32 * USTR)), po0, AUX_SYS);
  };
  unsigned key, meta;
#define ACQUIRE(u, asked, pt, pa, pb, pc, fw) { \
    unsigned tries = 0; \
    for (;;) { \
      if (!asked) for (;;) { \
        const unsigned v = probe(u); \
        check_clear(); \
        const unsigned seen = (unsigned)__builtin_amdgcn_ballot_w64(v != 0); \
        if ((seen & 1u) && ((seen & 2u) || look == (u) || ++tries >= DLIMIT)) break; \
        if ((seen & 4u) && look < 16) look_decode(look_load()); \
        if ((++spins & 63) == 0 && timed_out()) dead = true; \
        if (dead) break; \
        finalize(false); \
        if ((u) < 13) __builtin_amdgcn_s_sleep(POLL_SLEEP); \
      } \
      REQUEST("+v", u, asked ^ 1u, pt, pa, pb, pc, fw); \
      LANDED(0, pt, pa, pb, pc, fw); \
      const bool data = none(umin3(umin3(umin3(pa.x, pa.y, pa.z), umin3(pa.w, pb.x, pb.y), umin3(pb.z, pb.w, pc.x)), \
                                   min(pc.y, pc.z), pc.w) == 0); \
      const bool flds = FULL(fw); \
      key = __builtin_amdgcn_readlane(pt.x, 0); meta = __builtin_amdgcn_readlane(pt.y, 0); \
      unsigned kd = KD_LIST; \
      if (__builtin_expect(meta & 0x40000000u, 0)) kd = __builtin_amdgcn_readlane(pt.x, 1); \
      bool done = dead; \
      if (data && min(min(key, meta), kd) != 0) { \
        if (__builtin_expect(flds, 1)) { fw ^= kd; done = true; } \
        else if (lists(pt, fw)) { \
          done = true; \
          if (look == (u)) leave(fw, po0 + (u) * (32 * USTR)); \
        } \
      } \
      if (done) break; \
      if ((++spins & 63) == 0 && timed_out()) dead = true; \
      asked = 0; \
    } \
    if (look == (u)) look += 8; }
#define UNIT(i, pt, pa, pb, pc, fw, qt, qa, qb, qc, qw) { \
    const unsigned u = 2 * p + (i); \
    const unsigned pv = probe(u + 1); \
    if (u < 14) finalize(false); \
    if (u == 8) rezero(); \
    __builtin_amdgcn_s_setprio(2); \
    unsigned ka = key & ~(M01 * 0x60), ba = ((meta >> 16) & 0x7fu) * M01; \
    unsigned kb = ((key >> 5) & (M01 * 3)) | ((key >> 3) & (M01 * 0xc)) | ((key >> 1) & (M01 * 0x10)) | ((key << 1) & (M01 * 0x80)); \
    asm volatile("" : "+s"(ka), "+s"(kb), "+v"(ba));        \
    UNPACK(2 * (i), fw.x, fw.y, ka, kb, ba, pa.x, pa.y, pa.z, pa.w, pb.x, pb.y) \
    __builtin_amdgcn_sched_barrier(0); \
    const unsigned ahead = (unsigned)__builtin_amdgcn_ballot_w64(pv != 0); \
    unsigned asked = ((ahead & 3u) + 1u) >> 2; \
    REQUEST("=&v", u + 1, asked, qt, qa, qb, qc, qw); \
    __builtin_amdgcn_sched_barrier(0); \
    UNPACK(2 * (i) + 1, fw.z, fw.w, ka, kb, ba, pb.z, pb.w, pc.x, pc.y, pc.z, pc.w) \
    if (i) { \
      deposit(s * MS + p * 64 + rb * 16); \
      _Pragma("unroll") for (int t = 0; t < 4; ++t) acc[t] = f32x4{0.f, 0.f, 0.f, 0.f}; \
      DBG_FIN(); \
    } \
    __builtin_amdgcn_s_setprio(0); \
    if (u == 15) break; \
    if ((ahead & 4u) && look < 16 && look != u + 1) look_decode(look_load()); \
    ACQUIRE(u + 1, asked, qt, qa, qb, qc, qw) }
  auto timed_out = [&]() {
    if (__builtin_amdgcn_readfirstlane(__hip_atomic_load(tmo, __ATOMIC_RELAXED, __HIP_MEMORY_SCOPE_AGENT)) != call &&
        (unsigned)now() - t0 <= DEADLINE) return false;
    __hip_atomic_store(tmo, call, __ATOMIC_RELAXED, __HIP_MEMORY_SCOPE_AGENT);
    return true;
  };
  auto rezero = [&]() {
    const unsigned z = (set * 8 + nj) * 8 + wave;
    const __amdgpu_buffer_rsrc_t zr = mk_rsrc(myws + (size_t)((call + 2) % NSLOT) * SLOT, SLOT);
    const unsigned so = ((rank + 1 + (z >> 8)) & (R - 1)) * SH2 + (z & 255) * (2 * USTR);
    u32x4 zero = {0u, 0u, 0u, 0u};
    asm volatile("" : "+v"(zero));
#pragma unroll
    for (int i = 0; i < 9; ++i) __builtin_amdgcn_raw_buffer_store_b128(zero, zr, l16 + (i % 3) * 1024, so + (i / 3) * 3072, AUX_SYS);
    stored(zero);
  };
  {
    unsigned asked = 1;
    REQUEST("=&v", 0u, asked, pt0, pa0, pb0, pc0, w0);
    ACQUIRE(0u, asked, pt0, pa0, pb0, pc0, w0)
  }
#pragma unroll 1
  for (unsigned p = 0;; ++p) {
    UNIT(0, pt0, pa0, pb0, pc0, w0, pt1, pa1, pb1, pc1, w1)
    UNIT(1, pt1, pa1, pb1, pc1, w1, pt0, pa0, pb0, pc0, w0)
  }
  finalize(true);
  asm volatile("s_waitcnt vmcnt(0)" ::: "memory");
  STAMP(6);
}

extern "C" void kernel_launch(int rank, int nranks, ncclComm_t comm, hipStream_t stream,
                              void* const* d_in, const int* in_sizes, int n_in,
                              void* d_out, int out_size, const int* in_dtypes, int out_dtype,
                              void* const* d_out_all, void* const* d_ws_all, size_t ws_bytes) {
  (void)comm; (void)in_dtypes; (void)out_dtype; (void)d_out_all;
  if (nranks != R || (unsigned)rank >= (unsigned)R || n_in != 4 || ws_bytes < WS_NEED ||
      in_sizes[0] != (int)SHARD || in_sizes[1] != KD * NL || out_size != R * MS * NL) {
    fprintf(stderr, "unexpected problem shape\n");
    return;
  }
  static bool attr_set = false;
  if (!attr_set) {
    HIP_CHECK(hipFuncSetAttribute((const void*)ag_gemm_kernel, hipFuncAttributeMaxDynamicSharedMemorySize, LDS_BYTES));
    attr_set = true;
  }
  Params P;
  P.x = (const uint8_t*)d_in[0];
  P.w = (const uint8_t*)d_in[1];
  P.sx = (const float*)d_in[2];
  P.sw = (const float*)d_in[3];
  P.out = (float*)d_out;
  P.myws = (uint8_t*)d_ws_all[rank];
  for (int j = 0; j < R - 1; ++j) P.peer[j] = (uint8_t*)d_ws_all[(rank + 1 + j) & (R - 1)] + (size_t)rank * SH2;
  P.rank = rank;
  P.pad = 0;
  hipLaunchKernelGGL(ag_gemm_kernel, dim3(GRID), dim3(BLOCK), LDS_BYTES, stream, P);
  HIP_CHECK(hipGetLastError());
}
```

```cpp
#include <hip/hip_runtime.h>
#include <rccl/rccl.h>
#include <cstdint>
#include <cstdio>

#define HIP_CHECK(expr) do { hipError_t _e = (expr); if (_e != hipSuccess) { \
    fprintf(stderr, "HIP error %s at %s:%d\n", hipGetErrorString(_e), __FILE__, __LINE__); \
    return; } } while (0)

typedef unsigned u32x4 __attribute__((ext_vector_type(4)));
typedef unsigned u32x2 __attribute__((ext_vector_type(2)));
typedef int i32x2 __attribute__((ext_vector_type(2)));
typedef int i32x8 __attribute__((ext_vector_type(8)));
typedef float f32x4 __attribute__((ext_vector_type(4)));
#define LAS __attribute__((address_space(3)))

#define HAND 10
#define TOPADD 224
#define DLIMIT 4
#define TEARLY 1
#define POLL_SLEEP 4
#define NPROD 32
#define PW 5
#define PACE 10

constexpr int R = 8;
constexpr int MS = 512, KD = 4096, NL = 512;
constexpr unsigned SHARD = MS * KD;
constexpr unsigned NU = 512;
constexpr unsigned U_T = 3072;
constexpr unsigned U_PLANE = U_T + 512;
constexpr unsigned USTR = U_PLANE + 1024;
constexpr unsigned SH2 = NU * USTR;
constexpr unsigned SLOT = R * SH2;
constexpr int NSLOT = 3;
constexpr size_t CTRL = (size_t)NSLOT * SLOT;
constexpr size_t CNT_OFF = CTRL;
constexpr size_t TMO_OFF = CTRL + 32768;
constexpr size_t TICK_OFF = CTRL + 36864;
constexpr size_t DBG_OFF = CTRL + 65536;
constexpr int GRID = 256, BLOCK = 512;
constexpr size_t WS_NEED = CTRL + 65536 + GRID * (BLOCK / 64) * 128;
constexpr int LDS_P = 8192;
constexpr int LDS_H = 0;
constexpr int LDS_F = LDS_P + 131072;
constexpr int LDS_S = LDS_F + 64;
constexpr int LDS_BYTES = LDS_S + 2048;
constexpr unsigned DEADLINE = 150000000u;
constexpr int AUX_SYS = 17;
constexpr unsigned M01 = 0x01010101u, M55 = 0x55555555u;

struct Params {
  const uint8_t* x;
  const uint8_t* w;
  const float* sx;
  const float* sw;
  float* out;
  uint8_t* myws;
  uint8_t* peer[R - 1];
  int rank;
  int pad;
};

__device__ __forceinline__ __amdgpu_buffer_rsrc_t mk_rsrc(const void* p, unsigned bytes) {
  return __builtin_amdgcn_make_buffer_rsrc((void*)p, (short)0, (int)bytes, 0x00020000);
}
__device__ __forceinline__ void stored(const u32x4& v) { asm volatile("s_nop 1" :: "v"(v) : "memory"); }
__device__ __forceinline__ unsigned long long now() { return __builtin_amdgcn_s_memrealtime(); }
__device__ __forceinline__ unsigned umin3(unsigned a, unsigned b, unsigned c) { return min(min(a, b), c); }
__device__ __forceinline__ unsigned bfi(unsigned m, unsigned a, unsigned b) {
  return __builtin_amdgcn_bitop3_b32(m, a, b, 0xCA);
}
__device__ __forceinline__ unsigned shl(unsigned v, int s) { return s >= 0 ? v << s : v >> -s; }
__device__ __forceinline__ float silu(float acc, float scale) {
  const float y = acc * scale;
  const float c = fminf(fmaxf(y, -60.f), 60.f);
  return y * __builtin_amdgcn_rcpf(1.f + __builtin_amdgcn_exp2f(c * -1.44269504088896341f));
}
__device__ __forceinline__ void ld_x(u32x4& d, unsigned off, const uint8_t* base) {
  asm volatile("global_load_dwordx4 %0, %1, %2" : "=v"(d) : "v"(off), "s"(base) : "memory");
}
template <int CTRL_, int RM> __device__ __forceinline__ unsigned dpp0(unsigned v) {
  return __builtin_amdgcn_update_dpp(0u, v, CTRL_, RM, 0xf, false);
}
__device__ __forceinline__ unsigned scan_add(unsigned v) {
  v += dpp0<0x111, 0xf>(v); v += dpp0<0x112, 0xf>(v); v += dpp0<0x114, 0xf>(v); v += dpp0<0x118, 0xf>(v);
  v += dpp0<0x142, 0xa>(v); v += dpp0<0x143, 0xc>(v);
  return v;
}
__device__ __forceinline__ unsigned scan_max_excl(unsigned v) {
  v = max(v, dpp0<0x111, 0xf>(v)); v = max(v, dpp0<0x112, 0xf>(v)); v = max(v, dpp0<0x114, 0xf>(v));
  v = max(v, dpp0<0x118, 0xf>(v)); v = max(v, dpp0<0x142, 0xa>(v)); v = max(v, dpp0<0x143, 0xc>(v));
  return dpp0<0x138, 0xf>(v);
}
__device__ __forceinline__ unsigned ffbl(unsigned v) { return (unsigned)__builtin_ctzg(v, -1); }
__device__ __forceinline__ unsigned bits(unsigned v) { return 32 - (unsigned)__builtin_clzg(v, 32); }
__device__ __forceinline__ bool none(bool c) { return __builtin_amdgcn_ballot_w64(c) == 0; }
__device__ __forceinline__ void lds_or(LAS uint8_t* a, unsigned v) {
  __hip_atomic_fetch_or((LAS unsigned*)a, v, __ATOMIC_RELAXED, __HIP_MEMORY_SCOPE_WAVEFRONT);
}

template <int SB> struct GapList {
  static constexpr unsigned E = (1u << SB) - 1, PER = 32 / SB;
  unsigned lo[2], hi[2], skip[2], len[2], off, total;
  bool fail;
  __device__ __forceinline__ void build(unsigned m0, unsigned m1, unsigned lane) {
    unsigned m[2] = {m0, m1}, thr[2];
    const unsigned e0 = bits(m0), e1 = bits(m1);
    const unsigned prev = scan_max_excl(m1 ? lane * 64 + 32 + e1 : (m0 ? lane * 64 + e0 : 0u));
    const unsigned pv[2] = {prev, m0 ? lane * 64 + e0 : prev};
#pragma unroll
    for (int h = 0; h < 2; ++h) {
      const unsigned gap = lane * 64 + 32 * h + ffbl(m[h]) + 1 - pv[h];
      skip[h] = m[h] ? (SB == 4 ? (gap * 4370u) >> 16 : (gap * 4113u) >> 20) : 0u;
      thr[h] = pv[h] + E * skip[h] + E - 1 - (lane * 64 + 32 * h);
      lo[h] = hi[h] = 0;
    }
#pragma unroll
    for (unsigned g = 0; g < 2 * PER; g += 4) {
      if (g > 0 && none((m[0] | m[1]) != 0)) break;
#pragma unroll
      for (unsigned i = g; i < g + 4; ++i)
#pragma unroll
        for (int h = 0; h < 2; ++h) {
          const unsigned bb = ffbl(m[h]);
          const unsigned u = __builtin_elementwise_sub_sat(thr[h], bb);
          if (i < PER) lo[h] |= u << (SB * i); else hi[h] |= u << (SB * (i - PER));
          thr[h] = min(bb, thr[h]) + E;
          m[h] &= m[h] - (u != 0 ? 1u : 0u);
        }
    }
    fail = !none((m[0] | m[1]) != 0);
#pragma unroll
    for (int h = 0; h < 2; ++h)
      len[h] = skip[h] + (hi[h] ? PER + (bits(hi[h]) + SB - 1) / SB : (bits(lo[h]) + SB - 1) / SB);
    const unsigned incl = scan_add(len[0] + len[1]);
    off = incl - len[0] - len[1];
    total = __builtin_amdgcn_readlane(incl, 63);
  }
  __device__ __forceinline__ void place(LAS uint8_t* img) {
#pragma unroll
    for (int h = 0; h < 2; ++h) {
      const unsigned o = (off + (h ? len[0] : 0u) + skip[h]) * SB, sh = o & 31;
      const unsigned long long v = (((unsigned long long)hi[h] << 32) | lo[h]) << sh;
      LAS uint8_t* a = img + ((o >> 5) << 2);
      lds_or(a, (unsigned)v); lds_or(a + 4, (unsigned)(v >> 32)); lds_or(a + 8, (hi[h] >> 1) >> (31 - sh));
    }
  }
};

#define STAMP(i) do { } while (0)

__global__ void __launch_bounds__(BLOCK, 2) ag_gemm_kernel(Params P) {
  extern __shared__ __attribute__((aligned(16))) uint8_t lds[];
  const unsigned lane = threadIdx.x & 63;
  const int wave = __builtin_amdgcn_readfirstlane(threadIdx.x >> 6);
  const int b = blockIdx.x;
  uint8_t* const myws = P.myws;
  const uint8_t* const xp = P.x;
  asm volatile("" :: "s"(myws), "s"(xp));

  if (__builtin_expect(b < NPROD, 1)) {
    if (wave >= PW) return;
    unsigned* cntp = (unsigned*)(myws + CNT_OFF + b * 64 + wave * 4);
    unsigned* tick = (unsigned*)(myws + TICK_OFF);
    constexpr unsigned NPW = NPROD * PW;
    const unsigned vo = (lane & 15) * KD + (lane >> 4) * 16;
    u32x4 d[4];
    auto fetch = [&](unsigned u) {
      const unsigned o = vo + ((u >> 6) * 64 + ((u >> 3) & 3) * 16) * KD + ((u & 7) * 4 + ((u >> 5) & 1) * 2) * 128;
      ld_x(d[0], o, xp); ld_x(d[1], o + 64, xp); ld_x(d[2], o + 128, xp); ld_x(d[3], o + 192, xp);
    };
    unsigned c, cur = b + NPROD * wave, call, turn = wave;
    fetch(cur);
    asm volatile("global_load_dword %0, %1, %2 sc1" : "=v"(c) : "v"(0u), "s"(cntp) : "memory");
    __amdgpu_buffer_rsrc_t pr[R - 1];
#pragma unroll
    for (int j = 0; j < R - 1; ++j) pr[j] = mk_rsrc(P.peer[j], (NSLOT - 1) * SLOT + SH2);
    LAS uint8_t* stg = (LAS uint8_t*)lds + wave * 1024;
    volatile LAS unsigned* token = (volatile LAS unsigned*)((LAS uint8_t*)lds + 8192);
    auto pass = [&](unsigned v) {
      if (lane == 0) __hip_atomic_fetch_max((LAS unsigned*)token, v, __ATOMIC_RELAXED, __HIP_MEMORY_SCOPE_WORKGROUP);
    };
    volatile LAS unsigned* gone = token + 1;
    if (threadIdx.x < 8) token[threadIdx.x] = 0;
    asm volatile("s_waitcnt lgkmcnt(0)" ::: "memory");
    __builtin_amdgcn_s_barrier();
    bool first = true;
    for (;;) {
      asm volatile("s_waitcnt vmcnt(0)" : "+v"(c), "+v"(d[0]), "+v"(d[1]), "+v"(d[2]), "+v"(d[3]) :: "memory");
      call = __builtin_amdgcn_readfirstlane(c) + 1;
      const unsigned so = (call % NSLOT) * SLOT + cur * USTR;
      const bool early = first && wave == 0;
      unsigned t[16], e[12];
#pragma unroll
      for (int j = 0; j < 16; ++j) t[j] = d[j >> 2][j & 3];
      unsigned mx = 0;
#pragma unroll
      for (int j = 0; j < 16; ++j) mx = max(mx, t[j] & 0x7f000000u);
      const unsigned top = (__builtin_amdgcn_readlane(scan_add(mx >> 24), 63) + TOPADD) >> 6;
      const unsigned base = top > 127 ? 96 : (top > 31 ? top - 31 : 0);
#pragma unroll
      for (int j = 0; j < 16; ++j) t[j] -= base * M01;
#pragma unroll
      for (int g = 0; g < 4; ++g) {
        const unsigned* tt = t + (g >> 1) * 8 + (g & 1) * 3;
        const unsigned z = t[(g >> 1) * 8 + 6 + (g & 1)];
        e[3 * g] = bfi(M01 * 0x60, z << 5, tt[0]);
        e[3 * g + 1] = bfi(M01 * 0x60, z << 3, tt[1]);
        e[3 * g + 2] = bfi(M01 * 0x20, z << 1, bfi(M01 * 0x40, z >> 1, tt[2]));
      }
      unsigned key = 0;
      for (;;) {
        key += 0x9E3779B1u;
        unsigned m = 0xffffffffu;
#pragma unroll
        for (int j = 0; j < 12; j += 2) m = umin3(m, e[j] ^ key, e[j + 1] ^ key);
        if (none(m == 0)) break;
      }
      const u32x4 pA = {e[0] ^ key, e[1] ^ key, e[2] ^ key, e[3] ^ key};
      const u32x4 pB = {e[4] ^ key, e[5] ^ key, e[6] ^ key, e[7] ^ key};
      const u32x4 pC = {e[8] ^ key, e[9] ^ key, e[10] ^ key, e[11] ^ key};
#define PIECE(v, off) do { _Pragma("unroll") for (int j = 0; j < R - 1; ++j) \
        __builtin_amdgcn_raw_buffer_store_b128((v), pr[j], lane * 16 + (off), so, AUX_SYS); \
      stored(v); \
      __builtin_amdgcn_sched_barrier(0); } while (0)
      if (early) {
        PIECE(pA, 0);
        PIECE(pB, 1024); PIECE(pC, 2048);
        pass(1);
      }
      unsigned X[2], Y[2], w[4] = {0u, 0u, 0u, 0u};
#pragma unroll
      for (int j = 0; j < 16; ++j) w[j >> 2] = bfi((M01 * 3) << (2 * (j & 3)), shl(t[j], 2 * (j & 3) - 5), w[j >> 2]);
#pragma unroll
      for (int f = 0; f < 2; ++f) {
        X[f] = bfi(M55, w[2 * f], w[2 * f + 1] << 1);
        Y[f] = bfi(M55, w[2 * f] >> 1, w[2 * f + 1]);
      }
      *(LAS u32x4*)(stg + lane * 16) = u32x4{0u, 0u, 0u, 0u};
      GapList<4> G;
      GapList<8> H;
      G.build(X[0], X[1], lane);
      H.build(X[0] ^ Y[0], X[1] ^ Y[1], lane);
      unsigned l0 = 1 + ((H.total + 7) >> 3);
      unsigned nl = l0 + ((G.total + 15) >> 4);
      const bool dense = G.fail || H.fail || nl > 64;
      unsigned k2 = 0;
      if (!dense) {
        H.place(stg + 8);
        G.place(stg + l0 * 8);
      } else {
        nl = 2; l0 = 1;
        do k2 += 0x9E3779B1u; while (!none(w[0] == k2 || w[1] == k2 || w[2] == k2 || w[3] == k2));
      }
      asm volatile("" ::: "memory");
      u32x4 pT = ~*(const LAS u32x4*)(stg + lane * 16);
      if (lane == 0) {
        pT.x = key;
        pT.y = 0x80000000u | (dense ? 0x40000000u : 0u) | (base << 16) | (l0 << 8) | nl;
        if (dense) pT.z = k2;
      }
      const unsigned nT = ((nl + 7) >> 3) * 4;
      if (TEARLY && lane < nT) PIECE(pT, U_T);
      if (!early) {
        for (;;) {
          unsigned need = turn;
#pragma unroll
          for (unsigned k = 1; k < PW; ++k)
            if (need == turn - (k - 1) && __builtin_amdgcn_readfirstlane(gone[(wave + PW - k) % PW])) --need;
          if (__builtin_amdgcn_readfirstlane(*token) >= need) break;
          __builtin_amdgcn_s_sleep(1);
        }
      }
      if (!TEARLY && lane < nT) PIECE(pT, U_T);
      if (dense) {
        const u32x4 pD = {w[0] ^ k2, w[1] ^ k2, w[2] ^ k2, w[3] ^ k2};
        PIECE(pD, U_PLANE);
      }
      if (!early) {
        PIECE(pA, 0);
        asm volatile("s_waitcnt vmcnt(%0)" :: "n"(PACE) : "memory");
        PIECE(pB, 1024);
        asm volatile("s_waitcnt vmcnt(%0)" :: "n"(PACE) : "memory");
        PIECE(pC, 2048);
        asm volatile("s_waitcnt vmcnt(%0)" :: "n"(HAND) : "memory");
        pass(turn + 1);
      }
      unsigned tk;
      asm volatile("s_mov_b64 exec, 1\n\tglobal_atomic_add %0, %1, %2, %3 sc0\n\ts_mov_b64 exec, -1\n\ts_waitcnt vmcnt(0)"
                   : "=&v"(tk) : "v"(0u), "v"(1u), "s"(tick) : "memory");
      cur = __builtin_amdgcn_readfirstlane(tk) - ((call - 1) * NU - NPW);
      const bool more = cur < NU;
      if (more) fetch(cur);
      else if (lane == 0) gone[wave] = 1;
      if (!more) break;
      turn += PW;
      first = false;
    }
    if (lane == 0) __hip_atomic_store(cntp, call, __ATOMIC_RELAXED, __HIP_MEMORY_SCOPE_AGENT);
    STAMP(2);
    asm volatile("s_waitcnt vmcnt(0)" ::: "memory");
    STAMP(3);
    return;
  }

  const unsigned t0 = (unsigned)now();
  const int rank = P.rank;
  const int ci = (b >> 3) - 4, lo = (b & 7) * 4 + ci - 24;
  const int set = ci < 24 ? (ci >> 3) * 8 + (b & 7) : 24 + (lo >> 3);
  const int nj = ci < 24 ? (ci & 7) : (lo & 7);
  const int s = (rank + 1 + (set >> 2)) & (R - 1);
  const unsigned rb = set & 3;
  unsigned* cntp = (unsigned*)(myws + CNT_OFF + b * 64);
  const unsigned prev = __hip_atomic_load(cntp, __ATOMIC_RELAXED, __HIP_MEMORY_SCOPE_AGENT);
  const float scale = P.sx[0] * P.sw[0];
  LAS unsigned* arrived = (LAS unsigned*)((LAS uint8_t*)lds + LDS_F);
  if (threadIdx.x < 4) arrived[threadIdx.x] = 0;
  const unsigned q = lane >> 4, li = lane & 15;

  i32x8 wr[4][4];
  {
    const unsigned row = lane >> 2;
    const uint8_t* wsrc = P.w + (size_t)(wave * 512) * NL + nj * 64;
    const unsigned c0 = row * NL + (((lane & 3) ^ ((row >> 2) & 1)) << 4);
    LAS uint8_t* stg = (LAS uint8_t*)lds + LDS_P + wave * 16384;
    auto stage = [&](int kb) {
#pragma unroll
      for (int i = 0; i < 8; ++i) {
        unsigned keep;
        asm volatile("s_mov_b32 %0, m0\n\ts_mov_b32 m0, %3\n\ts_nop 0\n\tglobal_load_lds_dwordx4 %1, %2\n\ts_mov_b32 m0, %0"
                     : "=&s"(keep) : "v"(c0 ^ ((i & 1) << 5)), "s"(wsrc + (kb * 128 + i * 16) * NL),
                       "s"((unsigned)(uintptr_t)(stg + (kb & 1) * 8192 + i * 1024)) : "memory");
      }
    };
    const unsigned f = ((li >> 3) & 1) | ((q & 1) << 1);
    const unsigned lb = (q * 16 + (li >> 1)) * 64 + (li & 1) * 8;
    auto unstage = [&](int kb) {
#pragma unroll
      for (int t = 0; t < 4; ++t)
#pragma unroll
        for (int j = 0; j < 4; ++j) {
          const i32x2 r = __builtin_amdgcn_ds_read_tr8_b64_v2i32(
              (LAS i32x2*)(stg + (kb & 1) * 8192 + lb + ((t ^ f) << 4) + (j >> 1) * 4096 + (j & 1) * 512));
          wr[kb][t][2 * j] = r[0]; wr[kb][t][2 * j + 1] = r[1];
        }
      asm volatile("s_waitcnt lgkmcnt(0)" : "+v"(wr[kb][0]), "+v"(wr[kb][1]), "+v"(wr[kb][2]), "+v"(wr[kb][3]) :: "memory");
    };
    {
      const unsigned z = (b - NPROD) * 8 + wave;
      const __amdgpu_buffer_rsrc_t all = mk_rsrc(P.w, (unsigned)(KD * NL));
      const u32x4 v0 = __builtin_amdgcn_raw_buffer_load_b128(all, lane * 16, z * 1024, 0);
      const u32x4 v1 = __builtin_amdgcn_raw_buffer_load_b128(all, lane * 16, (z + 1792) * 1024, 0);
      asm volatile("" :: "v"(v0), "v"(v1));
    }
    stage(0); stage(1);
    asm volatile("s_waitcnt vmcnt(8)" ::: "memory");
    unstage(0); stage(2);
    asm volatile("s_waitcnt vmcnt(8)" ::: "memory");
    unstage(1); stage(3);
    asm volatile("s_waitcnt vmcnt(8)" ::: "memory");
    unstage(2);
    asm volatile("s_waitcnt vmcnt(0)" ::: "memory");
    unstage(3);
  }
  __builtin_amdgcn_s_barrier();
  asm volatile("" ::: "memory");
  const unsigned call = __builtin_amdgcn_readfirstlane(prev) + 1;
  if (threadIdx.x == 0) __hip_atomic_store(cntp, call, __ATOMIC_RELAXED, __HIP_MEMORY_SCOPE_AGENT);
  STAMP(1);

  unsigned* tmo = (unsigned*)(myws + TMO_OFF);
  const __amdgpu_buffer_rsrc_t orr = mk_rsrc(P.out, (unsigned)(R * MS * NL * 4));
  LAS uint8_t* hb = (LAS uint8_t*)lds + LDS_H + wave * 1024;
  bool dead = false;
  unsigned seq = 0, spins = 0;
  f32x4 acc[4];
#define MMA(xa, kb) do { _Pragma("unroll") for (int t_ = 0; t_ < 4; ++t_) \
    acc[t_] = __builtin_amdgcn_mfma_scale_f32_16x16x128_f8f6f4(wr[kb][t_], (xa), acc[t_], 0, 0, 0, 0, 0, 0); } while (0)
  auto lds_word = [&](unsigned i) { return __builtin_amdgcn_readfirstlane(*(volatile LAS unsigned*)(arrived + i)); };
  unsigned todo = ~0u, todo_row = 0;
  auto finalize = [&](bool block) {
    if (todo == ~0u) return;
    while (lds_word(todo & 1) < 8 * ((todo >> 1) + 1))
      if (!block) return;
    asm volatile("" ::: "memory");
    __builtin_amdgcn_s_setprio(2);
    const LAS uint8_t* part = (const LAS uint8_t*)lds + LDS_P + (todo & 1) * 32768 + (wave & 3) * 1024 + lane * 16;
    f32x4 v[5];
#pragma unroll
    for (int w = 0; w < 5; ++w) v[w] = *(const LAS f32x4*)(part + w * 4096);
    f32x4 sum = (v[0] + v[1]) + (v[2] + v[3]);
    asm volatile("" : "+v"(sum));
#pragma unroll
    for (int w = 0; w < 3; ++w) v[w] = *(const LAS f32x4*)(part + (5 + w) * 4096);
    sum += (v[4] + v[0]) + (v[1] + v[2]);
    f32x4 o;
#pragma unroll
    for (int r = 0; r < 4; ++r) o[r] = silu(sum[r], scale);
    __builtin_amdgcn_raw_buffer_store_b128(__builtin_bit_cast(u32x4, o), orr, (li * NL + q * 4) * 4,
                                           (todo_row * NL + nj * 64 + (wave & 3) * 16) * 4, AUX_SYS);
    stored(__builtin_bit_cast(u32x4, o));
    if (lane == 0) __hip_atomic_fetch_add(arrived + 2 + (todo & 1), 1u, __ATOMIC_RELAXED, __HIP_MEMORY_SCOPE_WORKGROUP);
    todo = ~0u;
    __builtin_amdgcn_s_setprio(0);
  };
  bool clear = true;
  auto check_clear = [&]() { if (!clear) clear = lds_word(2 + (seq & 1)) >= 4 * (seq >> 1); };
  auto deposit = [&](unsigned orow) {
    if (!clear) {
      if (todo == seq - 2) finalize(true);
      do check_clear(); while (!clear);
    }
    asm volatile("" ::: "memory");
    if (dead) {
#pragma unroll
      for (int t = 0; t < 4; ++t) acc[t][0] = acc[t][1] = acc[t][2] = acc[t][3] = __builtin_nanf("");
    }
    LAS uint8_t* part = (LAS uint8_t*)lds + LDS_P + (seq & 1) * 32768 + wave * 4096 + lane * 16;
#pragma unroll
    for (int t = 0; t < 4; ++t) *(LAS f32x4*)(part + t * 1024) = acc[t];
    asm volatile("" ::: "memory");
    if (lane == 0) __hip_atomic_fetch_add(arrived + (seq & 1), 1u, __ATOMIC_RELAXED, __HIP_MEMORY_SCOPE_WORKGROUP);
    if ((unsigned)(wave >> 2) == (seq & 1)) { todo = seq; todo_row = orow; }
    ++seq;
    clear = seq < 2;
  };

  const __amdgpu_buffer_rsrc_t ar = mk_rsrc(myws + (size_t)(call % NSLOT) * SLOT + (size_t)s * SH2, SH2);
  const uint64_t abase = (uint64_t)(myws + (size_t)(call % NSLOT) * SLOT + (size_t)s * SH2);
  const u32x4 asrd = {(unsigned)abase, (unsigned)(abase >> 32) & 0xffffu, SH2, 0x00020000u};
  const unsigned l16 = lane * 16, l8 = lane * 8;
  const unsigned po0 = (rb * 8 + wave) * USTR;
  unsigned look = (nj - wave) & 7;
  u32x4 pa0, pb0, pc0, pa1, pb1, pc1, w0, w1;
  u32x2 pt0, pt1;
#define REQUEST(c, u, on, pt, pa, pb, pc, w) \
    asm volatile("s_cmp_eq_u32 %9, 0\n\ts_cbranch_scc1 1f\n\t" \
                 "buffer_load_dwordx2 %0, %6, %7, %8 offen offset:3072 sc0 sc1\n\t" \
                 "buffer_load_dwordx4 %4, %5, %7, %8 offen offset:3584 sc0 sc1\n\t" \
                 "buffer_load_dwordx4 %1, %5, %7, %8 offen sc0 sc1\n\t" \
                 "buffer_load_dwordx4 %2, %5, %7, %8 offen offset:1024 sc0 sc1\n\t" \
                 "buffer_load_dwordx4 %3, %5, %7, %8 offen offset:2048 sc0 sc1\n1:" \
                 : c(pt), c(pa), c(pb), c(pc), c(w) \
                 : "v"(l16), "v"(l8), "s"(asrd), "s"(po0 + (u) * (32 * USTR)), "s"(on) : "memory", "scc")
#define LANDED(n, pt, pa, pb, pc, w) \
    asm volatile("s_waitcnt vmcnt(" #n ")" : "+v"(pt), "+v"(pa), "+v"(pb), "+v"(pc), "+v"(w) :: "memory")
#define UNPACK(fr, wa, wb, keyA, keyB, base, x0, x1, x2, x3, x4, x5) { \
      const unsigned base4 = base; \
      i32x8 xa; \
      xa[0] = (int)((bfi(M01 * 0x60, (wa) << 5, x0) ^ keyA) + base4); \
      xa[1] = (int)((bfi(M01 * 0x60, (wa) << 3, x1) ^ keyA) + base4); \
      xa[2] = (int)((bfi(M01 * 0x60, (wa) << 1, x2) ^ keyA) + base4); \
      xa[3] = (int)((bfi(M01 * 0x60, (wa) >> 1, x3) ^ keyA) + base4); \
      xa[4] = (int)((bfi(M01 * 0x60, (wb) << 5, x4) ^ keyA) + base4); \
      xa[5] = (int)((bfi(M01 * 0x60, (wb) << 3, x5) ^ keyA) + base4); \
      xa[6] = (int)((bfi(M01 * 3, (x0) >> 5, bfi(M01 * 0xc, (x1) >> 3, bfi(M01 * 0x10, (x2) >> 1, \
                     bfi(M01 * 0x80, (x2) << 1, (wb) << 1)))) ^ keyB) + base4); \
      xa[7] = (int)((bfi(M01 * 3, (x3) >> 5, bfi(M01 * 0xc, (x4) >> 3, bfi(M01 * 0x10, (x5) >> 1, \
                     bfi(M01 * 0x80, (x5) << 1, (wb) >> 1)))) ^ keyB) + base4); \
      MMA(xa, fr); }
#define FULL(v) none(min(min((v).x, (v).y), min((v).z, (v).w)) == 0)
  const unsigned sink = (unsigned)(uintptr_t)((LAS uint8_t*)lds + LDS_S) + threadIdx.x * 4;
  constexpr unsigned KD_LIST = 0xaaaaaaaau;
  auto lists = [&](const u32x2& pt, u32x4& w) {
    const unsigned meta = __builtin_amdgcn_readlane(pt.y, 0);
    const unsigned nl = meta & 0x7fu, l0 = (meta >> 8) & 0x7fu;
    if ((meta >> 30) != 2 || !none(lane < nl && min(pt.x, pt.y) == 0)) return false;
    *(LAS u32x4*)(hb + l16) = u32x4{0u, 0u, 0u, 0u};
    const bool on = lane - 1 < nl - 1, isb = lane < l0;
    const unsigned lo = on ? pt.x : ~0u, hi = on ? pt.y : ~0u;
    const unsigned F = M01 * 15;
    const unsigned tn = __builtin_amdgcn_sad_u8((lo & F) + (hi & F) + ((lo >> 4) & F) + ((hi >> 4) & F), 0u, 0u);
    const unsigned tb = __builtin_amdgcn_sad_u8(lo, 0u, __builtin_amdgcn_sad_u8(hi, 0u, 0u));
    const unsigned tot = lane == 0 ? 0u : (isb ? tb : tn);
    const unsigned incl = scan_add(tot);
    unsigned cur = (incl - tot - 1 + (isb ? 4096u : 0u - __builtin_amdgcn_readlane(incl, l0 - 1))) * 8193u +
                   ((unsigned)(uintptr_t)hb << 16);
    const unsigned sw = isb ? 8u : 4u, esc = isb ? 255u : 15u;
    const unsigned src[4] = {lo, isb ? hi : lo >> 16, isb ? ~0u : hi, isb ? ~0u : hi >> 16};
#pragma unroll
    for (int i = 0; i < 16; ++i) {
      const unsigned sy = __builtin_amdgcn_ubfe(src[i >> 2], sw * (i & 3), sw);
      cur += __umul24(sy, 8193u);
      __hip_atomic_fetch_xor((LAS unsigned*)(uintptr_t)(sy != esc ? (cur >> 16) & 0xfffcu : sink), 1u << (cur & 31),
                             __ATOMIC_RELAXED, __HIP_MEMORY_SCOPE_WAVEFRONT);
    }
    asm volatile("" ::: "memory");
    const u32x2 X = *(const LAS u32x2*)(hb + l8);
    const u32x2 Y = X ^ *(const LAS u32x2*)(hb + 512 + l8);
    w = u32x4{bfi(M55, X.x, Y.x << 1), bfi(M55, X.x >> 1, Y.x), bfi(M55, X.y, Y.y << 1), bfi(M55, X.y >> 1, Y.y)};
    return true;
  };
  auto leave = [&](const u32x4& w, unsigned uo) {
    const u32x4 v = w ^ KD_LIST;
    __builtin_amdgcn_raw_buffer_store_b128(v, ar, l16 + U_PLANE, uo, AUX_SYS);
    stored(v);
  };
  auto look_load = [&]() { return __builtin_amdgcn_raw_buffer_load_b64(ar, l8 + U_T, po0 + look * (32 * USTR), AUX_SYS); };
  auto look_decode = [&](const u32x2& pt) {
    u32x4 w;
    __builtin_amdgcn_s_setprio(3);
    const bool ok = lists(pt, w);
    __builtin_amdgcn_s_setprio(0);
    if (ok) { leave(w, po0 + look * (32 * USTR)); look += 8; }
    else if (__builtin_amdgcn_readlane(pt.y, 0) >> 30 == 3) look += 8;
  };
  if (look < 4) look_decode(look_load());
  {
    const __amdgpu_buffer_rsrc_t xr = mk_rsrc(xp, SHARD);
#pragma unroll 1
    for (int lblk = set; lblk < 32; lblk += 28) {
      u32x4 a[8];
#pragma unroll
      for (int i = 0; i < 8; ++i)
        a[i] = __builtin_amdgcn_raw_buffer_load_b128(xr, li * KD + q * 16 + i * 64, lblk * 16 * KD + wave * 512, 0);
      __builtin_amdgcn_sched_barrier(0);
#pragma unroll
      for (int t = 0; t < 4; ++t) acc[t] = f32x4{0.f, 0.f, 0.f, 0.f};
#pragma unroll
      for (int i = 0; i < 4; ++i) {
        const i32x8 xa = {(int)a[2 * i].x, (int)a[2 * i].y, (int)a[2 * i].z, (int)a[2 * i].w,
                          (int)a[2 * i + 1].x, (int)a[2 * i + 1].y, (int)a[2 * i + 1].z, (int)a[2 * i + 1].w};
        MMA(xa, i);
      }
      deposit(rank * MS + lblk * 16);
    }
    STAMP(7);
  }

#pragma unroll
  for (int t = 0; t < 4; ++t) acc[t] = f32x4{0.f, 0.f, 0.f, 0.f};
#define DBG_FIN() do { } while (0)
  const unsigned pconst = lane == 0 ? 3068 : (lane == 1 ? U_PLANE + 1020 : (lane == 2 ? U_T + 4 : SH2));
  auto probe = [&](unsigned u) {
    return __builtin_amdgcn_raw_buffer_load_b32(ar, pconst + (lane == 2 ? look * (32 * USTR) : u * (32 * USTR)), po0, AUX_SYS);
  };
  unsigned key, meta;
#define ACQUIRE(u, asked, pt, pa, pb, pc, fw) { \
    unsigned tries = 0; \
    for (;;) { \
      if (!asked) for (;;) { \
        const unsigned v = probe(u); \
        check_clear(); \
        const unsigned seen = (unsigned)__builtin_amdgcn_ballot_w64(v != 0); \
        if ((seen & 1u) && ((seen & 2u) || look == (u) || ++tries >= DLIMIT)) break; \
        if ((seen & 4u) && look < 16) look_decode(look_load()); \
        if ((++spins & 63) == 0 && timed_out()) dead = true; \
        if (dead) break; \
        finalize(false); \
        if ((u) < 13) __builtin_amdgcn_s_sleep(POLL_SLEEP); \
      } \
      REQUEST("+v", u, asked ^ 1u, pt, pa, pb, pc, fw); \
      LANDED(0, pt, pa, pb, pc, fw); \
      const bool data = none(umin3(umin3(umin3(pa.x, pa.y, pa.z), umin3(pa.w, pb.x, pb.y), umin3(pb.z, pb.w, pc.x)), \
                                   min(pc.y, pc.z), pc.w) == 0); \
      const bool flds = FULL(fw); \
      key = __builtin_amdgcn_readlane(pt.x, 0); meta = __builtin_amdgcn_readlane(pt.y, 0); \
      unsigned kd = KD_LIST; \
      if (__builtin_expect(meta & 0x40000000u, 0)) kd = __builtin_amdgcn_readlane(pt.x, 1); \
      bool done = dead; \
      if (data && min(min(key, meta), kd) != 0) { \
        if (__builtin_expect(flds, 1)) { fw ^= kd; done = true; } \
        else if (lists(pt, fw)) { \
          done = true; \
          if (look == (u)) leave(fw, po0 + (u) * (32 * USTR)); \
        } \
      } \
      if (done) break; \
      if ((++spins & 63) == 0 && timed_out()) dead = true; \
      asked = 0; \
    } \
    if (look == (u)) look += 8; }
#define UNIT(i, pt, pa, pb, pc, fw, qt, qa, qb, qc, qw) { \
    const unsigned u = 2 * p + (i); \
    const unsigned pv = probe(u + 1); \
    if (u < 14) finalize(false); \
    if (u == 8) rezero(); \
    __builtin_amdgcn_s_setprio(2); \
    unsigned ka = key & ~(M01 * 0x60), ba = ((meta >> 16) & 0x7fu) * M01; \
    unsigned kb = ((key >> 5) & (M01 * 3)) | ((key >> 3) & (M01 * 0xc)) | ((key >> 1) & (M01 * 0x10)) | ((key << 1) & (M01 * 0x80)); \
    asm volatile("" : "+s"(ka), "+s"(kb), "+v"(ba));        \
    UNPACK(2 * (i), fw.x, fw.y, ka, kb, ba, pa.x, pa.y, pa.z, pa.w, pb.x, pb.y) \
    __builtin_amdgcn_sched_barrier(0); \
    const unsigned ahead = (unsigned)__builtin_amdgcn_ballot_w64(pv != 0); \
    unsigned asked = ((ahead & 3u) + 1u) >> 2; \
    REQUEST("=&v", u + 1, asked, qt, qa, qb, qc, qw); \
    __builtin_amdgcn_sched_barrier(0); \
    UNPACK(2 * (i) + 1, fw.z, fw.w, ka, kb, ba, pb.z, pb.w, pc.x, pc.y, pc.z, pc.w) \
    if (i) { \
      deposit(s * MS + p * 64 + rb * 16); \
      _Pragma("unroll") for (int t = 0; t < 4; ++t) acc[t] = f32x4{0.f, 0.f, 0.f, 0.f}; \
      DBG_FIN(); \
    } \
    __builtin_amdgcn_s_setprio(0); \
    if (u == 15) break; \
    if ((ahead & 4u) && look < 16 && look != u + 1) look_decode(look_load()); \
    ACQUIRE(u + 1, asked, qt, qa, qb, qc, qw) }
  auto timed_out = [&]() {
    if (__builtin_amdgcn_readfirstlane(__hip_atomic_load(tmo, __ATOMIC_RELAXED, __HIP_MEMORY_SCOPE_AGENT)) != call &&
        (unsigned)now() - t0 <= DEADLINE) return false;
    __hip_atomic_store(tmo, call, __ATOMIC_RELAXED, __HIP_MEMORY_SCOPE_AGENT);
    return true;
  };
  auto rezero = [&]() {
    const unsigned z = (set * 8 + nj) * 8 + wave;
    const __amdgpu_buffer_rsrc_t zr = mk_rsrc(myws + (size_t)((call + 2) % NSLOT) * SLOT, SLOT);
    const unsigned so = ((rank + 1 + (z >> 8)) & (R - 1)) * SH2 + (z & 255) * (2 * USTR);
    u32x4 zero = {0u, 0u, 0u, 0u};
    asm volatile("" : "+v"(zero));
#pragma unroll
    for (int i = 0; i < 9; ++i) __builtin_amdgcn_raw_buffer_store_b128(zero, zr, l16 + (i % 3) * 1024, so + (i / 3) * 3072, AUX_SYS);
    stored(zero);
  };
  {
    unsigned asked = 1;
    REQUEST("=&v", 0u, asked, pt0, pa0, pb0, pc0, w0);
    ACQUIRE(0u, asked, pt0, pa0, pb0, pc0, w0)
  }
#pragma unroll 1
  for (unsigned p = 0;; ++p) {
    UNIT(0, pt0, pa0, pb0, pc0, w0, pt1, pa1, pb1, pc1, w1)
    UNIT(1, pt1, pa1, pb1, pc1, w1, pt0, pa0, pb0, pc0, w0)
  }
  finalize(true);
  asm volatile("s_waitcnt vmcnt(0)" ::: "memory");
  STAMP(6);
}

extern "C" void kernel_launch(int rank, int nranks, ncclComm_t comm, hipStream_t stream,
                              void* const* d_in, const int* in_sizes, int n_in,
                              void* d_out, int out_size, const int* in_dtypes, int out_dtype,
                              void* const* d_out_all, void* const* d_ws_all, size_t ws_bytes) {
  (void)comm; (void)in_dtypes; (void)out_dtype; (void)d_out_all;
  if (nranks != R || (unsigned)rank >= (unsigned)R || n_in != 4 || ws_bytes < WS_NEED ||
      in_sizes[0] != (int)SHARD || in_sizes[1] != KD * NL || out_size != R * MS * NL) {
    fprintf(stderr, "unexpected problem shape\n");
    return;
  }
  static bool attr_set = false;
  if (!attr_set) {
    HIP_CHECK(hipFuncSetAttribute((const void*)ag_gemm_kernel, hipFuncAttributeMaxDynamicSharedMemorySize, LDS_BYTES));
    attr_set = true;
  }
  Params P;
  P.x = (const uint8_t*)d_in[0];
  P.w = (const uint8_t*)d_in[1];
  P.sx = (const float*)d_in[2];
  P.sw = (const float*)d_in[3];
  P.out = (float*)d_out;
  P.myws = (uint8_t*)d_ws_all[rank];
  for (int j = 0; j < R - 1; ++j) P.peer[j] = (uint8_t*)d_ws_all[(rank + 1 + j) & (R - 1)] + (size_t)rank * SH2;
  P.rank = rank;
  P.pad = 0;
  hipLaunchKernelGGL(ag_gemm_kernel, dim3(GRID), dim3(BLOCK), LDS_BYTES, stream, P);
  HIP_CHECK(hipGetLastError());
}
```

```cpp
#include <hip/hip_runtime.h>
#include <rccl/rccl.h>
#include <cstdint>
#include <cstdio>

#define HIP_CHECK(expr) do { hipError_t _e = (expr); if (_e != hipSuccess) { \
    fprintf(stderr, "HIP error %s at %s:%d\n", hipGetErrorString(_e), __FILE__, __LINE__); \
    return; } } while (0)

typedef unsigned u32x4 __attribute__((ext_vector_type(4)));
typedef unsigned u32x2 __attribute__((ext_vector_type(2)));
typedef int i32x2 __attribute__((ext_vector_type(2)));
typedef int i32x8 __attribute__((ext_vector_type(8)));
typedef float f32x4 __attribute__((ext_vector_type(4)));
#define LAS __attribute__((address_space(3)))

#define HAND 10
#define TOPADD 224
#define DLIMIT 4
#define TOUCH 9
#define TEARLY 1
#define POLL_SLEEP 4
#define NPROD 32
#define PW 5
#define PACE 10

constexpr int R = 8;
constexpr int MS = 512, KD = 4096, NL = 512;
constexpr unsigned SHARD = MS * KD;
constexpr unsigned NU = 512;
constexpr unsigned U_T = 3072;
constexpr unsigned U_PLANE = U_T + 512;
constexpr unsigned USTR = U_PLANE + 1024;
constexpr unsigned SH2 = NU * USTR;
constexpr unsigned SLOT = R * SH2;
constexpr int NSLOT = 3;
constexpr size_t CTRL = (size_t)NSLOT * SLOT;
constexpr size_t CNT_OFF = CTRL;
constexpr size_t TMO_OFF = CTRL + 32768;
constexpr size_t TICK_OFF = CTRL + 36864;
constexpr size_t DBG_OFF = CTRL + 65536;
constexpr int GRID = 256, BLOCK = 512;
constexpr size_t WS_NEED = CTRL + 65536 + GRID * (BLOCK / 64) * 128;
constexpr int LDS_P = 8192;
constexpr int LDS_H = 0;
constexpr int LDS_F = LDS_P + 131072;
constexpr int LDS_S = LDS_F + 64;
constexpr int LDS_BYTES = LDS_S + 2048;
constexpr unsigned DEADLINE = 150000000u;
constexpr int AUX_SYS = 17;
constexpr unsigned M01 = 0x01010101u, M55 = 0x55555555u;

struct Params {
  const uint8_t* x;
  const uint8_t* w;
  const float* sx;
  const float* sw;
  float* out;
  uint8_t* myws;
  uint8_t* peer[R - 1];
  int rank;
  int pad;
};

__device__ __forceinline__ __amdgpu_buffer_rsrc_t mk_rsrc(const void* p, unsigned bytes) {
  return __builtin_amdgcn_make_buffer_rsrc((void*)p, (short)0, (int)bytes, 0x00020000);
}
__device__ __forceinline__ void stored(const u32x4& v) { asm volatile("s_nop 1" :: "v"(v) : "memory"); }
__device__ __forceinline__ unsigned long long now() { return __builtin_amdgcn_s_memrealtime(); }
__device__ __forceinline__ unsigned umin3(unsigned a, unsigned b, unsigned c) { return min(min(a, b), c); }
__device__ __forceinline__ unsigned bfi(unsigned m, unsigned a, unsigned b) {
  return __builtin_amdgcn_bitop3_b32(m, a, b, 0xCA);
}
__device__ __forceinline__ unsigned shl(unsigned v, int s) { return s >= 0 ? v << s : v >> -s; }
__device__ __forceinline__ float silu(float acc, float scale) {
  const float y = acc * scale;
  const float c = fminf(fmaxf(y, -60.f), 60.f);
  return y * __builtin_amdgcn_rcpf(1.f + __builtin_amdgcn_exp2f(c * -1.44269504088896341f));
}
__device__ __forceinline__ void ld_x(u32x4& d, unsigned off, const uint8_t* base) {
  asm volatile("global_load_dwordx4 %0, %1, %2" : "=v"(d) : "v"(off), "s"(base) : "memory");
}
template <int CTRL_, int RM> __device__ __forceinline__ unsigned dpp0(unsigned v) {
  return __builtin_amdgcn_update_dpp(0u, v, CTRL_, RM, 0xf, false);
}
__device__ __forceinline__ unsigned scan_add(unsigned v) {
  v += dpp0<0x111, 0xf>(v); v += dpp0<0x112, 0xf>(v); v += dpp0<0x114, 0xf>(v); v += dpp0<0x118, 0xf>(v);
  v += dpp0<0x142, 0xa>(v); v += dpp0<0x143, 0xc>(v);
  return v;
}
__device__ __forceinline__ unsigned scan_max_excl(unsigned v) {
  v = max(v, dpp0<0x111, 0xf>(v)); v = max(v, dpp0<0x112, 0xf>(v)); v = max(v, dpp0<0x114, 0xf>(v));
  v = max(v, dpp0<0x118, 0xf>(v)); v = max(v, dpp0<0x142, 0xa>(v)); v = max(v, dpp0<0x143, 0xc>(v));
  return dpp0<0x138, 0xf>(v);
}
__device__ __forceinline__ unsigned ffbl(unsigned v) { return (unsigned)__builtin_ctzg(v, -1); }
__device__ __forceinline__ unsigned bits(unsigned v) { return 32 - (unsigned)__builtin_clzg(v, 32); }
__device__ __forceinline__ bool none(bool c) { return __builtin_amdgcn_ballot_w64(c) == 0; }
__device__ __forceinline__ void lds_or(LAS uint8_t* a, unsigned v) {
  __hip_atomic_fetch_or((LAS unsigned*)a, v, __ATOMIC_RELAXED, __HIP_MEMORY_SCOPE_WAVEFRONT);
}

template <int SB> struct GapList {
  static constexpr unsigned E = (1u << SB) - 1, PER = 32 / SB;
  unsigned lo[2], hi[2], skip[2], len[2], off, total;
  bool fail;
  __device__ __forceinline__ void build(unsigned m0, unsigned m1, unsigned lane) {
    unsigned m[2] = {m0, m1}, thr[2];
    const unsigned e0 = bits(m0), e1 = bits(m1);
    const unsigned prev = scan_max_excl(m1 ? lane * 64 + 32 + e1 : (m0 ? lane * 64 + e0 : 0u));
    const unsigned pv[2] = {prev, m0 ? lane * 64 + e0 : prev};
#pragma unroll
    for (int h = 0; h < 2; ++h) {
      const unsigned gap = lane * 64 + 32 * h + ffbl(m[h]) + 1 - pv[h];
      skip[h] = m[h] ? (SB == 4 ? (gap * 4370u) >> 16 : (gap * 4113u) >> 20) : 0u;
      thr[h] = pv[h] + E * skip[h] + E - 1 - (lane * 64 + 32 * h);
      lo[h] = hi[h] = 0;
    }
#pragma unroll
    for (unsigned g = 0; g < 2 * PER; g += 4) {
      if (g > 0 && none((m[0] | m[1]) != 0)) break;
#pragma unroll
      for (unsigned i = g; i < g + 4; ++i)
#pragma unroll
        for (int h = 0; h < 2; ++h) {
          const unsigned bb = ffbl(m[h]);
          const unsigned u = __builtin_elementwise_sub_sat(thr[h], bb);
          if (i < PER) lo[h] |= u << (SB * i); else hi[h] |= u << (SB * (i - PER));
          thr[h] = min(bb, thr[h]) + E;
          m[h] &= m[h] - (u != 0 ? 1u : 0u);
        }
    }
    fail = !none((m[0] | m[1]) != 0);
#pragma unroll
    for (int h = 0; h < 2; ++h)
      len[h] = skip[h] + (hi[h] ? PER + (bits(hi[h]) + SB - 1) / SB : (bits(lo[h]) + SB - 1) / SB);
    const unsigned incl = scan_add(len[0] + len[1]);
    off = incl - len[0] - len[1];
    total = __builtin_amdgcn_readlane(incl, 63);
  }
  __device__ __forceinline__ void place(LAS uint8_t* img) {
#pragma unroll
    for (int h = 0; h < 2; ++h) {
      const unsigned o = (off + (h ? len[0] : 0u) + skip[h]) * SB, sh = o & 31;
      const unsigned long long v = (((unsigned long long)hi[h] << 32) | lo[h]) << sh;
      LAS uint8_t* a = img + ((o >> 5) << 2);
      lds_or(a, (unsigned)v); lds_or(a + 4, (unsigned)(v >> 32)); lds_or(a + 8, (hi[h] >> 1) >> (31 - sh));
    }
  }
};

#define STAMP(i) do { } while (0)

__global__ void __launch_bounds__(BLOCK, 2) ag_gemm_kernel(Params P) {
  extern __shared__ __attribute__((aligned(16))) uint8_t lds[];
  const unsigned lane = threadIdx.x & 63;
  const int wave = __builtin_amdgcn_readfirstlane(threadIdx.x >> 6);
  const int b = blockIdx.x;
  uint8_t* const myws = P.myws;
  const uint8_t* const xp = P.x;
  asm volatile("" :: "s"(myws), "s"(xp));

  if (__builtin_expect(b < NPROD, 1)) {
    if (wave >= PW) return;
    unsigned* cntp = (unsigned*)(myws + CNT_OFF + b * 64 + wave * 4);
    unsigned* tick = (unsigned*)(myws + TICK_OFF);
    constexpr unsigned NPW = NPROD * PW;
    const unsigned vo = (lane & 15) * KD + (lane >> 4) * 16;
    u32x4 d[4];
    auto fetch = [&](unsigned u) {
      const unsigned o = vo + ((u >> 6) * 64 + ((u >> 3) & 3) * 16) * KD + ((u & 7) * 4 + ((u >> 5) & 1) * 2) * 128;
      ld_x(d[0], o, xp); ld_x(d[1], o + 64, xp); ld_x(d[2], o + 128, xp); ld_x(d[3], o + 192, xp);
    };
    unsigned c, cur = b + NPROD * wave, call, turn = wave;
    fetch(cur);
    asm volatile("global_load_dword %0, %1, %2 sc1" : "=v"(c) : "v"(0u), "s"(cntp) : "memory");
    __amdgpu_buffer_rsrc_t pr[R - 1];
#pragma unroll
    for (int j = 0; j < R - 1; ++j) pr[j] = mk_rsrc(P.peer[j], (NSLOT - 1) * SLOT + SH2);
    LAS uint8_t* stg = (LAS uint8_t*)lds + wave * 1024;
    volatile LAS unsigned* token = (volatile LAS unsigned*)((LAS uint8_t*)lds + 8192);
    auto pass = [&](unsigned v) {
      if (lane == 0) __hip_atomic_fetch_max((LAS unsigned*)token, v, __ATOMIC_RELAXED, __HIP_MEMORY_SCOPE_WORKGROUP);
    };
    volatile LAS unsigned* gone = token + 1;
    if (threadIdx.x < 8) token[threadIdx.x] = 0;
    asm volatile("s_waitcnt lgkmcnt(0)" ::: "memory");
    __builtin_amdgcn_s_barrier();
    bool first = true;
    for (;;) {
      asm volatile("s_waitcnt vmcnt(0)" : "+v"(c), "+v"(d[0]), "+v"(d[1]), "+v"(d[2]), "+v"(d[3]) :: "memory");
      call = __builtin_amdgcn_readfirstlane(c) + 1;
      const unsigned so = (call % NSLOT) * SLOT + cur * USTR;
      const bool early = first && wave == 0;
      unsigned t[16], e[12];
#pragma unroll
      for (int j = 0; j < 16; ++j) t[j] = d[j >> 2][j & 3];
      unsigned mx = 0;
#pragma unroll
      for (int j = 0; j < 16; ++j) mx = max(mx, t[j] & 0x7f000000u);
      const unsigned top = (__builtin_amdgcn_readlane(scan_add(mx >> 24), 63) + TOPADD) >> 6;
      const unsigned base = top > 127 ? 96 : (top > 31 ? top - 31 : 0);
#pragma unroll
      for (int j = 0; j < 16; ++j) t[j] -= base * M01;
#pragma unroll
      for (int g = 0; g < 4; ++g) {
        const unsigned* tt = t + (g >> 1) * 8 + (g & 1) * 3;
        const unsigned z = t[(g >> 1) * 8 + 6 + (g & 1)];
        e[3 * g] = bfi(M01 * 0x60, z << 5, tt[0]);
        e[3 * g + 1] = bfi(M01 * 0x60, z << 3, tt[1]);
        e[3 * g + 2] = bfi(M01 * 0x20, z << 1, bfi(M01 * 0x40, z >> 1, tt[2]));
      }
      unsigned key = 0;
      for (;;) {
        key += 0x9E3779B1u;
        unsigned m = 0xffffffffu;
#pragma unroll
        for (int j = 0; j < 12; j += 2) m = umin3(m, e[j] ^ key, e[j + 1] ^ key);
        if (none(m == 0)) break;
      }
      const u32x4 pA = {e[0] ^ key, e[1] ^ key, e[2] ^ key, e[3] ^ key};
      const u32x4 pB = {e[4] ^ key, e[5] ^ key, e[6] ^ key, e[7] ^ key};
      const u32x4 pC = {e[8] ^ key, e[9] ^ key, e[10] ^ key, e[11] ^ key};
#define PIECE(v, off) do { _Pragma("unroll") for (int j = 0; j < R - 1; ++j) \
        __builtin_amdgcn_raw_buffer_store_b128((v), pr[j], lane * 16 + (off), so, AUX_SYS); \
      stored(v); \
      __builtin_amdgcn_sched_barrier(0); } while (0)
      if (early) {
        PIECE(pA, 0);
        PIECE(pB, 1024); PIECE(pC, 2048);
        pass(1);
      }
      unsigned X[2], Y[2], w[4] = {0u, 0u, 0u, 0u};
#pragma unroll
      for (int j = 0; j < 16; ++j) w[j >> 2] = bfi((M01 * 3) << (2 * (j & 3)), shl(t[j], 2 * (j & 3) - 5), w[j >> 2]);
#pragma unroll
      for (int f = 0; f < 2; ++f) {
        X[f] = bfi(M55, w[2 * f], w[2 * f + 1] << 1);
        Y[f] = bfi(M55, w[2 * f] >> 1, w[2 * f + 1]);
      }
      *(LAS u32x4*)(stg + lane * 16) = u32x4{0u, 0u, 0u, 0u};
      GapList<4> G;
      GapList<8> H;
      G.build(X[0], X[1], lane);
      H.build(X[0] ^ Y[0], X[1] ^ Y[1], lane);
      unsigned l0 = 1 + ((H.total + 7) >> 3);
      unsigned nl = l0 + ((G.total + 15) >> 4);
      const bool dense = G.fail || H.fail || nl > 64;
      unsigned k2 = 0;
      if (!dense) {
        H.place(stg + 8);
        G.place(stg + l0 * 8);
      } else {
        nl = 2; l0 = 1;
        do k2 += 0x9E3779B1u; while (!none(w[0] == k2 || w[1] == k2 || w[2] == k2 || w[3] == k2));
      }
      asm volatile("" ::: "memory");
      u32x4 pT = ~*(const LAS u32x4*)(stg + lane * 16);
      if (lane == 0) {
        pT.x = key;
        pT.y = 0x80000000u | (dense ? 0x40000000u : 0u) | (base << 16) | (l0 << 8) | nl;
        if (dense) pT.z = k2;
      }
      const unsigned nT = ((nl + 7) >> 3) * 4;
      if (TEARLY && lane < nT) PIECE(pT, U_T);
      if (!early) {
        for (;;) {
          unsigned need = turn;
#pragma unroll
          for (unsigned k = 1; k < PW; ++k)
            if (need == turn - (k - 1) && __builtin_amdgcn_readfirstlane(gone[(wave + PW - k) % PW])) --need;
          if (__builtin_amdgcn_readfirstlane(*token) >= need) break;
          __builtin_amdgcn_s_sleep(1);
        }
      }
      if (!TEARLY && lane < nT) PIECE(pT, U_T);
      if (dense) {
        const u32x4 pD = {w[0] ^ k2, w[1] ^ k2, w[2] ^ k2, w[3] ^ k2};
        PIECE(pD, U_PLANE);
      }
      if (!early) {
        PIECE(pA, 0);
        asm volatile("s_waitcnt vmcnt(%0)" :: "n"(PACE) : "memory");
        PIECE(pB, 1024);
        asm volatile("s_waitcnt vmcnt(%0)" :: "n"(PACE) : "memory");
        PIECE(pC, 2048);
        asm volatile("s_waitcnt vmcnt(%0)" :: "n"(HAND) : "memory");
        pass(turn + 1);
      }
      unsigned tk;
      asm volatile("s_mov_b64 exec, 1\n\tglobal_atomic_add %0, %1, %2, %3 sc0\n\ts_mov_b64 exec, -1\n\ts_waitcnt vmcnt(0)"
                   : "=&v"(tk) : "v"(0u), "v"(1u), "s"(tick) : "memory");
      cur = __builtin_amdgcn_readfirstlane(tk) - ((call - 1) * NU - NPW);
      const bool more = cur < NU;
      if (more) fetch(cur);
      else if (lane == 0) gone[wave] = 1;
      if (!more) break;
      turn += PW;
      first = false;
    }
    if (lane == 0) __hip_atomic_store(cntp, call, __ATOMIC_RELAXED, __HIP_MEMORY_SCOPE_AGENT);
    STAMP(2);
    asm volatile("s_waitcnt vmcnt(0)" ::: "memory");
    STAMP(3);
    return;
  }

  if (TOUCH && b < NPROD + 8 && wave == 7 && lane < TOUCH) {
    const unsigned o = (lane / 3) * SLOT + (lane % 3) * (SH2 / 2 - 4);
#pragma unroll
    for (int i = 0; i < R - 1; ++i) {
      unsigned v;
      asm volatile("global_load_dword %0, %1, %2" : "=&v"(v) : "v"(o), "s"(P.peer[i]) : "memory");
    }
    asm volatile("s_waitcnt vmcnt(0)" ::: "memory");
  }
  const unsigned t0 = (unsigned)now();
  const int rank = P.rank;
  const int ci = (b >> 3) - 4, lo = (b & 7) * 4 + ci - 24;
  const int set = ci < 24 ? (ci >> 3) * 8 + (b & 7) : 24 + (lo >> 3);
  const int nj = ci < 24 ? (ci & 7) : (lo & 7);
  const int s = (rank + 1 + (set >> 2)) & (R - 1);
  const unsigned rb = set & 3;
  unsigned* cntp = (unsigned*)(myws + CNT_OFF + b * 64);
  const unsigned prev = __hip_atomic_load(cntp, __ATOMIC_RELAXED, __HIP_MEMORY_SCOPE_AGENT);
  const float scale = P.sx[0] * P.sw[0];
  LAS unsigned* arrived = (LAS unsigned*)((LAS uint8_t*)lds + LDS_F);
  if (threadIdx.x < 4) arrived[threadIdx.x] = 0;
  const unsigned q = lane >> 4, li = lane & 15;

  i32x8 wr[4][4];
  {
    const unsigned row = lane >> 2;
    const uint8_t* wsrc = P.w + (size_t)(wave * 512) * NL + nj * 64;
    const unsigned c0 = row * NL + (((lane & 3) ^ ((row >> 2) & 1)) << 4);
    LAS uint8_t* stg = (LAS uint8_t*)lds + LDS_P + wave * 16384;
    auto stage = [&](int kb) {
#pragma unroll
      for (int i = 0; i < 8; ++i) {
        unsigned keep;
        asm volatile("s_mov_b32 %0, m0\n\ts_mov_b32 m0, %3\n\ts_nop 0\n\tglobal_load_lds_dwordx4 %1, %2\n\ts_mov_b32 m0, %0"
                     : "=&s"(keep) : "v"(c0 ^ ((i & 1) << 5)), "s"(wsrc + (kb * 128 + i * 16) * NL),
                       "s"((unsigned)(uintptr_t)(stg + (kb & 1) * 8192 + i * 1024)) : "memory");
      }
    };
    const unsigned f = ((li >> 3) & 1) | ((q & 1) << 1);
    const unsigned lb = (q * 16 + (li >> 1)) * 64 + (li & 1) * 8;
    auto unstage = [&](int kb) {
#pragma unroll
      for (int t = 0; t < 4; ++t)
#pragma unroll
        for (int j = 0; j < 4; ++j) {
          const i32x2 r = __builtin_amdgcn_ds_read_tr8_b64_v2i32(
              (LAS i32x2*)(stg + (kb & 1) * 8192 + lb + ((t ^ f) << 4) + (j >> 1) * 4096 + (j & 1) * 512));
          wr[kb][t][2 * j] = r[0]; wr[kb][t][2 * j + 1] = r[1];
        }
      asm volatile("s_waitcnt lgkmcnt(0)" : "+v"(wr[kb][0]), "+v"(wr[kb][1]), "+v"(wr[kb][2]), "+v"(wr[kb][3]) :: "memory");
    };
    {
      const unsigned z = (b - NPROD) * 8 + wave;
      const __amdgpu_buffer_rsrc_t all = mk_rsrc(P.w, (unsigned)(KD * NL));
      const u32x4 v0 = __builtin_amdgcn_raw_buffer_load_b128(all, lane * 16, z * 1024, 0);
      const u32x4 v1 = __builtin_amdgcn_raw_buffer_load_b128(all, lane * 16, (z + 1792) * 1024, 0);
      asm volatile("" :: "v"(v0), "v"(v1));
    }
    stage(0); stage(1);
    asm volatile("s_waitcnt vmcnt(8)" ::: "memory");
    unstage(0); stage(2);
    asm volatile("s_waitcnt vmcnt(8)" ::: "memory");
    unstage(1); stage(3);
    asm volatile("s_waitcnt vmcnt(8)" ::: "memory");
    unstage(2);
    asm volatile("s_waitcnt vmcnt(0)" ::: "memory");
    unstage(3);
  }
  __builtin_amdgcn_s_barrier();
  asm volatile("" ::: "memory");
  const unsigned call = __builtin_amdgcn_readfirstlane(prev) + 1;
  if (threadIdx.x == 0) __hip_atomic_store(cntp, call, __ATOMIC_RELAXED, __HIP_MEMORY_SCOPE_AGENT);
  STAMP(1);

  unsigned* tmo = (unsigned*)(myws + TMO_OFF);
  const __amdgpu_buffer_rsrc_t orr = mk_rsrc(P.out, (unsigned)(R * MS * NL * 4));
  LAS uint8_t* hb = (LAS uint8_t*)lds + LDS_H + wave * 1024;
  bool dead = false;
  unsigned seq = 0, spins = 0;
  f32x4 acc[4];
#define MMA(xa, kb) do { _Pragma("unroll") for (int t_ = 0; t_ < 4; ++t_) \
    acc[t_] = __builtin_amdgcn_mfma_scale_f32_16x16x128_f8f6f4(wr[kb][t_], (xa), acc[t_], 0, 0, 0, 0, 0, 0); } while (0)
  auto lds_word = [&](unsigned i) { return __builtin_amdgcn_readfirstlane(*(volatile LAS unsigned*)(arrived + i)); };
  unsigned todo = ~0u, todo_row = 0;
  auto finalize = [&](bool block) {
    if (todo == ~0u) return;
    while (lds_word(todo & 1) < 8 * ((todo >> 1) + 1))
      if (!block) return;
    asm volatile("" ::: "memory");
    __builtin_amdgcn_s_setprio(2);
    const LAS uint8_t* part = (const LAS uint8_t*)lds + LDS_P + (todo & 1) * 32768 + (wave & 3) * 1024 + lane * 16;
    f32x4 v[5];
#pragma unroll
    for (int w = 0; w < 5; ++w) v[w] = *(const LAS f32x4*)(part + w * 4096);
    f32x4 sum = (v[0] + v[1]) + (v[2] + v[3]);
    asm volatile("" : "+v"(sum));
#pragma unroll
    for (int w = 0; w < 3; ++w) v[w] = *(const LAS f32x4*)(part + (5 + w) * 4096);
    sum += (v[4] + v[0]) + (v[1] + v[2]);
    f32x4 o;
#pragma unroll
    for (int r = 0; r < 4; ++r) o[r] = silu(sum[r], scale);
    __builtin_amdgcn_raw_buffer_store_b128(__builtin_bit_cast(u32x4, o), orr, (li * NL + q * 4) * 4,
                                           (todo_row * NL + nj * 64 + (wave & 3) * 16) * 4, AUX_SYS);
    stored(__builtin_bit_cast(u32x4, o));
    if (lane == 0) __hip_atomic_fetch_add(arrived + 2 + (todo & 1), 1u, __ATOMIC_RELAXED, __HIP_MEMORY_SCOPE_WORKGROUP);
    todo = ~0u;
    __builtin_amdgcn_s_setprio(0);
  };
  bool clear = true;
  auto check_clear = [&]() { if (!clear) clear = lds_word(2 + (seq & 1)) >= 4 * (seq >> 1); };
  auto deposit = [&](unsigned orow) {
    if (!clear) {
      if (todo == seq - 2) finalize(true);
      do check_clear(); while (!clear);
    }
    asm volatile("" ::: "memory");
    if (dead) {
#pragma unroll
      for (int t = 0; t < 4; ++t) acc[t][0] = acc[t][1] = acc[t][2] = acc[t][3] = __builtin_nanf("");
    }
    LAS uint8_t* part = (LAS uint8_t*)lds + LDS_P + (seq & 1) * 32768 + wave * 4096 + lane * 16;
#pragma unroll
    for (int t = 0; t < 4; ++t) *(LAS f32x4*)(part + t * 1024) = acc[t];
    asm volatile("" ::: "memory");
    if (lane == 0) __hip_atomic_fetch_add(arrived + (seq & 1), 1u, __ATOMIC_RELAXED, __HIP_MEMORY_SCOPE_WORKGROUP);
    if ((unsigned)(wave >> 2) == (seq & 1)) { todo = seq; todo_row = orow; }
    ++seq;
    clear = seq < 2;
  };

  const __amdgpu_buffer_rsrc_t ar = mk_rsrc(myws + (size_t)(call % NSLOT) * SLOT + (size_t)s * SH2, SH2);
  const uint64_t abase = (uint64_t)(myws + (size_t)(call % NSLOT) * SLOT + (size_t)s * SH2);
  const u32x4 asrd = {(unsigned)abase, (unsigned)(abase >> 32) & 0xffffu, SH2, 0x00020000u};
  const unsigned l16 = lane * 16, l8 = lane * 8;
  const unsigned po0 = (rb * 8 + wave) * USTR;
  unsigned look = (nj - wave) & 7;
  u32x4 pa0, pb0, pc0, pa1, pb1, pc1, w0, w1;
  u32x2 pt0, pt1;
#define REQUEST(c, u, on, pt, pa, pb, pc, w) \
    asm volatile("s_cmp_eq_u32 %9, 0\n\ts_cbranch_scc1 1f\n\t" \
                 "buffer_load_dwordx2 %0, %6, %7, %8 offen offset:3072 sc0 sc1\n\t" \
                 "buffer_load_dwordx4 %4, %5, %7, %8 offen offset:3584 sc0 sc1\n\t" \
                 "buffer_load_dwordx4 %1, %5, %7, %8 offen sc0 sc1\n\t" \
                 "buffer_load_dwordx4 %2, %5, %7, %8 offen offset:1024 sc0 sc1\n\t" \
                 "buffer_load_dwordx4 %3, %5, %7, %8 offen offset:2048 sc0 sc1\n1:" \
                 : c(pt), c(pa), c(pb), c(pc), c(w) \
                 : "v"(l16), "v"(l8), "s"(asrd), "s"(po0 + (u) * (32 * USTR)), "s"(on) : "memory", "scc")
#define LANDED(n, pt, pa, pb, pc, w) \
    asm volatile("s_waitcnt vmcnt(" #n ")" : "+v"(pt), "+v"(pa), "+v"(pb), "+v"(pc), "+v"(w) :: "memory")
#define UNPACK(fr, wa, wb, keyA, keyB, base, x0, x1, x2, x3, x4, x5) { \
      const unsigned base4 = base; \
      i32x8 xa; \
      xa[0] = (int)((bfi(M01 * 0x60, (wa) << 5, x0) ^ keyA) + base4); \
      xa[1] = (int)((bfi(M01 * 0x60, (wa) << 3, x1) ^ keyA) + base4); \
      xa[2] = (int)((bfi(M01 * 0x60, (wa) << 1, x2) ^ keyA) + base4); \
      xa[3] = (int)((bfi(M01 * 0x60, (wa) >> 1, x3) ^ keyA) + base4); \
      xa[4] = (int)((bfi(M01 * 0x60, (wb) << 5, x4) ^ keyA) + base4); \
      xa[5] = (int)((bfi(M01 * 0x60, (wb) << 3, x5) ^ keyA) + base4); \
      xa[6] = (int)((bfi(M01 * 3, (x0) >> 5, bfi(M01 * 0xc, (x1) >> 3, bfi(M01 * 0x10, (x2) >> 1, \
                     bfi(M01 * 0x80, (x2) << 1, (wb) << 1)))) ^ keyB) + base4); \
      xa[7] = (int)((bfi(M01 * 3, (x3) >> 5, bfi(M01 * 0xc, (x4) >> 3, bfi(M01 * 0x10, (x5) >> 1, \
                     bfi(M01 * 0x80, (x5) << 1, (wb) >> 1)))) ^ keyB) + base4); \
      MMA(xa, fr); }
#define FULL(v) none(min(min((v).x, (v).y), min((v).z, (v).w)) == 0)
  const unsigned sink = (unsigned)(uintptr_t)((LAS uint8_t*)lds + LDS_S) + threadIdx.x * 4;
  constexpr unsigned KD_LIST = 0xaaaaaaaau;
  auto lists = [&](const u32x2& pt, u32x4& w) {
    const unsigned meta = __builtin_amdgcn_readlane(pt.y, 0);
    const unsigned nl = meta & 0x7fu, l0 = (meta >> 8) & 0x7fu;
    if ((meta >> 30) != 2 || !none(lane < nl && min(pt.x, pt.y) == 0)) return false;
    *(LAS u32x4*)(hb + l16) = u32x4{0u, 0u, 0u, 0u};
    const bool on = lane - 1 < nl - 1, isb = lane < l0;
    const unsigned lo = on ? pt.x : ~0u, hi = on ? pt.y : ~0u;
    const unsigned F = M01 * 15;
    const unsigned tn = __builtin_amdgcn_sad_u8((lo & F) + (hi & F) + ((lo >> 4) & F) + ((hi >> 4) & F), 0u, 0u);
    const unsigned tb = __builtin_amdgcn_sad_u8(lo, 0u, __builtin_amdgcn_sad_u8(hi, 0u, 0u));
    const unsigned tot = lane == 0 ? 0u : (isb ? tb : tn);
    const unsigned incl = scan_add(tot);
    unsigned cur = (incl - tot - 1 + (isb ? 4096u : 0u - __builtin_amdgcn_readlane(incl, l0 - 1))) * 8193u +
                   ((unsigned)(uintptr_t)hb << 16);
    const unsigned sw = isb ? 8u : 4u, esc = isb ? 255u : 15u;
    const unsigned src[4] = {lo, isb ? hi : lo >> 16, isb ? ~0u : hi, isb ? ~0u : hi >> 16};
#pragma unroll
    for (int i = 0; i < 16; ++i) {
      const unsigned sy = __builtin_amdgcn_ubfe(src[i >> 2], sw * (i & 3), sw);
      cur += __umul24(sy, 8193u);
      __hip_atomic_fetch_xor((LAS unsigned*)(uintptr_t)(sy != esc ? (cur >> 16) & 0xfffcu : sink), 1u << (cur & 31),
                             __ATOMIC_RELAXED, __HIP_MEMORY_SCOPE_WAVEFRONT);
    }
    asm volatile("" ::: "memory");
    const u32x2 X = *(const LAS u32x2*)(hb + l8);
    const u32x2 Y = X ^ *(const LAS u32x2*)(hb + 512 + l8);
    w = u32x4{bfi(M55, X.x, Y.x << 1), bfi(M55, X.x >> 1, Y.x), bfi(M55, X.y, Y.y << 1), bfi(M55, X.y >> 1, Y.y)};
    return true;
  };
  auto leave = [&](const u32x4& w, unsigned uo) {
    const u32x4 v = w ^ KD_LIST;
    __builtin_amdgcn_raw_buffer_store_b128(v, ar, l16 + U_PLANE, uo, AUX_SYS);
    stored(v);
  };
  auto look_load = [&]() { return __builtin_amdgcn_raw_buffer_load_b64(ar, l8 + U_T, po0 + look * (32 * USTR), AUX_SYS); };
  auto look_decode = [&](const u32x2& pt) {
    u32x4 w;
    __builtin_amdgcn_s_setprio(3);
    const bool ok = lists(pt, w);
    __builtin_amdgcn_s_setprio(0);
    if (ok) { leave(w, po0 + look * (32 * USTR)); look += 8; }
    else if (__builtin_amdgcn_readlane(pt.y, 0) >> 30 == 3) look += 8;
  };
  if (look < 4) look_decode(look_load());
  {
    const __amdgpu_buffer_rsrc_t xr = mk_rsrc(xp, SHARD);
#pragma unroll 1
    for (int lblk = set; lblk < 32; lblk += 28) {
      u32x4 a[8];
#pragma unroll
      for (int i = 0; i < 8; ++i)
        a[i] = __builtin_amdgcn_raw_buffer_load_b128(xr, li * KD + q * 16 + i * 64, lblk * 16 * KD + wave * 512, 0);
      __builtin_amdgcn_sched_barrier(0);
#pragma unroll
      for (int t = 0; t < 4; ++t) acc[t] = f32x4{0.f, 0.f, 0.f, 0.f};
#pragma unroll
      for (int i = 0; i < 4; ++i) {
        const i32x8 xa = {(int)a[2 * i].x, (int)a[2 * i].y, (int)a[2 * i].z, (int)a[2 * i].w,
                          (int)a[2 * i + 1].x, (int)a[2 * i + 1].y, (int)a[2 * i + 1].z, (int)a[2 * i + 1].w};
        MMA(xa, i);
      }
      deposit(rank * MS + lblk * 16);
    }
    STAMP(7);
  }

#pragma unroll
  for (int t = 0; t < 4; ++t) acc[t] = f32x4{0.f, 0.f, 0.f, 0.f};
#define DBG_FIN() do { } while (0)
  const unsigned pconst = lane == 0 ? 3068 : (lane == 1 ? U_PLANE + 1020 : (lane == 2 ? U_T + 4 : SH2));
  auto probe = [&](unsigned u) {
    return __builtin_amdgcn_raw_buffer_load_b32(ar, pconst + (lane == 2 ? look * (32 * USTR) : u * (32 * USTR)), po0, AUX_SYS);
  };
  unsigned key, meta;
#define ACQUIRE(u, asked, pt, pa, pb, pc, fw) { \
    unsigned tries = 0; \
    for (;;) { \
      if (!asked) for (;;) { \
        const unsigned v = probe(u); \
        check_clear(); \
        const unsigned seen = (unsigned)__builtin_amdgcn_ballot_w64(v != 0); \
        if ((seen & 1u) && ((seen & 2u) || look == (u) || ++tries >= DLIMIT)) break; \
        if ((seen & 4u) && look < 16) look_decode(look_load()); \
        if ((++spins & 63) == 0 && timed_out()) dead = true; \
        if (dead) break; \
        finalize(false); \
        if ((u) < 13) __builtin_amdgcn_s_sleep(POLL_SLEEP); \
      } \
      REQUEST("+v", u, asked ^ 1u, pt, pa, pb, pc, fw); \
      LANDED(0, pt, pa, pb, pc, fw); \
      const bool data = none(umin3(umin3(umin3(pa.x, pa.y, pa.z), umin3(pa.w, pb.x, pb.y), umin3(pb.z, pb.w, pc.x)), \
                                   min(pc.y, pc.z), pc.w) == 0); \
      const bool flds = FULL(fw); \
      key = __builtin_amdgcn_readlane(pt.x, 0); meta = __builtin_amdgcn_readlane(pt.y, 0); \
      unsigned kd = KD_LIST; \
      if (__builtin_expect(meta & 0x40000000u, 0)) kd = __builtin_amdgcn_readlane(pt.x, 1); \
      bool done = dead; \
      if (data && min(min(key, meta), kd) != 0) { \
        if (__builtin_expect(flds, 1)) { fw ^= kd; done = true; } \
        else if (lists(pt, fw)) { \
          done = true; \
          if (look == (u)) leave(fw, po0 + (u) * (32 * USTR)); \
        } \
      } \
      if (done) break; \
      if ((++spins & 63) == 0 && timed_out()) dead = true; \
      asked = 0; \
    } \
    if (look == (u)) look += 8; }
#define UNIT(i, pt, pa, pb, pc, fw, qt, qa, qb, qc, qw) { \
    const unsigned u = 2 * p + (i); \
    const unsigned pv = probe(u + 1); \
    if (u < 14) finalize(false); \
    if (u == 8) rezero(); \
    __builtin_amdgcn_s_setprio(2); \
    unsigned ka = key & ~(M01 * 0x60), ba = ((meta >> 16) & 0x7fu) * M01; \
    unsigned kb = ((key >> 5) & (M01 * 3)) | ((key >> 3) & (M01 * 0xc)) | ((key >> 1) & (M01 * 0x10)) | ((key << 1) & (M01 * 0x80)); \
    asm volatile("" : "+s"(ka), "+s"(kb), "+v"(ba));        \
    UNPACK(2 * (i), fw.x, fw.y, ka, kb, ba, pa.x, pa.y, pa.z, pa.w, pb.x, pb.y) \
    __builtin_amdgcn_sched_barrier(0); \
    const unsigned ahead = (unsigned)__builtin_amdgcn_ballot_w64(pv != 0); \
    unsigned asked = ((ahead & 3u) + 1u) >> 2; \
    REQUEST("=&v", u + 1, asked, qt, qa, qb, qc, qw); \
    __builtin_amdgcn_sched_barrier(0); \
    UNPACK(2 * (i) + 1, fw.z, fw.w, ka, kb, ba, pb.z, pb.w, pc.x, pc.y, pc.z, pc.w) \
    if (i) { \
      deposit(s * MS + p * 64 + rb * 16); \
      _Pragma("unroll") for (int t = 0; t < 4; ++t) acc[t] = f32x4{0.f, 0.f, 0.f, 0.f}; \
      DBG_FIN(); \
    } \
    __builtin_amdgcn_s_setprio(0); \
    if (u == 15) break; \
    if ((ahead & 4u) && look < 16 && look != u + 1) look_decode(look_load()); \
    ACQUIRE(u + 1, asked, qt, qa, qb, qc, qw) }
  auto timed_out = [&]() {
    if (__builtin_amdgcn_readfirstlane(__hip_atomic_load(tmo, __ATOMIC_RELAXED, __HIP_MEMORY_SCOPE_AGENT)) != call &&
        (unsigned)now() - t0 <= DEADLINE) return false;
    __hip_atomic_store(tmo, call, __ATOMIC_RELAXED, __HIP_MEMORY_SCOPE_AGENT);
    return true;
  };
  auto rezero = [&]() {
    const unsigned z = (set * 8 + nj) * 8 + wave;
    const __amdgpu_buffer_rsrc_t zr = mk_rsrc(myws + (size_t)((call + 2) % NSLOT) * SLOT, SLOT);
    const unsigned so = ((rank + 1 + (z >> 8)) & (R - 1)) * SH2 + (z & 255) * (2 * USTR);
    u32x4 zero = {0u, 0u, 0u, 0u};
    asm volatile("" : "+v"(zero));
#pragma unroll
    for (int i = 0; i < 9; ++i) __builtin_amdgcn_raw_buffer_store_b128(zero, zr, l16 + (i % 3) * 1024, so + (i / 3) * 3072, AUX_SYS);
    stored(zero);
  };
  {
    unsigned asked = 1;
    REQUEST("=&v", 0u, asked, pt0, pa0, pb0, pc0, w0);
    ACQUIRE(0u, asked, pt0, pa0, pb0, pc0, w0)
  }
#pragma unroll 1
  for (unsigned p = 0;; ++p) {
    UNIT(0, pt0, pa0, pb0, pc0, w0, pt1, pa1, pb1, pc1, w1)
    UNIT(1, pt1, pa1, pb1, pc1, w1, pt0, pa0, pb0, pc0, w0)
  }
  finalize(true);
  asm volatile("s_waitcnt vmcnt(0)" ::: "memory");
  STAMP(6);
}

extern "C" void kernel_launch(int rank, int nranks, ncclComm_t comm, hipStream_t stream,
                              void* const* d_in, const int* in_sizes, int n_in,
                              void* d_out, int out_size, const int* in_dtypes, int out_dtype,
                              void* const* d_out_all, void* const* d_ws_all, size_t ws_bytes) {
  (void)comm; (void)in_dtypes; (void)out_dtype; (void)d_out_all;
  if (nranks != R || (unsigned)rank >= (unsigned)R || n_in != 4 || ws_bytes < WS_NEED ||
      in_sizes[0] != (int)SHARD || in_sizes[1] != KD * NL || out_size != R * MS * NL) {
    fprintf(stderr, "unexpected problem shape\n");
    return;
  }
  static bool attr_set = false;
  if (!attr_set) {
    HIP_CHECK(hipFuncSetAttribute((const void*)ag_gemm_kernel, hipFuncAttributeMaxDynamicSharedMemorySize, LDS_BYTES));
    attr_set = true;
  }
  Params P;
  P.x = (const uint8_t*)d_in[0];
  P.w = (const uint8_t*)d_in[1];
  P.sx = (const float*)d_in[2];
  P.sw = (const float*)d_in[3];
  P.out = (float*)d_out;
  P.myws = (uint8_t*)d_ws_all[rank];
  for (int j = 0; j < R - 1; ++j) P.peer[j] = (uint8_t*)d_ws_all[(rank + 1 + j) & (R - 1)] + (size_t)rank * SH2;
  P.rank = rank;
  P.pad = 0;
  hipLaunchKernelGGL(ag_gemm_kernel, dim3(GRID), dim3(BLOCK), LDS_BYTES, stream, P);
  HIP_CHECK(hipGetLastError());
}
```

```cpp
#include <hip/hip_runtime.h>
#include <rccl/rccl.h>
#include <cstddef>
#include <cstdint>
#include <cstdio>

#define HIP_CHECK(expr) do { hipError_t _e = (expr); if (_e != hipSuccess) { \
    fprintf(stderr, "HIP error %s at %s:%d\n", hipGetErrorString(_e), __FILE__, __LINE__); \
    return; } } while (0)

typedef unsigned u32x4 __attribute__((ext_vector_type(4)));
typedef unsigned u32x2 __attribute__((ext_vector_type(2)));
typedef int i32x2 __attribute__((ext_vector_type(2)));
typedef int i32x8 __attribute__((ext_vector_type(8)));
typedef float f32x4 __attribute__((ext_vector_type(4)));
#define LAS __attribute__((address_space(3)))

constexpr int NPROD = 32;
constexpr int NSEND = 24;
constexpr int PW = 3;
constexpr int PACE = 16;
constexpr unsigned TOPADD = 224;
constexpr unsigned RFROM = 336;
constexpr unsigned RMAX = 16;
constexpr unsigned DLIMIT = 4;

constexpr int R = 8;
constexpr int MS = 512, KD = 4096, NL = 512;
constexpr unsigned SHARD = MS * KD;
constexpr unsigned NU = 512;
constexpr unsigned U_T = 3072;
constexpr unsigned U_PLANE = U_T + 512;
constexpr unsigned USTR = U_PLANE + 1024;
constexpr unsigned SH2 = NU * USTR;
constexpr unsigned SLOT = R * SH2;
constexpr int NSLOT = 3;
constexpr size_t CTRL = (size_t)NSLOT * SLOT;
constexpr size_t CNT_OFF = CTRL;
constexpr size_t TMO_OFF = CTRL + 32768;
constexpr size_t TICK_OFF = CTRL + 36864;
constexpr size_t XTRA_OFF = CTRL + 38912;
constexpr size_t PAIR_OFF = CTRL + 40960;
constexpr int GRID = 256, BLOCK = 512;
constexpr size_t WS_NEED = CTRL + 65536;
constexpr int LDS_P = 8192;
constexpr int LDS_H = 0;
constexpr int LDS_F = LDS_P + 131072;
constexpr int LDS_S = LDS_F + 64;
constexpr int LDS_BYTES = LDS_S + 2048;
constexpr unsigned DEADLINE = 150000000u;
constexpr int AUX_SYS = 17;
constexpr unsigned M01 = 0x01010101u, M55 = 0x55555555u;

struct Params {
  const uint8_t* x;
  const uint8_t* w;
  const float* sx;
  const float* sw;
  float* out;
  uint8_t* myws;
  uint8_t* peer[R - 1];
  int rank;
  int pad;
};

__device__ __forceinline__ __amdgpu_buffer_rsrc_t mk_rsrc(const void* p, unsigned bytes) {
  return __builtin_amdgcn_make_buffer_rsrc((void*)p, (short)0, (int)bytes, 0x00020000);
}
__device__ __forceinline__ void stored(const u32x4& v) { asm volatile("s_nop 1" :: "v"(v) : "memory"); }
__device__ __forceinline__ unsigned long long now() { return __builtin_amdgcn_s_memrealtime(); }
__device__ __forceinline__ unsigned umin3(unsigned a, unsigned b, unsigned c) { return min(min(a, b), c); }
__device__ __forceinline__ unsigned bfi(unsigned m, unsigned a, unsigned b) {
  return __builtin_amdgcn_bitop3_b32(m, a, b, 0xCA);
}
__device__ __forceinline__ unsigned shl(unsigned v, int s) { return s >= 0 ? v << s : v >> -s; }
__device__ __forceinline__ float silu(float acc, float scale) {
  const float y = acc * scale;
  const float c = fminf(fmaxf(y, -60.f), 60.f);
  return y * __builtin_amdgcn_rcpf(1.f + __builtin_amdgcn_exp2f(c * -1.44269504088896341f));
}
__device__ __forceinline__ void lds_count(const LAS unsigned* a) {
  asm volatile("s_mov_b64 exec, 1\n\tds_add_u32 %0, %1\n\ts_mov_b64 exec, -1"
               :: "v"((unsigned)(uintptr_t)a), "v"(1u) : "memory");
}
template <int CTRL_, int RM> __device__ __forceinline__ unsigned dpp0(unsigned v) {
  return __builtin_amdgcn_update_dpp(0u, v, CTRL_, RM, 0xf, false);
}
__device__ __forceinline__ unsigned scan_add(unsigned v) {
  v += dpp0<0x111, 0xf>(v); v += dpp0<0x112, 0xf>(v); v += dpp0<0x114, 0xf>(v); v += dpp0<0x118, 0xf>(v);
  v += dpp0<0x142, 0xa>(v); v += dpp0<0x143, 0xc>(v);
  return v;
}
__device__ __forceinline__ unsigned scan_max_excl(unsigned v) {
  v = max(v, dpp0<0x111, 0xf>(v)); v = max(v, dpp0<0x112, 0xf>(v)); v = max(v, dpp0<0x114, 0xf>(v));
  v = max(v, dpp0<0x118, 0xf>(v)); v = max(v, dpp0<0x142, 0xa>(v)); v = max(v, dpp0<0x143, 0xc>(v));
  return dpp0<0x138, 0xf>(v);
}
__device__ __forceinline__ unsigned ffbl(unsigned v) { return (unsigned)__builtin_ctzg(v, -1); }
__device__ __forceinline__ unsigned bits(unsigned v) { return 32 - (unsigned)__builtin_clzg(v, 32); }
__device__ __forceinline__ bool none(bool c) { return __builtin_amdgcn_ballot_w64(c) == 0; }
__device__ __forceinline__ void lds_or(LAS uint8_t* a, unsigned v) {
  __hip_atomic_fetch_or((LAS unsigned*)a, v, __ATOMIC_RELAXED, __HIP_MEMORY_SCOPE_WAVEFRONT);
}

template <int SB> struct GapList {
  static constexpr unsigned E = (1u << SB) - 1, PER = 32 / SB;
  unsigned lo[2], hi[2], skip[2], len[2], off, total;
  bool fail;
  __device__ __forceinline__ void build(unsigned m0, unsigned m1, unsigned lane) {
    unsigned m[2] = {m0, m1}, thr[2];
    const unsigned e0 = bits(m0), e1 = bits(m1);
    const unsigned prev = scan_max_excl(m1 ? lane * 64 + 32 + e1 : (m0 ? lane * 64 + e0 : 0u));
    const unsigned pv[2] = {prev, m0 ? lane * 64 + e0 : prev};
#pragma unroll
    for (int h = 0; h < 2; ++h) {
      const unsigned gap = lane * 64 + 32 * h + ffbl(m[h]) + 1 - pv[h];
      skip[h] = m[h] ? (SB == 4 ? (gap * 4370u) >> 16 : (gap * 4113u) >> 20) : 0u;
      thr[h] = pv[h] + E * skip[h] + E - 1 - (lane * 64 + 32 * h);
      lo[h] = hi[h] = 0;
    }
#pragma unroll
    for (unsigned g = 0; g < 2 * PER; g += 4) {
      if (g > 0 && none((m[0] | m[1]) != 0)) break;
#pragma unroll
      for (unsigned i = g; i < g + 4; ++i)
#pragma unroll
        for (int h = 0; h < 2; ++h) {
          const unsigned bb = ffbl(m[h]);
          const unsigned u = __builtin_elementwise_sub_sat(thr[h], bb);
          if (i < PER) lo[h] |= u << (SB * i); else hi[h] |= u << (SB * (i - PER));
          thr[h] = min(bb, thr[h]) + E;
          m[h] &= m[h] - (u != 0 ? 1u : 0u);
        }
    }
    fail = !none((m[0] | m[1]) != 0);
#pragma unroll
    for (int h = 0; h < 2; ++h)
      len[h] = skip[h] + (hi[h] ? PER + (bits(hi[h]) + SB - 1) / SB : (bits(lo[h]) + SB - 1) / SB);
    const unsigned incl = scan_add(len[0] + len[1]);
    off = incl - len[0] - len[1];
    total = __builtin_amdgcn_readlane(incl, 63);
  }
  __device__ __forceinline__ void place(LAS uint8_t* img) {
#pragma unroll
    for (int h = 0; h < 2; ++h) {
      const unsigned o = (off + (h ? len[0] : 0u) + skip[h]) * SB, sh = o & 31;
      const unsigned long long v = (((unsigned long long)hi[h] << 32) | lo[h]) << sh;
      LAS uint8_t* a = img + ((o >> 5) << 2);
      lds_or(a, (unsigned)v); lds_or(a + 4, (unsigned)(v >> 32)); lds_or(a + 8, (hi[h] >> 1) >> (31 - sh));
    }
  }
};

__global__ void __launch_bounds__(BLOCK, 2) ag_gemm_kernel(Params P) {
  extern __shared__ __attribute__((aligned(16))) uint8_t lds[];
  const unsigned lane = threadIdx.x & 63;
  const int wave = __builtin_amdgcn_readfirstlane(threadIdx.x >> 6);
  const int b = blockIdx.x;
  const unsigned ti = min((unsigned)(b >> 3) - 4, 21u), tj = (ti * 43) >> 7;
  uint64_t kx, kws, ktp;
  asm volatile("s_load_dwordx2 %0, %3, 0x0\n\ts_load_dwordx2 %1, %3, 0x28\n\ts_load_dwordx2 %2, %3, %4 offset:0x30"
               : "=&s"(kx), "=&s"(kws), "=&s"(ktp)
               : "s"(__builtin_amdgcn_kernarg_segment_ptr()), "s"(min(tj, 6u) * 8) : "memory");
  static_assert(offsetof(Params, x) == 0 && offsetof(Params, myws) == 0x28 && offsetof(Params, peer) == 0x30, "");
  auto xoff = [&](unsigned u) {
    return (lane & 15) * KD + (lane >> 4) * 16 + ((u >> 6) * 64 + ((u >> 3) & 3) * 16) * KD + ((u & 7) * 4 + ((u >> 5) & 1) * 2) * 128;
  };
  constexpr unsigned TSPLIT = 384;
  const unsigned u0 = b < NSEND ? b + NSEND * max(wave - 1, 0) : TSPLIT + (b - NSEND) + (NPROD - NSEND) * (wave & 3);
  unsigned xo0 = xoff(u0), co = (unsigned)CNT_OFF + b * 64 + wave * 4;
  unsigned to = (ti - 3 * tj) * SLOT + lane * (SH2 / 2 - 4), touched = 0;
  asm volatile("s_waitcnt lgkmcnt(0)" : "+s"(kx), "+s"(kws), "+s"(ktp), "+v"(xo0), "+v"(co), "+v"(to) :: "memory");
  if (__builtin_expect(ti < 21 && wave == 0, 1))
    asm volatile("s_mov_b64 exec, 7\n\tglobal_load_dword %0, %1, %2\n\ts_mov_b64 exec, -1"
                 : "+v"(touched) : "v"(to), "s"(ktp) : "memory");
  uint8_t* const myws = P.myws;
  const uint8_t* const xp = P.x;

  if (__builtin_expect(b < NPROD, 1)) {
    if (wave >= PW && b < NSEND) return;
    u32x4 d[4];
    auto fetch = [&](unsigned o) {
      asm volatile("global_load_dwordx4 %0, %4, %5\n\tglobal_load_dwordx4 %1, %4, %5 offset:64\n\t"
                   "global_load_dwordx4 %2, %4, %5 offset:128\n\tglobal_load_dwordx4 %3, %4, %5 offset:192"
                   : "=&v"(d[0]), "=&v"(d[1]), "=&v"(d[2]), "=&v"(d[3]) : "v"(o), "s"(kx) : "memory");
    };
    unsigned c, call;
    __amdgpu_buffer_rsrc_t pr[R - 1];
    auto open = [&]() {
      fetch(xo0);
      asm volatile("global_load_dword %0, %1, %2 sc1" : "=&v"(c) : "v"(co), "s"(kws) : "memory");
      __builtin_amdgcn_sched_barrier(0);
#pragma unroll
      for (int j = 0; j < R - 1; ++j) pr[j] = mk_rsrc(P.peer[j], (NSLOT - 1) * SLOT + SH2);
    };
    unsigned t[16], e[12], base, key;
    auto pack = [&]() {
#pragma unroll
      for (int j = 0; j < 16; ++j) t[j] = d[j >> 2][j & 3];
      unsigned mx = 0;
#pragma unroll
      for (int j = 0; j < 16; ++j) mx = max(mx, t[j] & 0x7f000000u);
      const unsigned top = (__builtin_amdgcn_readlane(scan_add(mx >> 24), 63) + TOPADD) >> 6;
      base = top > 127 ? 96 : (top > 31 ? top - 31 : 0);
#pragma unroll
      for (int j = 0; j < 16; ++j) t[j] -= base * M01;
#pragma unroll
      for (int g = 0; g < 4; ++g) {
        const unsigned* tt = t + (g >> 1) * 8 + (g & 1) * 3;
        const unsigned z = t[(g >> 1) * 8 + 6 + (g & 1)];
        e[3 * g] = bfi(M01 * 0x60, z << 5, tt[0]);
        e[3 * g + 1] = bfi(M01 * 0x60, z << 3, tt[1]);
        e[3 * g + 2] = bfi(M01 * 0x20, z << 1, bfi(M01 * 0x40, z >> 1, tt[2]));
      }
      key = 0;
      for (;;) {
        key += 0x9E3779B1u;
        unsigned m = 0xffffffffu;
#pragma unroll
        for (int j = 0; j < 12; j += 2) m = umin3(m, e[j] ^ key, e[j + 1] ^ key);
        if (none(m == 0)) break;
      }
    };
#define PIECE(v, off) do { _Pragma("unroll") for (int j = 0; j < R - 1; ++j) \
        __builtin_amdgcn_raw_buffer_store_b128((v), pr[j], lane * 16 + (off), so, AUX_SYS); \
      stored(v); \
      __builtin_amdgcn_sched_barrier(0); } while (0)
    if (__builtin_expect(wave == 0 && b < NSEND, 1)) {
      open();
      unsigned cur = b;
      u32x2 tk = {0u, 0u};
      bool first = true;
      for (;;) {
        if (first) asm volatile("s_waitcnt vmcnt(0)" ::: "memory");
        asm volatile("s_waitcnt vmcnt(15)" : "+v"(c), "+v"(d[0]), "+v"(d[1]), "+v"(d[2]), "+v"(d[3]) :: "memory");
        call = __builtin_amdgcn_readfirstlane(c) + 1;
        const unsigned so = (call % NSLOT) * SLOT + cur * USTR;
        pack();
        const u32x4 pA = {e[0] ^ key, e[1] ^ key, e[2] ^ key, e[3] ^ key};
        const u32x4 pB = {e[4] ^ key, e[5] ^ key, e[6] ^ key, e[7] ^ key};
        const u32x4 pC = {e[8] ^ key, e[9] ^ key, e[10] ^ key, e[11] ^ key};
        asm volatile("s_waitcnt vmcnt(%0)" :: "n"(PACE) : "memory");
        PIECE(pA, 0);
        asm volatile("s_waitcnt vmcnt(14)" : "+v"(tk) :: "memory");
        cur = first ? b + NSEND : __builtin_amdgcn_readfirstlane(tk.x) + 2 * NSEND;
        if (cur >= RFROM) cur += __builtin_amdgcn_readfirstlane(tk.y);
        const bool more = cur < NU;
        if (more) fetch(xoff(cur));
        asm volatile("s_waitcnt vmcnt(%0)" :: "n"(PACE + 4) : "memory");
        PIECE(pB, 1024);
        asm volatile("s_mov_b64 exec, 1\n\tglobal_atomic_add_x2 %0, %1, %2, %3 sc0\n\ts_mov_b64 exec, -1"
                     : "=&v"(tk) : "v"((unsigned)TICK_OFF + (call % NSLOT) * 128), "v"(u32x2{1u, 0u}), "s"(kws) : "memory");
        asm volatile("s_waitcnt vmcnt(%0)" :: "n"(PACE + 5) : "memory");
        PIECE(pC, 2048);
        if (!more) break;
        first = false;
      }
      asm volatile("s_waitcnt vmcnt(7)" : "+v"(tk) :: "memory");
    } else if (b >= NSEND && wave < 4) {
      const unsigned j = b - NSEND, rank = P.rank, pj = (rank + 1 + j) & (R - 1);
      volatile LAS unsigned* job = (volatile LAS unsigned*)lds;
      unsigned* tmo = (unsigned*)(myws + TMO_OFF);
      unsigned p0;
      asm volatile("global_load_dword %0, %1, %2 sc1\n\ts_waitcnt vmcnt(0)" : "=&v"(p0) : "v"(co), "s"(kws) : "memory");
      if (threadIdx.x < 3) job[threadIdx.x] = threadIdx.x ? ~0u : p0 + 1;
      asm volatile("s_waitcnt lgkmcnt(0)" ::: "memory");
      __builtin_amdgcn_s_barrier();
      call = __builtin_amdgcn_readfirstlane(job[0]);
      const unsigned t0 = (unsigned)now();
      bool dead = false;
      auto late = [&]() {
        if (__builtin_amdgcn_readfirstlane(__hip_atomic_load(tmo, __ATOMIC_RELAXED, __HIP_MEMORY_SCOPE_AGENT)) != call &&
            (unsigned)now() - t0 <= DEADLINE) return false;
        __hip_atomic_store(tmo, call, __ATOMIC_RELAXED, __HIP_MEMORY_SCOPE_AGENT);
        return dead = true;
      };
      auto await = [&](int i) {
        unsigned v;
        while ((v = __builtin_amdgcn_readfirstlane(job[i])) == ~0u) __builtin_amdgcn_s_sleep(8);
        return v;
      };
      if (j == R - 1) {
        if (threadIdx.x == 0) job[1] = 0;
        if (threadIdx.x < 2)
          __hip_atomic_store((unsigned*)(myws + TICK_OFF + ((call + 2) % NSLOT) * 128) + lane, 0u, __ATOMIC_RELAXED,
                             __HIP_MEMORY_SCOPE_AGENT);
      } else {
        unsigned* tell = (unsigned*)(P.peer[j] - (size_t)rank * SH2 + PAIR_OFF + rank * 512);
        const unsigned* hear = (const unsigned*)(myws + PAIR_OFF + pj * 512);
        auto say = [&](int i, unsigned v) {
          if (lane == 0) __hip_atomic_store(tell + 32 * i, call << 16 | v, __ATOMIC_RELAXED, __HIP_MEMORY_SCOPE_SYSTEM);
        };
        unsigned t1 = 0;
        auto listen = [&](int i) {
          for (unsigned n = 1;; ++n) {
            const unsigned v = __builtin_amdgcn_readfirstlane(
                __hip_atomic_load(hear + 32 * i, __ATOMIC_RELAXED, __HIP_MEMORY_SCOPE_SYSTEM));
            if (n == 1) t1 = (unsigned)now();
            if (v >> 16 == (call & 0xffffu)) return v & 0xffffu;
            if (dead || ((n & 63) == 0 && late())) return 0u;
          }
        };
        unsigned offer = 0;
        if (wave == 0) {
          say(0, 0);
          listen(0);
          const unsigned dt = min((unsigned)now() - t1, 100000u);
          offer = min(((max(dt, 10u) - 10u) * 27u + 512u) >> 10, RMAX);
          say(1, offer);
          const unsigned m = min(listen(1), RMAX);
          unsigned v = 0;
          if (m) {
            unsigned long long old = 0;
            if (lane == 0)
              old = __hip_atomic_fetch_add((unsigned long long*)(myws + TICK_OFF + (call % NSLOT) * 128),
                                           (unsigned long long)m << 32, __ATOMIC_RELAXED, __HIP_MEMORY_SCOPE_AGENT);
            const unsigned lo = min(max(2 * NSEND + __builtin_amdgcn_readfirstlane((unsigned)old), RFROM) +
                                    __builtin_amdgcn_readfirstlane((unsigned)(old >> 32)), NU);
            v = lo << 5 | min(m, NU - lo);
          }
          say(2, v);
          if (lane == 0) job[1] = v;
        }
        const unsigned snd = await(1);
#pragma unroll
        for (int k = 0; k < R - 1; ++k) pr[k] = mk_rsrc(P.peer[k], k == j ? (NSLOT - 1) * SLOT + SH2 : 0u);
        for (unsigned i = wave; i < (snd & 31); i += 4) {
          const unsigned cur = (snd >> 5) + i, so = (call % NSLOT) * SLOT + cur * USTR;
          fetch(xoff(cur));
          asm volatile("s_waitcnt vmcnt(0)" : "+v"(d[0]), "+v"(d[1]), "+v"(d[2]), "+v"(d[3]) :: "memory");
          pack();
          PIECE((u32x4{e[0] ^ key, e[1] ^ key, e[2] ^ key, e[3] ^ key}), 0);
          PIECE((u32x4{e[4] ^ key, e[5] ^ key, e[6] ^ key, e[7] ^ key}), 1024);
          PIECE((u32x4{e[8] ^ key, e[9] ^ key, e[10] ^ key, e[11] ^ key}), 2048);
        }
        if (wave == 0) {
          const unsigned v = offer ? listen(2) : 0u;
          if (lane == 0) job[2] = v;
        }
        const unsigned rel = await(2);
        const __amdgpu_buffer_rsrc_t in = mk_rsrc(myws + (size_t)pj * SH2, (NSLOT - 1) * SLOT + SH2);
#pragma unroll
        for (int k = 0; k < R - 1; ++k)
          pr[k] = mk_rsrc(P.peer[k] - (size_t)rank * SH2 + (size_t)pj * SH2, k == j ? 0u : (NSLOT - 1) * SLOT + SH2);
        for (unsigned i = wave, n = 1; i < (rel & 31) && !dead; ++n) {
          const unsigned so = (call % NSLOT) * SLOT + ((rel >> 5) + i) * USTR;
          const u32x4 a0 = __builtin_amdgcn_raw_buffer_load_b128(in, lane * 16, so, AUX_SYS);
          const u32x4 a1 = __builtin_amdgcn_raw_buffer_load_b128(in, lane * 16 + 1024, so, AUX_SYS);
          const u32x4 a2 = __builtin_amdgcn_raw_buffer_load_b128(in, lane * 16 + 2048, so, AUX_SYS);
          if (none(umin3(umin3(umin3(a0.x, a0.y, a0.z), umin3(a0.w, a1.x, a1.y), umin3(a1.z, a1.w, a2.x)),
                         min(a2.y, a2.z), a2.w) == 0)) {
            PIECE(a0, 0); PIECE(a1, 1024); PIECE(a2, 2048);
            i += 4;
          } else {
            if ((n & 63) == 0) late();
            __builtin_amdgcn_s_sleep(8);
          }
        }
      }
    } else {
      __builtin_amdgcn_s_sleep(4);
      if (b >= NSEND) {
        __builtin_amdgcn_s_barrier();
        while (__builtin_amdgcn_readfirstlane(((volatile LAS unsigned*)lds)[1]) == ~0u) __builtin_amdgcn_s_sleep(8);
      }
      open();
      const unsigned step = b < NSEND ? NSEND * (PW - 1) : (NPROD - NSEND) * 4, end = b < NSEND ? TSPLIT : NU;
      LAS uint8_t* stg = (LAS uint8_t*)lds + wave * 1024;
      bool first = true;
      for (unsigned cur = u0;;) {
        if (first) asm volatile("s_waitcnt vmcnt(0)" ::: "memory");
        asm volatile("s_waitcnt vmcnt(7)" : "+v"(c), "+v"(d[0]), "+v"(d[1]), "+v"(d[2]), "+v"(d[3]) :: "memory");
        call = __builtin_amdgcn_readfirstlane(c) + 1;
        const unsigned so = (call % NSLOT) * SLOT + cur * USTR;
        pack();
        unsigned X[2], Y[2], w[4] = {0u, 0u, 0u, 0u};
#pragma unroll
        for (int j = 0; j < 16; ++j) w[j >> 2] = bfi((M01 * 3) << (2 * (j & 3)), shl(t[j], 2 * (j & 3) - 5), w[j >> 2]);
        asm volatile("" : "+v"(w[0]), "+v"(w[1]), "+v"(w[2]), "+v"(w[3]));
        cur += step;
        const bool more = cur < end;
        if (more) fetch(xoff(cur));
#pragma unroll
        for (int f = 0; f < 2; ++f) {
          X[f] = bfi(M55, w[2 * f], w[2 * f + 1] << 1);
          Y[f] = bfi(M55, w[2 * f] >> 1, w[2 * f + 1]);
        }
        *(LAS u32x4*)(stg + lane * 16) = u32x4{0u, 0u, 0u, 0u};
        GapList<4> G;
        GapList<8> H;
        G.build(X[0], X[1], lane);
        H.build(X[0] ^ Y[0], X[1] ^ Y[1], lane);
        unsigned l0 = 1 + ((H.total + 7) >> 3);
        unsigned nl = l0 + ((G.total + 15) >> 4);
        const bool dense = G.fail || H.fail || nl > 64;
        unsigned k2 = 0;
        if (!dense) {
          H.place(stg + 8);
          G.place(stg + l0 * 8);
        } else {
          nl = 2; l0 = 1;
          do k2 += 0x9E3779B1u; while (!none(w[0] == k2 || w[1] == k2 || w[2] == k2 || w[3] == k2));
          const u32x4 pD = {w[0] ^ k2, w[1] ^ k2, w[2] ^ k2, w[3] ^ k2};
          PIECE(pD, U_PLANE);
        }
        asm volatile("" ::: "memory");
        u32x4 pT = ~*(const LAS u32x4*)(stg + lane * 16);
        if (lane == 0) {
          pT.x = key;
          pT.y = 0x80000000u | (dense ? 0x40000000u : 0u) | (base << 16) | (l0 << 8) | nl;
          if (dense) pT.z = k2;
        }
        if (lane < ((nl + 3) >> 2) * 2) PIECE(pT, U_T);
        if (!more) break;
        first = false;
      }
      asm volatile("s_waitcnt vmcnt(7)" ::: "memory");
    }
    asm volatile("s_mov_b64 exec, 1\n\tglobal_store_dword %0, %1, %2 sc1\n\ts_mov_b64 exec, -1" :: "v"(co), "v"(call), "s"(kws) : "memory");
    asm volatile("s_waitcnt vmcnt(0)" ::: "memory");
    return;
  }

  const unsigned t0 = (unsigned)now();
  const int rank = P.rank;
  const int ci = (b >> 3) - 4, lo = (b & 7) * 4 + ci - 24;
  const int set = ci < 24 ? (ci >> 3) * 8 + (b & 7) : 24 + (lo >> 3);
  const int nj = ci < 24 ? (ci & 7) : (lo & 7);
  const int s = (rank + 1 + (set >> 2)) & (R - 1);
  const unsigned rb = set & 3;
  unsigned* cntp = (unsigned*)(myws + CNT_OFF + b * 64);
  const unsigned prev = __hip_atomic_load(cntp, __ATOMIC_RELAXED, __HIP_MEMORY_SCOPE_AGENT);
  const float scale = P.sx[0] * P.sw[0];
  LAS unsigned* arrived = (LAS unsigned*)((LAS uint8_t*)lds + LDS_F);
  if (threadIdx.x < 5) arrived[threadIdx.x] = threadIdx.x < 4 ? 0u : ~0u;
  const unsigned q = lane >> 4, li = lane & 15;

  i32x8 wr[4][4];
  {
    const unsigned row = lane >> 2;
    const uint8_t* wsrc = P.w + (size_t)(wave * 512) * NL + nj * 64;
    const unsigned c0 = row * NL + (((lane & 3) ^ ((row >> 2) & 1)) << 4);
    LAS uint8_t* stg = (LAS uint8_t*)lds + LDS_P + wave * 16384;
    auto stage = [&](int kb) {
#pragma unroll
      for (int i = 0; i < 8; ++i) {
        unsigned keep;
        asm volatile("s_mov_b32 %0, m0\n\ts_mov_b32 m0, %3\n\ts_nop 0\n\tglobal_load_lds_dwordx4 %1, %2\n\ts_mov_b32 m0, %0"
                     : "=&s"(keep) : "v"(c0 ^ ((i & 1) << 5)), "s"(wsrc + (kb * 128 + i * 16) * NL),
                       "s"((unsigned)(uintptr_t)(stg + (kb & 1) * 8192 + i * 1024)) : "memory");
      }
    };
    const unsigned f = ((li >> 3) & 1) | ((q & 1) << 1);
    const unsigned lb = (q * 16 + (li >> 1)) * 64 + (li & 1) * 8;
    auto unstage = [&](int kb) {
#pragma unroll
      for (int t = 0; t < 4; ++t)
#pragma unroll
        for (int j = 0; j < 4; ++j) {
          const i32x2 r = __builtin_amdgcn_ds_read_tr8_b64_v2i32(
              (LAS i32x2*)(stg + (kb & 1) * 8192 + lb + ((t ^ f) << 4) + (j >> 1) * 4096 + (j & 1) * 512));
          wr[kb][t][2 * j] = r[0]; wr[kb][t][2 * j + 1] = r[1];
        }
      asm volatile("s_waitcnt lgkmcnt(0)" : "+v"(wr[kb][0]), "+v"(wr[kb][1]), "+v"(wr[kb][2]), "+v"(wr[kb][3]) :: "memory");
    };
    {
      __builtin_amdgcn_s_sleep(8);
      const unsigned z = (b - NPROD) * 8 + wave;
      const __amdgpu_buffer_rsrc_t all = mk_rsrc(P.w, (unsigned)(KD * NL));
      const u32x4 v0 = __builtin_amdgcn_raw_buffer_load_b128(all, lane * 16, z * 1024, 0);
      const u32x4 v1 = __builtin_amdgcn_raw_buffer_load_b128(all, lane * 16, (z + 1792) * 1024, 0);
      asm volatile("" :: "v"(v0), "v"(v1));
      asm volatile("" :: "v"(touched));
    }
    stage(0); stage(1);
    asm volatile("s_waitcnt vmcnt(8)" ::: "memory");
    unstage(0); stage(2);
    asm volatile("s_waitcnt vmcnt(8)" ::: "memory");
    unstage(1); stage(3);
    asm volatile("s_waitcnt vmcnt(8)" ::: "memory");
    unstage(2);
    asm volatile("s_waitcnt vmcnt(0)" ::: "memory");
    unstage(3);
  }
  __builtin_amdgcn_s_barrier();
  asm volatile("" ::: "memory");
  const unsigned call = __builtin_amdgcn_readfirstlane(prev) + 1;
  if (threadIdx.x == 0) __hip_atomic_store(cntp, call, __ATOMIC_RELAXED, __HIP_MEMORY_SCOPE_AGENT);

  unsigned* tmo = (unsigned*)(myws + TMO_OFF);
  const __amdgpu_buffer_rsrc_t orr = mk_rsrc(P.out, (unsigned)(R * MS * NL * 4));
  LAS uint8_t* hb = (LAS uint8_t*)lds + LDS_H + wave * 1024;
  bool dead = false;
  unsigned seq = 0, spins = 0;
  f32x4 acc[4];
#define MMA(xa, kb) do { _Pragma("unroll") for (int t_ = 0; t_ < 4; ++t_) \
    acc[t_] = __builtin_amdgcn_mfma_scale_f32_16x16x128_f8f6f4(wr[kb][t_], (xa), acc[t_], 0, 0, 0, 0, 0, 0); } while (0)
  auto lds_word = [&](unsigned i) { return __builtin_amdgcn_readfirstlane(*(volatile LAS unsigned*)(arrived + i)); };
  unsigned todo = ~0u, todo_row = 0;
  auto finalize = [&](bool block) {
    if (todo == ~0u) return;
    unsigned pa_ = LDS_P + (todo & 1) * 32768 + (wave & 3) * 1024 + lane * 16;
    unsigned so_ = (todo_row * NL + nj * 64 + (wave & 3) * 16) * 4;
    if (block) asm volatile("" : "+v"(pa_), "+s"(so_));
    while (lds_word(todo & 1) < 8 * ((todo >> 1) + 1))
      if (!block) return;
    asm volatile("" ::: "memory");
    __builtin_amdgcn_s_setprio(2);
    const LAS uint8_t* part = (const LAS uint8_t*)lds + pa_;
    f32x4 v[8];
#pragma unroll
    for (int w = 0; w < 8; ++w) v[w] = *(const LAS f32x4*)(part + w * 4096);
    const f32x4 sum = ((v[0] + v[1]) + (v[2] + v[3])) + ((v[4] + v[5]) + (v[6] + v[7]));
    f32x4 o;
#pragma unroll
    for (int r = 0; r < 4; ++r) o[r] = silu(sum[r], scale);
    __builtin_amdgcn_raw_buffer_store_b128(__builtin_bit_cast(u32x4, o), orr, (li * NL + q * 4) * 4, so_, AUX_SYS);
    stored(__builtin_bit_cast(u32x4, o));
    lds_count(arrived + 2 + (todo & 1));
    todo = ~0u;
    __builtin_amdgcn_s_setprio(0);
  };
  bool clear = true;
  auto check_clear = [&]() { if (!clear) clear = lds_word(2 + (seq & 1)) >= 4 * (seq >> 1); };
  auto deposit = [&](unsigned orow) {
    if (!clear) {
      if (todo == seq - 2) finalize(true);
      do check_clear(); while (!clear);
    }
    asm volatile("" ::: "memory");
    if (dead) {
#pragma unroll
      for (int t = 0; t < 4; ++t) acc[t][0] = acc[t][1] = acc[t][2] = acc[t][3] = __builtin_nanf("");
    }
    LAS uint8_t* part = (LAS uint8_t*)lds + LDS_P + (seq & 1) * 32768 + wave * 4096 + lane * 16;
#pragma unroll
    for (int t = 0; t < 4; ++t) *(LAS f32x4*)(part + t * 1024) = acc[t];
    lds_count(arrived + (seq & 1));
    if ((unsigned)(wave >> 2) == (seq & 1)) { todo = seq; todo_row = orow; }
    ++seq;
    clear = seq < 2;
  };

  const __amdgpu_buffer_rsrc_t ar = mk_rsrc(myws + (size_t)(call % NSLOT) * SLOT + (size_t)s * SH2, SH2);
  const uint64_t abase = (uint64_t)(myws + (size_t)(call % NSLOT) * SLOT + (size_t)s * SH2);
  const u32x4 asrd = {(unsigned)abase, (unsigned)(abase >> 32) & 0xffffu, SH2, 0x00020000u};
  const unsigned l16 = lane * 16, l8 = lane * 8;
  const unsigned po0 = (rb * 8 + wave) * USTR;
  unsigned look = (nj - wave) & 7;
  u32x4 pa0, pb0, pc0, pa1, pb1, pc1, w0, w1;
  u32x2 pt0, pt1;
#define REQUEST(c, u, on, pt, pa, pb, pc, w) \
    asm volatile("s_cmp_eq_u32 %9, 0\n\ts_cbranch_scc1 1f\n\t" \
                 "buffer_load_dwordx2 %0, %6, %7, %8 offen offset:3072 sc0 sc1\n\t" \
                 "buffer_load_dwordx4 %4, %5, %7, %8 offen offset:3584 sc0 sc1\n\t" \
                 "buffer_load_dwordx4 %1, %5, %7, %8 offen sc0 sc1\n\t" \
                 "buffer_load_dwordx4 %2, %5, %7, %8 offen offset:1024 sc0 sc1\n\t" \
                 "buffer_load_dwordx4 %3, %5, %7, %8 offen offset:2048 sc0 sc1\n1:" \
                 : c(pt), c(pa), c(pb), c(pc), c(w) \
                 : "v"(l16), "v"(l8), "s"(asrd), "s"(po0 + (u) * (32 * USTR)), "s"(on) : "memory", "scc")
#define LANDED(n, pt, pa, pb, pc, w) \
    asm volatile("s_waitcnt vmcnt(" #n ")" : "+v"(pt), "+v"(pa), "+v"(pb), "+v"(pc), "+v"(w) :: "memory")
#define UNPACK(fr, wa, wb, keyA, keyB, base, x0, x1, x2, x3, x4, x5) { \
      const unsigned base4 = base; \
      i32x8 xa; \
      xa[0] = (int)((bfi(M01 * 0x60, (wa) << 5, x0) ^ keyA) + base4); \
      xa[1] = (int)((bfi(M01 * 0x60, (wa) << 3, x1) ^ keyA) + base4); \
      xa[2] = (int)((bfi(M01 * 0x60, (wa) << 1, x2) ^ keyA) + base4); \
      xa[3] = (int)((bfi(M01 * 0x60, (wa) >> 1, x3) ^ keyA) + base4); \
      xa[4] = (int)((bfi(M01 * 0x60, (wb) << 5, x4) ^ keyA) + base4); \
      xa[5] = (int)((bfi(M01 * 0x60, (wb) << 3, x5) ^ keyA) + base4); \
      xa[6] = (int)((bfi(M01 * 3, (x0) >> 5, bfi(M01 * 0xc, (x1) >> 3, bfi(M01 * 0x10, (x2) >> 1, \
                     bfi(M01 * 0x80, (x2) << 1, (wb) << 1)))) ^ keyB) + base4); \
      xa[7] = (int)((bfi(M01 * 3, (x3) >> 5, bfi(M01 * 0xc, (x4) >> 3, bfi(M01 * 0x10, (x5) >> 1, \
                     bfi(M01 * 0x80, (x5) << 1, (wb) >> 1)))) ^ keyB) + base4); \
      MMA(xa, fr); }
#define FULL(v) none(min(min((v).x, (v).y), min((v).z, (v).w)) == 0)
  const unsigned sink = (unsigned)(uintptr_t)((LAS uint8_t*)lds + LDS_S) + threadIdx.x * 4;
  constexpr unsigned KD_LIST = 0xaaaaaaaau;
  auto lists = [&](const u32x2& pt, u32x4& w) {
    const unsigned meta = __builtin_amdgcn_readlane(pt.y, 0);
    const unsigned nl = meta & 0x7fu, l0 = (meta >> 8) & 0x7fu;
    if ((meta >> 30) != 2 || !none(lane < nl && min(pt.x, pt.y) == 0)) return false;
    *(LAS u32x4*)(hb + l16) = u32x4{0u, 0u, 0u, 0u};
    const bool on = lane - 1 < nl - 1, isb = lane < l0;
    const unsigned lo = on ? pt.x : ~0u, hi = on ? pt.y : ~0u;
    const unsigned F = M01 * 15;
    const unsigned tn = __builtin_amdgcn_sad_u8((lo & F) + (hi & F) + ((lo >> 4) & F) + ((hi >> 4) & F), 0u, 0u);
    const unsigned tb = __builtin_amdgcn_sad_u8(lo, 0u, __builtin_amdgcn_sad_u8(hi, 0u, 0u));
    const unsigned tot = lane == 0 ? 0u : (isb ? tb : tn);
    const unsigned incl = scan_add(tot);
    unsigned cur = (incl - tot - 1 + (isb ? 4096u : 0u - __builtin_amdgcn_readlane(incl, l0 - 1))) * 8193u +
                   ((unsigned)(uintptr_t)hb << 16);
    const unsigned sw = isb ? 8u : 4u, esc = isb ? 255u : 15u;
    const unsigned src[4] = {lo, isb ? hi : lo >> 16, isb ? ~0u : hi, isb ? ~0u : hi >> 16};
#pragma unroll
    for (int i = 0; i < 16; ++i) {
      const unsigned sy = __builtin_amdgcn_ubfe(src[i >> 2], sw * (i & 3), sw);
      cur += __umul24(sy, 8193u);
      __hip_atomic_fetch_xor((LAS unsigned*)(uintptr_t)(sy != esc ? (cur >> 16) & 0xfffcu : sink), 1u << (cur & 31),
                             __ATOMIC_RELAXED, __HIP_MEMORY_SCOPE_WAVEFRONT);
    }
    asm volatile("" ::: "memory");
    const u32x2 X = *(const LAS u32x2*)(hb + l8);
    const u32x2 Y = X ^ *(const LAS u32x2*)(hb + 512 + l8);
    w = u32x4{bfi(M55, X.x, Y.x << 1), bfi(M55, X.x >> 1, Y.x), bfi(M55, X.y, Y.y << 1), bfi(M55, X.y >> 1, Y.y)};
    return true;
  };
  auto leave = [&](const u32x4& w, unsigned uo) {
    const u32x4 v = w ^ KD_LIST;
    __builtin_amdgcn_raw_buffer_store_b128(v, ar, l16 + U_PLANE, uo, AUX_SYS);
    stored(v);
  };
  auto look_load = [&]() { return __builtin_amdgcn_raw_buffer_load_b64(ar, l8 + U_T, po0 + look * (32 * USTR), AUX_SYS); };
  auto look_decode = [&](const u32x2& pt) {
    u32x4 w;
    __builtin_amdgcn_s_setprio(3);
    const bool ok = lists(pt, w);
    __builtin_amdgcn_s_setprio(0);
    if (ok) { leave(w, po0 + look * (32 * USTR)); look += 8; }
    else if (__builtin_amdgcn_readlane(pt.y, 0) >> 30 == 3) look += 8;
  };
  if (threadIdx.x == 0)
    *(volatile LAS unsigned*)(arrived + 4) = __hip_atomic_fetch_add((unsigned*)(myws + XTRA_OFF + nj * 128), 1u,
        __ATOMIC_RELAXED, __HIP_MEMORY_SCOPE_AGENT) - (call - 1) * 28;
  look_decode(look_load());
  {
    const __amdgpu_buffer_rsrc_t xr = mk_rsrc(xp, SHARD);
#pragma unroll 1
    for (unsigned lblk = set; lblk < 32;) {
      u32x4 a[8];
#pragma unroll
      for (int i = 0; i < 8; ++i)
        a[i] = __builtin_amdgcn_raw_buffer_load_b128(xr, li * KD + q * 16 + i * 64, lblk * 16 * KD + wave * 512, 0);
      __builtin_amdgcn_sched_barrier(0);
#pragma unroll
      for (int t = 0; t < 4; ++t) acc[t] = f32x4{0.f, 0.f, 0.f, 0.f};
#pragma unroll
      for (int i = 0; i < 4; ++i) {
        const i32x8 xa = {(int)a[2 * i].x, (int)a[2 * i].y, (int)a[2 * i].z, (int)a[2 * i].w,
                          (int)a[2 * i + 1].x, (int)a[2 * i + 1].y, (int)a[2 * i + 1].z, (int)a[2 * i + 1].w};
        MMA(xa, i);
      }
      deposit(rank * MS + lblk * 16);
      if (lblk >= 28) break;
      unsigned k;
      while ((k = lds_word(4)) == ~0u) { }
      lblk = 28 + k;
    }
  }

#pragma unroll
  for (int t = 0; t < 4; ++t) acc[t] = f32x4{0.f, 0.f, 0.f, 0.f};
  const unsigned pconst = lane == 0 ? 2044 : (lane == 1 ? U_PLANE + 1020 : (lane == 2 ? U_T + 4 : SH2));
  auto probe = [&](unsigned u) {
    return __builtin_amdgcn_raw_buffer_load_b32(ar, pconst + (lane == 2 ? look * (32 * USTR) : u * (32 * USTR)), po0, AUX_SYS);
  };
  unsigned key, meta;
#define ACQUIRE(u, asked, pt, pa, pb, pc, fw) { \
    unsigned tries = 0; \
    for (;;) { \
      if (!asked) for (;;) { \
        const unsigned v = probe(u); \
        check_clear(); \
        const unsigned seen = (unsigned)__builtin_amdgcn_ballot_w64(v != 0); \
        if ((seen & 1u) && ((seen & 2u) || look == (u) || ++tries >= DLIMIT)) break; \
        if ((seen & 4u) && look < 16) look_decode(look_load()); \
        if ((++spins & 63) == 0 && timed_out()) dead = true; \
        if (dead) break; \
        finalize(false); \
        if ((u) < 13) __builtin_amdgcn_s_sleep(4); \
      } \
      REQUEST("+v", u, asked ^ 1u, pt, pa, pb, pc, fw); \
      LANDED(0, pt, pa, pb, pc, fw); \
      const bool data = none(umin3(umin3(pa.x, pa.y, pa.z), umin3(pa.w, pb.x, pb.y), min(pb.z, pb.w)) == 0); \
      const bool flds = FULL(fw); \
      key = __builtin_amdgcn_readlane(pt.x, 0); meta = __builtin_amdgcn_readlane(pt.y, 0); \
      unsigned kd = KD_LIST; \
      if (__builtin_expect(meta & 0x40000000u, 0)) kd = __builtin_amdgcn_readlane(pt.x, 1); \
      bool done = dead; \
      if (data && min(min(key, meta), kd) != 0) { \
        if (__builtin_expect(flds, 1)) { fw ^= kd; done = true; } \
        else if (lists(pt, fw)) { \
          done = true; \
          if (look == (u)) leave(fw, po0 + (u) * (32 * USTR)); \
        } \
      } \
      if (done) break; \
      if ((++spins & 63) == 0 && timed_out()) dead = true; \
      asked = 0; \
    } \
    if (look == (u)) look += 8; }
#define UNIT(i, pt, pa, pb, pc, fw, qt, qa, qb, qc, qw) { \
    const unsigned u = 2 * p + (i); \
    const unsigned pv = probe(u + 1); \
    if (u < 14) finalize(false); \
    if (u - 1 < 12) rezero(u - 1); \
    __builtin_amdgcn_s_setprio(2); \
    unsigned ka = key & ~(M01 * 0x60), ba = ((meta >> 16) & 0x7fu) * M01; \
    unsigned kb = ((key >> 5) & (M01 * 3)) | ((key >> 3) & (M01 * 0xc)) | ((key >> 1) & (M01 * 0x10)) | ((key << 1) & (M01 * 0x80)); \
    asm volatile("" : "+s"(ka), "+s"(kb), "+v"(ba));        \
    UNPACK(2 * (i), fw.x, fw.y, ka, kb, ba, pa.x, pa.y, pa.z, pa.w, pb.x, pb.y) \
    __builtin_amdgcn_sched_barrier(0); \
    while (!FULL(pc) && !dead) {                            \
      asm volatile("buffer_load_dwordx4 %0, %1, %2, %3 offen offset:2048 sc0 sc1\n\ts_waitcnt vmcnt(0)" \
                   : "=&v"(pc) : "v"(l16), "s"(asrd), "s"(po0 + u * (32 * USTR)) : "memory"); \
      if ((++spins & 63) == 0 && timed_out()) dead = true; \
    } \
    const unsigned ahead = (unsigned)__builtin_amdgcn_ballot_w64(pv != 0); \
    unsigned asked = ((ahead & 3u) + 1u) >> 2; \
    REQUEST("=&v", u + 1, asked, qt, qa, qb, qc, qw); \
    __builtin_amdgcn_sched_barrier(0); \
    UNPACK(2 * (i) + 1, fw.z, fw.w, ka, kb, ba, pb.z, pb.w, pc.x, pc.y, pc.z, pc.w) \
    if (i) { \
      deposit(s * MS + p * 64 + rb * 16); \
      _Pragma("unroll") for (int t = 0; t < 4; ++t) acc[t] = f32x4{0.f, 0.f, 0.f, 0.f}; \
    } \
    __builtin_amdgcn_s_setprio(0); \
    if (u == 15) break; \
    if ((ahead & 4u) && look < 16 && look != u + 1) look_decode(look_load()); \
    ACQUIRE(u + 1, asked, qt, qa, qb, qc, qw) }
  auto timed_out = [&]() {
    if (__builtin_amdgcn_readfirstlane(__hip_atomic_load(tmo, __ATOMIC_RELAXED, __HIP_MEMORY_SCOPE_AGENT)) != call &&
        (unsigned)now() - t0 <= DEADLINE) return false;
    __hip_atomic_store(tmo, call, __ATOMIC_RELAXED, __HIP_MEMORY_SCOPE_AGENT);
    return true;
  };
  auto rezero = [&](unsigned part) {
    const unsigned z = (set * 8 + nj) * 8 + wave;
    const __amdgpu_buffer_rsrc_t zr = mk_rsrc(myws + (size_t)((call + 2) % NSLOT) * SLOT, SLOT);
    const unsigned so = ((rank + 1 + (z >> 8)) & (R - 1)) * SH2 + (z & 255) * (2 * USTR) + part * 768;
    u32x4 zero = {0u, 0u, 0u, 0u};
    asm volatile("" : "+v"(zero));
    __builtin_amdgcn_raw_buffer_store_b128(zero, zr, l16 < 768 ? l16 : SLOT, so, AUX_SYS);
    stored(zero);
  };
  {
    unsigned asked = 1;
    REQUEST("=&v", 0u, asked, pt0, pa0, pb0, pc0, w0);
    ACQUIRE(0u, asked, pt0, pa0, pb0, pc0, w0)
  }
#pragma unroll 1
  for (unsigned p = 0;; ++p) {
    UNIT(0, pt0, pa0, pb0, pc0, w0, pt1, pa1, pb1, pc1, w1)
    UNIT(1, pt1, pa1, pb1, pc1, w1, pt0, pa0, pb0, pc0, w0)
  }
  finalize(true);
  asm volatile("s_waitcnt vmcnt(0)" ::: "memory");
}

extern "C" void kernel_launch(int rank, int nranks, ncclComm_t comm, hipStream_t stream,
                              void* const* d_in, const int* in_sizes, int n_in,
                              void* d_out, int out_size, const int* in_dtypes, int out_dtype,
                              void* const* d_out_all, void* const* d_ws_all, size_t ws_bytes) {
  (void)comm; (void)in_dtypes; (void)out_dtype; (void)d_out_all;
  if (nranks != R || (unsigned)rank >= (unsigned)R || n_in != 4 || ws_bytes < WS_NEED ||
      in_sizes[0] != (int)SHARD || in_sizes[1] != KD * NL || out_size != R * MS * NL) {
    fprintf(stderr, "unexpected problem shape\n");
    return;
  }
  static bool attr_set = false;
  if (!attr_set) {
    HIP_CHECK(hipFuncSetAttribute((const void*)ag_gemm_kernel, hipFuncAttributeMaxDynamicSharedMemorySize, LDS_BYTES));
    attr_set = true;
  }
  Params P;
  P.x = (const uint8_t*)d_in[0];
  P.w = (const uint8_t*)d_in[1];
  P.sx = (const float*)d_in[2];
  P.sw = (const float*)d_in[3];
  P.out = (float*)d_out;
  P.myws = (uint8_t*)d_ws_all[rank];
  for (int j = 0; j < R - 1; ++j) P.peer[j] = (uint8_t*)d_ws_all[(rank + 1 + j) & (R - 1)] + (size_t)rank * SH2;
  P.rank = rank;
  P.pad = 0;
  hipLaunchKernelGGL(ag_gemm_kernel, dim3(GRID), dim3(BLOCK), LDS_BYTES, stream, P);
  HIP_CHECK(hipGetLastError());
}
```
